```python
import numpy as np

kernel_name = "gpu_gemm_ar_m4096_k4096_n4096_f32_q8"


ABSMAX_RTOL = 0.02

COMPUTE_DTYPE = "float32"
EPILOGUE = "quant_int8"
IN_QUANT = None
OUT_QUANT = "int8"
IN_DTYPES = {"x": "float32", "w_mat": "float32"}
WORLD_SIZE = 8

def setup_inputs():

    x = np.random.randn(4096, 4096).astype(np.float32)
    w_mat = (np.random.randn(4096, 4096) / np.sqrt(4096)).astype(np.float32)
    return {"x": x, "w_mat": w_mat}

def _quant_dequant(y, amax):
    scale = amax / 127.0
    q = np.clip(np.round(y / scale), -127.0, 127.0)
    return (q * scale).astype(np.float32)
def _epilogue(y):
    return _quant_dequant(y, np.abs(y).max())

def reference(x, w_mat):
    y = (x @ w_mat).astype(np.float32)
    return _epilogue(y)
```

```cpp
#include <hip/hip_runtime.h>
#include <rccl/rccl.h>
#include <cstdio>
#include <cstdint>

#define STAMPS 0

namespace {

#define GAS __attribute__((address_space(1)))
#define LAS __attribute__((address_space(3)))
typedef unsigned u32;
typedef unsigned long long u64;
typedef unsigned v4u __attribute__((ext_vector_type(4)));
typedef unsigned v2u __attribute__((ext_vector_type(2)));
typedef float f32x4 __attribute__((ext_vector_type(4)));
typedef float f32x2 __attribute__((ext_vector_type(2)));
typedef short bf16x8 __attribute__((ext_vector_type(8)));
typedef __bf16 bf16x2 __attribute__((ext_vector_type(2)));

constexpr int NR = 8;
constexpr int MM = 4096, NN = 4096, KS = 512;
constexpr int NGEMM = 128;
constexpr int NPW = 16;
constexpr int NMOV = 7 * NPW;
constexpr int NWG = NGEMM + NMOV;
constexpr int NTASK = 320;
constexpr int TPW = NTASK / NPW;
constexpr int NOUT = 512;
constexpr int OPWV = NOUT / NPW;
constexpr int NSTEP = 65;
static_assert(NTASK % NPW == 0 && NOUT % NPW == 0 && TPW <= 64 && OPWV <= 64 && NWG <= 256, "task split");

constexpr u32 BLK = 8192;
constexpr u32 OFF_CNT = 0;
constexpr u32 OFF_TMO = 8192;
constexpr u32 OFF_AM = 16384;
constexpr u32 OFF_GF = 32768;
constexpr u32 OFF_GP = OFF_GF + 8192;
constexpr u32 OFF_PF = 65536;
constexpr u32 PF_SRC = 16384;
constexpr u32 PF_P = 512, PF_O = 1024;
constexpr u32 OFF_STAMP = OFF_PF + 8 * PF_SRC;
constexpr u32 OFF_EXP = 1u << 20;
constexpr u32 OFF_EXPP = OFF_EXP + 1024 * BLK;
constexpr u32 OFF_OP = OFF_EXPP + 512 * BLK;
constexpr u32 OFF_OPP = OFF_OP + 2048 * BLK;
constexpr u32 OFF_EXPO = OFF_OPP + 512 * BLK;
constexpr u32 WS_USED = OFF_EXPO + 512 * 4096;
static_assert(OFF_GP + 128 * 128 <= OFF_PF && OFF_STAMP + 256 * 8 * 32 * 8 <= OFF_EXP, "control area");

constexpr u64 DEADLINE = 100000000ull;
constexpr int AUX_SYS = 17;
constexpr int RSRC3 = 0x00020000;

constexpr int NSLOT = 4;
constexpr u32 L_RING = 0;
constexpr u32 L_CTL = NSLOT * 4 * BLK;
constexpr u32 L_FULL = L_CTL, L_FREE = L_CTL + 16, L_ABORT = L_CTL + 32, L_MAX1 = L_CTL + 36, L_CNT1 = L_CTL + 40,
              L_MAX2 = L_CTL + 44, L_CNT2 = L_CTL + 48;
constexpr u32 LDS_BYTES = L_CTL + 64;

struct Tables {
    unsigned short pull[7][NTASK];
    unsigned short conv[1024];
    unsigned char step[64][NSTEP + 3];
};
constexpr int w_order(int nch, int (&gs)[40], int (&cs)[40]) {
    int n = 0;
    for (int key = 0; key <= 7 + nch - 1; ++key)
        for (int c = nch - 1; c >= 0; --c) {
            int g = key - c;
            if (g < 0 || g > 7) continue;
            gs[n] = g; cs[n] = c; ++n;
        }
    return n;
}
constexpr Tables make_tables() {
    Tables t{};
    int posX[7][8][32] = {}, posW[7][8][32] = {}, posP[7][128] = {};
    for (int L = 0; L < 7; ++L) {
        const bool in = L < 3;
        const int nch = in ? 5 : 3, c0 = in ? 0 : 5;
        int n = 0;
        for (int c = 0; c < nch; ++c)
            for (int l = 0; l < 16; ++l) { posX[L][c0 + c][l] = n; t.pull[L][n++] = (unsigned short)(((c0 + c) << 5) | l); }
        int gs[40] = {}, cs[40] = {};
        const int nw = w_order(nch, gs, cs);
        for (int e = 0; e < nw; ++e)
            for (int k = 0; k < 4; ++k) {
                const int l = (k >> 1) * 16 + 2 * gs[e] + (k & 1);
                posW[L][c0 + cs[e]][l] = n;
                t.pull[L][n++] = (unsigned short)(256 | ((c0 + cs[e]) << 5) | l);
            }
        for (int c = 0; c < nch; ++c)
            for (int l = 0; l < 16; ++l) { posX[L][c0 + c][16 + l] = n; t.pull[L][n++] = (unsigned short)(((c0 + c) << 5) | (16 + l)); }
        if (!in)
            for (int g = 0; g < 8; ++g)
                for (int r = 0; r < 4; ++r)
                    for (int cn = 0; cn < 4; ++cn) {
                        const int id = (r << 5) | (g << 2) | cn;
                        posP[L][id] = n;
                        t.pull[L][n++] = (unsigned short)(512 | id);
                    }
    }
    {
        int n = 0;
        for (int pos = 0; pos < NTASK; ++pos)
            for (int L = 0; L < 4; L += 3) {
                const int code = t.pull[L][pos];
                if (code & 512) continue;
                const int kind = code >> 8, c = (code >> 5) & 7, l = code & 31;
                for (int ij = 0; ij < 2; ++ij) t.conv[n++] = (unsigned short)(kind * 512 + c * 64 + ij * 32 + l);
            }
    }
    for (int rt = 0; rt < 8; ++rt)
        for (int g = 0; g < 8; ++g) {
            int key[NSTEP] = {}, code[NSTEP] = {};
            int n = 0;
            for (int so = 0; so < 2; ++so)
                for (int kc = 0; kc < 32; ++kc) {
                    int k = 0;
                    if (kc < 5) k = -100 + kc * 2 + so;
                    else {
                        const int L = kc < 20 ? kc / 5 - 1 : 3 + (kc - 20) / 3;
                        const int c = kc < 20 ? kc % 5 : 5 + (kc - 20) % 3;
                        const int a = posX[L][c][so * 16 + 2 * rt + 1], b = posW[L][c][16 + 2 * g + 1];
                        k = a > b ? a : b;
                    }
                    key[n] = k * 8 + so; code[n] = (so << 5) | kc; ++n;
                }
            key[n] = posP[3][((rt & 3) << 5) | (g << 2) | 3] * 8 + 2; code[n] = 2 << 5; ++n;
            for (int a = 1; a < NSTEP; ++a) {
                const int kk = key[a], cc = code[a];
                int b = a - 1;
                while (b >= 0 && key[b] > kk) { key[b + 1] = key[b]; code[b + 1] = code[b]; --b; }
                key[b + 1] = kk; code[b + 1] = cc;
            }
            for (int a = 0; a < NSTEP; ++a) t.step[rt * 8 + g][a] = (unsigned char)code[a];
        }
    return t;
}
constexpr bool tables_ok() {
    const Tables t = make_tables();
    for (int w = 0; w < 64; ++w) {
        int lastS = -1, sP = -1;
        for (int s = 0; s < NSTEP; ++s) {
            if ((t.step[w][s] >> 5) == 0) lastS = s;
            if ((t.step[w][s] >> 5) == 2) sP = s;
        }
        if (lastS < 1 || lastS > 63 || sP <= lastS) return false;
    }
    return true;
}
static_assert(tables_ok(), "the partial step must follow the last S step");
__constant__ const Tables TB = make_tables();

struct Params {
    unsigned char* ws[NR];
    const float* x;
    const float* w;
    signed char* out;
    int rank;
    int pad;
};

__device__ __forceinline__ u64 rt_now() { return __builtin_amdgcn_s_memrealtime(); }
__device__ __forceinline__ u32 rfl(u32 v) { return __builtin_amdgcn_readfirstlane(v); }
__device__ __forceinline__ u32 rlane(u32 v, u32 l) { return __builtin_amdgcn_readlane(v, l); }
__device__ __forceinline__ __amdgpu_buffer_rsrc_t mkrs(const void* p, u32 bytes) {
    return __builtin_amdgcn_make_buffer_rsrc((void*)p, 0, (int)bytes, RSRC3);
}
__device__ __forceinline__ v4u ld128(__amdgpu_buffer_rsrc_t rs, u32 voff, u32 soff) {
    return __builtin_amdgcn_raw_buffer_load_b128(rs, voff, soff, AUX_SYS);
}
__device__ __forceinline__ void st128(v4u v, __amdgpu_buffer_rsrc_t rs, u32 voff, u32 soff) {
    __builtin_amdgcn_raw_buffer_store_b128(v, rs, voff, soff, AUX_SYS);
}
#define VM_WAIT(n) asm volatile("s_waitcnt vmcnt(" #n ")" ::: "memory")
#define LGKM0() asm volatile("s_waitcnt lgkmcnt(0)" ::: "memory")
#define SCHED() __builtin_amdgcn_sched_barrier(0)
__device__ __forceinline__ u32 cvt2(float a, float b) {
    return __builtin_bit_cast(u32, __builtin_convertvector(f32x2{a, b}, bf16x2));
}
__device__ __forceinline__ u32 perm(u32 s0, u32 s1, u32 sel) { return __builtin_amdgcn_perm(s0, s1, sel); }
__device__ __forceinline__ u32 wave_max_u32(u32 v) {
    u32 t;
    t = __builtin_amdgcn_update_dpp(0u, v, 0x111, 0xf, 0xf, true); v = v > t ? v : t;
    t = __builtin_amdgcn_update_dpp(0u, v, 0x112, 0xf, 0xf, true); v = v > t ? v : t;
    t = __builtin_amdgcn_update_dpp(0u, v, 0x114, 0xf, 0xf, true); v = v > t ? v : t;
    t = __builtin_amdgcn_update_dpp(0u, v, 0x118, 0xf, 0xf, true); v = v > t ? v : t;
    t = __builtin_amdgcn_update_dpp(0u, v, 0x142, 0xa, 0xf, false); v = v > t ? v : t;
    t = __builtin_amdgcn_update_dpp(0u, v, 0x143, 0xc, 0xf, false); v = v > t ? v : t;
    return rlane(v, 63);
}
__device__ __forceinline__ void lds_st(u32 addr, u32 v) { asm volatile("ds_write_b32 %0, %1" ::"v"(addr), "v"(v) : "memory"); }
__device__ __forceinline__ v4u lds_ld4(u32 addr) {
    v4u v;
    asm volatile("ds_read_b128 %0, %1\n\ts_waitcnt lgkmcnt(0)" : "=&v"(v) : "v"(addr) : "memory");
    return v;
}
__device__ __forceinline__ u32 lds_ld(u32 addr) {
    u32 v;
    asm volatile("ds_read_b32 %0, %1\n\ts_waitcnt lgkmcnt(0)" : "=&v"(v) : "v"(addr) : "memory");
    return v;
}
__device__ __forceinline__ void glds4(const void* gsrc, u32 lds_dst) {
    u32 keep;
    asm volatile(
        "s_mov_b32 %0, m0\n\ts_mov_b32 m0, %2\n\ts_nop 0\n\t"
        "global_load_lds_dwordx4 %1, off sc0 sc1\n\t"
        "global_load_lds_dwordx4 %1, off offset:1024 sc0 sc1\n\t"
        "global_load_lds_dwordx4 %1, off offset:2048 sc0 sc1\n\t"
        "global_load_lds_dwordx4 %1, off offset:3072 sc0 sc1\n\t"
        "s_mov_b32 m0, %0"
        : "=&s"(keep) : "v"(gsrc), "s"(lds_dst) : "memory");
}

struct Ctx {
    unsigned char* me;
    u64 t0;
    u32 call;
    int lane;
    bool dead;
};
__device__ __forceinline__ bool give_up(Ctx& c) {
    GAS u32* tmo = (GAS u32*)(c.me + OFF_TMO);
    const u32 t = rfl(__hip_atomic_load(tmo, __ATOMIC_RELAXED, __HIP_MEMORY_SCOPE_AGENT));
    if (t == c.call || rt_now() - c.t0 > DEADLINE) {
        __hip_atomic_store(tmo, c.call, __ATOMIC_RELAXED, __HIP_MEMORY_SCOPE_AGENT);
        c.dead = true;
        return true;
    }
    return false;
}

#define STAMP(i) do {} while (0)

__device__ __forceinline__ u32 encode_block(const u32 (&D)[32], u32 maxbits, __amdgpu_buffer_rsrc_t rs, u32 soff, int lane) {
    const u32 mx = wave_max_u32(maxbits);
    const u32 e7 = (cvt2(__builtin_bit_cast(float, mx), 0.f) >> 8) & 0x7Fu;
    const u32 base = rfl(e7 > 7u ? e7 - 7u : 0u);
    const u32 base4 = base * 0x01010101u;
    u32 L[16], H[16], Nn[8];
    u32 anybad = 0;
#pragma unroll
    for (int f = 0; f < 8; ++f) {
        u32 cc[2];
#pragma unroll
        for (int h = 0; h < 2; ++h) {
            const u32 d0 = D[4 * f + 2 * h], d1 = D[4 * f + 2 * h + 1];
            L[2 * f + h] = perm(d1, d0, 0x06040200u);
            const u32 hh = perm(d1, d0, 0x07050301u);
            H[2 * f + h] = hh;
            const u32 t = (hh | 0x80808080u) - base4;
            anybad |= (t ^ 0x80808080u) & 0xF8F8F8F8u;
            cc[h] = (hh & 0x80808080u) | (t & 0x07070707u);
        }
        Nn[f] = cc[0] | __builtin_amdgcn_alignbit(cc[1], cc[1], 28);
    }
    u32 cnt = 0;
    if (__builtin_amdgcn_ballot_w64(anybad != 0) != 0) {
#pragma unroll
        for (int i = 0; i < 16; ++i) {
            const u32 x = ((((H[i] | 0x80808080u) - base4) ^ 0x80808080u) & 0xF8F8F8F8u);
            if (__builtin_amdgcn_ballot_w64(x != 0) != 0) {
#pragma unroll 1
                for (int b = 0; b < 4; ++b) {
                    const bool bad = ((x >> (8 * b)) & 0xFFu) != 0;
                    const u64 m = __builtin_amdgcn_ballot_w64(bad);
                    const u32 idx = cnt + __builtin_amdgcn_mbcnt_hi((u32)(m >> 32), __builtin_amdgcn_mbcnt_lo((u32)m, 0u));
                    const u32 ent = ((((u32)i << 8) | ((u32)b << 6) | (u32)lane) << 8) | ((H[i] >> (8 * b)) & 0xFFu);
                    if (bad && idx < 64u) __builtin_amdgcn_raw_buffer_store_b32(ent, rs, 6144u + idx * 4u, soff, AUX_SYS);
                    cnt += (u32)__builtin_popcountll(m);
                }
            }
        }
    }
    const u32 vo = (u32)lane * 16u;
    if (cnt <= 64u) {
#pragma unroll
        for (int u = 0; u < 2; ++u) {
            st128(v4u{L[8 * u], L[8 * u + 1], L[8 * u + 2], L[8 * u + 3]}, rs, vo + u * 3072, soff);
            st128(v4u{L[8 * u + 4], L[8 * u + 5], L[8 * u + 6], L[8 * u + 7]}, rs, vo + u * 3072 + 1024, soff);
            st128(v4u{Nn[4 * u], Nn[4 * u + 1], Nn[4 * u + 2], Nn[4 * u + 3]}, rs, vo + u * 3072 + 2048, soff);
        }
        return base | (cnt << 7);
    }
    VM_WAIT(0);
#pragma unroll
    for (int f = 0; f < 8; ++f) st128(v4u{D[4 * f], D[4 * f + 1], D[4 * f + 2], D[4 * f + 3]}, rs, vo + f * 1024, soff);
    return 1u << 14;
}

__device__ __forceinline__ void st_flag64(unsigned char* p, u32 call, u32 info) {
    __hip_atomic_store((GAS u64*)p, ((u64)call << 32) | info, __ATOMIC_RELAXED, __HIP_MEMORY_SCOPE_SYSTEM);
}
__device__ __forceinline__ void st_flag32(unsigned char* p, u32 call) {
    __hip_atomic_store((GAS u32*)p, call, __ATOMIC_RELAXED, __HIP_MEMORY_SCOPE_SYSTEM);
}
__device__ __forceinline__ u32 row_block(u32 G, u32 i, u32 l) {
    const u32 so = l >> 4, q = (l >> 3) & 1, idx = l & 7;
    const u32 a = G ? i + 2 * q : 2 * i + q, hf = G ? 1 - so : so;
    return 16 * a + 8 * hf + idx;
}
__device__ __forceinline__ u32 col_block(u32 G, u32 j, u32 l) {
    const u32 q = l >> 4, idx = l & 15;
    return 16 * (G ? j + 2 * q : 2 * j + q) + idx;
}

__global__ void __launch_bounds__(512, 2) fused_gemm_ar_q8(Params p) {
    __shared__ __attribute__((aligned(16))) unsigned char smem[LDS_BYTES];
    const int tid = threadIdx.x, lane = tid & 63;
    const u32 wave = rfl(tid >> 6);
    const u32 bid = blockIdx.x;
    const u32 rank = p.rank, G = rank >> 2, U = rank & 3, myi = U >> 1, myj = U & 1;
    Ctx cx;
    cx.me = p.ws[rank];
    cx.t0 = rt_now();
    cx.lane = lane;
    cx.dead = false;
    const bool is_gemm = bid < (u32)NGEMM;
    if (!is_gemm && wave != 0) return;
    GAS u32* cntp = (GAS u32*)(cx.me + OFF_CNT) + bid * 8 + wave;
    const u32 cnt_v = __hip_atomic_load(cntp, __ATOMIC_RELAXED, __HIP_MEMORY_SCOPE_AGENT);
    const __amdgpu_buffer_rsrc_t rsMe = mkrs(cx.me, WS_USED);
    const u32 vo16 = (u32)lane * 16u;
    STAMP(0);

    if (is_gemm) {
        const u32 lds0 = (u32)(size_t)(LAS unsigned char*)smem;
        if (tid < 16) lds_st(lds0 + L_CTL + tid * 4, 0u);
        LGKM0();
        __builtin_amdgcn_s_barrier();
        asm volatile("" ::: "memory");
        {
            const u32 slot = TB.conv[wave * NGEMM + bid];
            const u32 kind = slot >> 9, c = (slot >> 6) & 7, ij = (slot >> 5) & 1, l = slot & 31;
            const u32 dG = c < 5 ? G : 1 - G;
            for (u32 s = 0; s < wave; ++s) __builtin_amdgcn_s_sleep(12);
            u32 D[32];
            float mx = 0.f;
            const u32 r16 = lane & 15, g4 = lane >> 4;
            if (kind == 0) {
                const u32 rb = row_block(dG, ij, l);
                const GAS f32x4* src = (const GAS f32x4*)(p.x + (size_t)(64 * rb + r16) * KS + 64 * c + 4 * g4);
                f32x4 v[16];
#pragma unroll
                for (int mt = 0; mt < 4; ++mt)
#pragma unroll
                    for (int i = 0; i < 4; ++i) v[mt * 4 + i] = __builtin_nontemporal_load(src + mt * 16 * (KS / 4) + i * 4);
                SCHED();
#pragma unroll
                for (int mt = 0; mt < 4; ++mt)
#pragma unroll
                    for (int i = 0; i < 4; ++i) {
                        const f32x4 a = v[mt * 4 + i];
                        const int f = (i >> 1) * 4 + mt, o = 4 * f + 2 * (i & 1);
                        D[o] = cvt2(a.x, a.y);
                        D[o + 1] = cvt2(a.z, a.w);
                        mx = fmaxf(fmaxf(mx, fabsf(a.x)), fabsf(a.y));
                        mx = fmaxf(fmaxf(mx, fabsf(a.z)), fabsf(a.w));
                    }
            } else {
                const u32 cb = col_block(dG, ij, l);
                const GAS f32x4* src = (const GAS f32x4*)(p.w + (size_t)(64 * c + 4 * g4) * NN + 64 * cb + 4 * r16);
                f32x4 v[16];
#pragma unroll
                for (int j = 0; j < 16; ++j) v[j] = __builtin_nontemporal_load(src + ((j >> 2) * 16 + (j & 3)) * (NN / 4));
                SCHED();
#pragma unroll
                for (int q = 0; q < 2; ++q)
#pragma unroll
                    for (int pp = 0; pp < 4; ++pp) {
                        const f32x4 a = v[q * 8 + 2 * pp], b = v[q * 8 + 2 * pp + 1];
#pragma unroll
                        for (int t = 0; t < 4; ++t) {
                            D[4 * (q * 4 + t) + pp] = cvt2(a[t], b[t]);
                            mx = fmaxf(fmaxf(mx, fabsf(a[t])), fabsf(b[t]));
                        }
                    }
            }
            cx.call = rfl(cnt_v) + 1u;
            const u32 info = encode_block(D, __builtin_bit_cast(u32, mx), rsMe, OFF_EXP + slot * BLK, lane);
            const bool self = dG == G && ij == (kind ? myj : myi);
            if (self) {
                const u32 so = OFF_OP + ((kind * 32 + l) * 32 + c) * BLK;
#pragma unroll
                for (int f = 0; f < 8; ++f) st128(v4u{D[4 * f], D[4 * f + 1], D[4 * f + 2], D[4 * f + 3]}, rsMe, vo16 + f * 1024, so);
            }
            VM_WAIT(0);
            if (lane < 2) {
                const u32 du = kind ? (u32)lane * 2 + ij : ij * 2 + (u32)lane;
                const u32 d = dG * 4 + du;
                if (d == rank) st_flag32(cx.me + OFF_GF + (kind * 32 + l) * 128 + c * 4, cx.call);
                else st_flag64(p.ws[d & 7] + OFF_PF + rank * PF_SRC + (kind * 256 + c * 32 + l) * 8, cx.call, info);
            }
            if (lane == 0) *cntp = cx.call;
        }
        STAMP(1);

        const u32 xc = bid & 7, k = bid >> 3;
        const u32 rt = 4 * (xc & 1) + (k & 3), g = 2 * (xc >> 1) + ((k >> 2) & 1), ct = g + 8 * (k >> 3);
        const u32 tile = rt * 16 + ct;
        const u32 code0 = TB.step[rt * 8 + g][0];
        const u32 codeV = TB.step[rt * 8 + g][1 + lane];

        if (wave >= 4) {
            const u32 j = wave - 4;
            const u32 pc = j == 0 ? 0u : j == 1 ? 3u : j == 2 ? 2u : 1u;
            auto blk_of = [&](u32 code, u32& boff, u32& foff) {
                const u32 ty = code >> 5, kc = code & 31;
                const u32 row = j < 2 ? (ty & 1) * 16 + 2 * rt + j : 32 + 2 * ct + (j - 2);
                boff = ty == 2 ? OFF_OPP + (tile * 4 + pc) * BLK : OFF_OP + (row * 32 + kc) * BLK;
                foff = ty == 2 ? OFF_GP + tile * 128 + pc * 4 : OFF_GF + row * 128 + kc * 4;
            };
            u32 b0, f0, bV, fV;
            blk_of(code0, b0, f0);
            blk_of(codeV, bV, fV);
            const u32 ringj = lds0 + L_RING + j * BLK;
            const u32 aFull = lds0 + L_FULL + j * 4, aFree = lds0 + L_FREE;
            const unsigned char* gl = cx.me + vo16;
            u64 ready = 0;
            u32 pub = 0;
            {
                u32 spins = 0;
                while (rfl(__hip_atomic_load((GAS u32*)(cx.me + f0), __ATOMIC_RELAXED, __HIP_MEMORY_SCOPE_SYSTEM)) < cx.call) {
                    __builtin_amdgcn_s_sleep(2);
                    if ((++spins & 63u) == 0u && give_up(cx)) break;
                }
                glds4(gl + b0, ringj);
                glds4(gl + b0 + 4096, ringj + 4096);
            }
            for (u32 s = 1; s < (u32)NSTEP; ++s) {
                if (!((ready >> (s - 1)) & 1ull) && !cx.dead) {
                    u32 spins = 0;
                    for (;;) {
                        const u32 fv = __hip_atomic_load((GAS u32*)(cx.me + fV), __ATOMIC_RELAXED, __HIP_MEMORY_SCOPE_SYSTEM);
                        VM_WAIT(0);
                        if (pub < s) { pub = s; lds_st(aFull, pub); }
                        ready = __builtin_amdgcn_ballot_w64(fv >= cx.call);
                        if ((ready >> (s - 1)) & 1ull) break;
                        __builtin_amdgcn_s_sleep(2);
                        if ((++spins & 63u) == 0u && give_up(cx)) break;
                    }
                }
                if (s >= (u32)NSLOT) {
                    const u32 need = s - NSLOT + 1;
                    u32 spins = 0;
                    for (;;) {
                        const v4u fr = lds_ld4(aFree);
                        const u32 a = j == 0 ? fr.x : j == 1 ? fr.z : j == 2 ? fr.x : fr.y;
                        const u32 b = j == 0 ? fr.y : j == 1 ? fr.w : j == 2 ? fr.z : fr.w;
                        if (rfl(a < b ? a : b) >= need) break;
                        if (pub < s) { VM_WAIT(0); pub = s; lds_st(aFull, pub); }
                        __builtin_amdgcn_s_sleep(1);
                        if ((++spins & 1023u) == 0u && give_up(cx)) break;
                    }
                }
                const u32 bo = rlane(bV, s - 1);
                const u32 dst = ringj + (s & (NSLOT - 1)) * (4 * BLK);
                glds4(gl + bo, dst);
                glds4(gl + bo + 4096, dst + 4096);
                if (s + 1 - pub > 2) { VM_WAIT(16); pub = s - 1; lds_st(aFull, pub); }
            }
            if (pub < (u32)NSTEP - 1) { VM_WAIT(8); lds_st(aFull, NSTEP - 1); }
            VM_WAIT(0);
            if (cx.dead) lds_st(lds0 + L_ABORT, 1u);
            lds_st(aFull, NSTEP);
            LGKM0();
            STAMP(2);
            return;
        }

        const u32 wm = wave >> 1, wn = wave & 1;
        const u32 pj = wave == 0 ? 0u : wave == 1 ? 3u : wave == 2 ? 2u : 1u;
        const u64 smask = __builtin_amdgcn_ballot_w64((codeV >> 5) == 0);
        const u32 lastS = 64u - (u32)__builtin_clzll(smask);
        f32x4 accS[4][4], accO[4][4];
#pragma unroll
        for (int a = 0; a < 4; ++a)
#pragma unroll
            for (int b = 0; b < 4; ++b) { accS[a][b] = f32x4{0.f, 0.f, 0.f, 0.f}; accO[a][b] = f32x4{0.f, 0.f, 0.f, 0.f}; }
        const LAS unsigned char* ring = (const LAS unsigned char*)smem + vo16;
        u32 have = 0;
        auto wait_full = [&](u32 s) {
            if (have > s) return;
            u32 spins = 0;
            for (;;) {
                const v4u fu = lds_ld4(lds0 + L_FULL);
                const u32 a = wm ? fu.y : fu.x, b = wn ? fu.w : fu.z;
                have = rfl(a < b ? a : b);
                if (have > s) break;
                if ((++spins & 4095u) == 0u && give_up(cx)) { have = NSTEP; break; }
            }
        };
        const u32 aFreeMe = lds0 + L_FREE + wave * 4;
#define MMA(ACC, q) \
    _Pragma("unroll") for (int mt = 0; mt < 4; ++mt) _Pragma("unroll") for (int t = 0; t < 4; ++t) \
        ACC[mt][t] = __builtin_amdgcn_mfma_f32_16x16x32_bf16(__builtin_bit_cast(bf16x8, wf[(q) * 4 + t]), \
                                                             __builtin_bit_cast(bf16x8, xf[(q) * 4 + mt]), ACC[mt][t], 0, 0, 0)
#define GEMM_STEP(ACC) do { \
        const LAS unsigned char* sl = ring + (s & (NSLOT - 1)) * (4 * BLK); \
        v4u xf[8], wf[8]; \
        _Pragma("unroll") for (int q = 0; q < 2; ++q) { \
            _Pragma("unroll") for (int t = 0; t < 4; ++t) wf[q * 4 + t] = *(const LAS v4u*)(sl + (2 + wn) * BLK + (q * 4 + t) * 1024); \
            _Pragma("unroll") for (int t = 0; t < 4; ++t) xf[q * 4 + t] = *(const LAS v4u*)(sl + wm * BLK + (q * 4 + t) * 1024); \
        } \
        SCHED(); \
        MMA(ACC, 0); \
        SCHED(); \
        LGKM0(); \
        lds_st(aFreeMe, s + 1); \
        SCHED(); \
        MMA(ACC, 1); \
    } while (0)
#define PART_STEP() do { \
        const LAS unsigned char* sl = ring + (s & (NSLOT - 1)) * (4 * BLK); \
        v4u pv[8]; \
        _Pragma("unroll") for (int f = 0; f < 8; ++f) pv[f] = *(const LAS v4u*)(sl + pj * BLK + f * 1024); \
        SCHED(); \
        LGKM0(); \
        lds_st(aFreeMe, s + 1); \
        SCHED(); \
        _Pragma("unroll") for (int i = 0; i < 32; ++i) { \
            const int mt = i >> 3, t = (i >> 1) & 3, h = i & 1; \
            accO[mt][t][2 * h] += __builtin_bit_cast(float, pv[i >> 2][i & 3] << 16); \
            accO[mt][t][2 * h + 1] += __builtin_bit_cast(float, pv[i >> 2][i & 3] & 0xFFFF0000u); \
        } \
    } while (0)
        u32 s = 0;
        {
            const u64 sm = (smask << 1) | (u64)((code0 >> 5) == 0);
            while (s <= lastS) {
                while (s <= lastS && ((sm >> s) & 1ull)) { wait_full(s); GEMM_STEP(accS); ++s; }
                while (s <= lastS && !((sm >> s) & 1ull)) { wait_full(s); GEMM_STEP(accO); ++s; }
            }
        }
        {
            STAMP(3);
            u32 D[32];
            float mx = 0.f;
#pragma unroll
            for (int i = 0; i < 32; ++i) {
                const int mt = i >> 3, t = (i >> 1) & 3, h = i & 1;
                const float a = accS[mt][t][2 * h], b = accS[mt][t][2 * h + 1];
                D[i] = cvt2(a, b);
                mx = fmaxf(fmaxf(mx, fabsf(a)), fabsf(b));
            }
            const u32 info = encode_block(D, __builtin_bit_cast(u32, mx), rsMe, OFF_EXPP + (tile * 4 + wave) * BLK, lane);
            VM_WAIT(0);
            const u32 d = (1 - G) * 4 + (rt >> 2) * 2 + (ct >> 3);
            if (lane == 0)
                st_flag64(p.ws[d & 7] + OFF_PF + rank * PF_SRC + (PF_P + (((rt & 3) << 5) | (g << 2) | wave)) * 8, cx.call, info);
            STAMP(4);
        }
        {
            const u32 sP = 1u + (u32)__builtin_ctzll(__builtin_amdgcn_ballot_w64((codeV >> 5) == 2));
            for (; s < sP; ++s) { wait_full(s); GEMM_STEP(accO); }
            wait_full(s);
            PART_STEP();
            for (++s; s < (u32)NSTEP; ++s) { wait_full(s); GEMM_STEP(accO); }
        }
        STAMP(5);
        float mx = 0.f;
#pragma unroll
        for (int a = 0; a < 4; ++a)
#pragma unroll
            for (int b = 0; b < 4; ++b) {
                mx = fmaxf(fmaxf(mx, fabsf(accO[a][b][0])), fabsf(accO[a][b][1]));
                mx = fmaxf(fmaxf(mx, fabsf(accO[a][b][2])), fabsf(accO[a][b][3]));
            }
        unsigned char* peer = p.ws[lane & 7];
        {
            const u32 wmax = wave_max_u32(__builtin_bit_cast(u32, mx));
            u32 old = 0;
            if (lane == 0) {
                __hip_atomic_fetch_max((LAS u32*)(smem + L_MAX1), wmax, __ATOMIC_RELAXED, __HIP_MEMORY_SCOPE_WORKGROUP);
                old = __hip_atomic_fetch_add((LAS u32*)(smem + L_CNT1), 1u, __ATOMIC_RELAXED, __HIP_MEMORY_SCOPE_WORKGROUP);
            }
            if (rfl(old) == 3u) {
                const u32 m = lds_ld(lds0 + L_MAX1);
                if (lane < NR) st_flag64(peer + OFF_AM + (rank * NGEMM + bid) * 8, cx.call, m);
            }
        }
        u32 amax_bits;
        {
            u32 spins = 0, m = 0;
            for (;;) {
                const v4u a = ld128(rsMe, vo16, OFF_AM + wave * 2048), b = ld128(rsMe, vo16 + 1024, OFF_AM + wave * 2048);
                const bool ok = a.y >= cx.call && a.w >= cx.call && b.y >= cx.call && b.w >= cx.call;
                const u32 m0 = a.x > a.z ? a.x : a.z, m1 = b.x > b.z ? b.x : b.z;
                m = m0 > m1 ? m0 : m1;
                if (__builtin_amdgcn_ballot_w64(!ok) == 0 || cx.dead) break;
                __builtin_amdgcn_s_sleep(1);
                if ((++spins & 63u) == 0u && give_up(cx)) break;
            }
            const u32 wmax = wave_max_u32(m);
            if (cx.dead) lds_st(lds0 + L_ABORT, 1u);
            if (lane == 0) {
                __hip_atomic_fetch_max((LAS u32*)(smem + L_MAX2), wmax, __ATOMIC_RELAXED, __HIP_MEMORY_SCOPE_WORKGROUP);
                __hip_atomic_fetch_add((LAS u32*)(smem + L_CNT2), 1u, __ATOMIC_RELAXED, __HIP_MEMORY_SCOPE_WORKGROUP);
            }
            spins = 0;
            while (rfl(lds_ld(lds0 + L_CNT2)) < 4u) {
                if ((++spins & 4095u) == 0u && give_up(cx)) break;
            }
            const v4u c4 = lds_ld4(lds0 + L_ABORT);
            amax_bits = rfl(c4.w);
            if (rfl(c4.x) != 0u) cx.dead = true;
        }
        STAMP(6);
        {
            const float amax = __builtin_bit_cast(float, amax_bits);
            const float scale = amax / 127.0f;
            const float inv = scale > 0.f ? 1.0f / scale : 0.f;
            const float MAGIC = 12582912.0f;
            const __amdgpu_buffer_rsrc_t rsOut = mkrs(p.out, (u32)MM * NN);
            const u32 rb = row_block(G, myi, 16 + 2 * rt + wm), cb = col_block(G, myj, 2 * ct + wn);
            const u32 ovo = (u32)(lane & 15) * NN + (u32)(lane >> 4) * 16u, oso = 64 * rb * NN + 64 * cb;
            v4u qv[4];
#pragma unroll
            for (int mt = 0; mt < 4; ++mt) {
#pragma unroll
                for (int r = 0; r < 4; ++r) {
                    u32 tq[4];
#pragma unroll
                    for (int t = 0; t < 4; ++t) {
                        const float z = __builtin_amdgcn_fmed3f(__builtin_fmaf(accO[mt][t][r], inv, MAGIC), MAGIC - 127.0f, MAGIC + 127.0f);
                        tq[t] = __builtin_bit_cast(u32, z);
                    }
                    const u32 lo = perm(tq[1], tq[0], 0x0c0c0400u), hi = perm(tq[3], tq[2], 0x04000c0cu);
                    qv[mt][r] = cx.dead ? 0x80808080u : (lo | hi);
                }
                st128(qv[mt], rsMe, vo16 + mt * 1024, OFF_EXPO + (tile * 4 + wave) * 4096);
            }
            VM_WAIT(0);
            if (lane < NR && (u32)lane != rank) st_flag64(peer + OFF_PF + rank * PF_SRC + (PF_O + tile * 4 + wave) * 8, cx.call, 0u);
#pragma unroll
            for (int mt = 0; mt < 4; ++mt) st128(qv[mt], rsOut, ovo + mt * 16 * NN, oso);
        }
        STAMP(7);
        return;
    }

    {
        const u32 mid = bid - NGEMM, L = mid % 7u, jw = mid / 7u;
        const u32 v = L < 3 ? L + 1 : L - 3;
        const u32 sG = L < 3 ? G : 1 - G, sU = U ^ v, src = (sG * 4 + sU) & 7;
        const __amdgpu_buffer_rsrc_t rsSrc = mkrs(p.ws[src], WS_USED);
        cx.call = rfl(cnt_v) + 1u;
        if (lane == 0) *cntp = cx.call;
        const u32 pfb = OFF_PF + src * PF_SRC;
        u32 sOff, dOff, gOff, pOff;
        {
            const u32 n = lane < TPW ? (u32)lane : 0u;
            const u32 code = TB.pull[L][jw + NPW * n];
            const u32 kind = code >> 8, c = (code >> 5) & 7, l = code & 31;
            if (kind < 2) {
                const u32 kc = L < 3 ? 5 * v + c : 20 + 3 * v + (c - 5);
                sOff = OFF_EXP + (kind * 512 + c * 64 + (kind ? myj : myi) * 32 + l) * BLK;
                dOff = OFF_OP + ((kind * 32 + l) * 32 + kc) * BLK;
                gOff = OFF_GF + (kind * 32 + l) * 128 + kc * 4;
                pOff = pfb + (kind * 256 + c * 32 + l) * 8;
            } else {
                const u32 id = code & 127, r = id >> 5, gg = (id >> 2) & 7, cn = id & 3;
                const u32 tS = (4 * myi + r) * 16 + 8 * myj + gg, tO = (4 * (sU >> 1) + r) * 16 + 8 * (sU & 1) + gg;
                sOff = OFF_EXPP + (tS * 4 + cn) * BLK;
                dOff = OFF_OPP + (tO * 4 + cn) * BLK;
                gOff = OFF_GP + tO * 128 + cn * 4;
                pOff = pfb + (PF_P + id) * 8;
            }
        }
        u64 ready = 0;
        u32 infoV = 0;
        u32 pend = 0;
        auto wait_ready = [&](u32 n) {
            if (((ready >> n) & 1ull) || cx.dead) return;
            u32 spins = 0;
            for (;;) {
                const u64 fv = __hip_atomic_load((GAS u64*)(cx.me + pOff), __ATOMIC_RELAXED, __HIP_MEMORY_SCOPE_SYSTEM);
                VM_WAIT(0);
                if (pend) { if (lane == 0) st_flag32(cx.me + pend, cx.call); pend = 0; }
                infoV = (u32)fv;
                ready = __builtin_amdgcn_ballot_w64((u32)(fv >> 32) >= cx.call);
                if ((ready >> n) & 1ull) break;
                __builtin_amdgcn_s_sleep(1);
                if ((++spins & 63u) == 0u && give_up(cx)) break;
            }
        };
        for (u32 n = 0; n < (u32)TPW; ++n) {
            wait_ready(n);
            if (cx.dead) break;
            if (n == 0) STAMP(1);
            const u32 info = rlane(infoV, n), so = rlane(sOff, n), dso = rlane(dOff, n);
            if (info & (1u << 14)) {
                v4u r[8];
#pragma unroll
                for (int f = 0; f < 8; ++f) r[f] = ld128(rsSrc, vo16 + f * 1024, so);
                VM_WAIT(0);
                if (pend) { if (lane == 0) st_flag32(cx.me + pend, cx.call); pend = 0; }
#pragma unroll
                for (int f = 0; f < 8; ++f) st128(r[f], rsMe, vo16 + f * 1024, dso);
            } else {
                const u32 cnt = (info >> 7) & 127u;
                const u32 base4 = (info & 127u) * 0x01010101u;
                v4u pk[6];
                const u32 ex = __builtin_amdgcn_raw_buffer_load_b32(rsSrc, (u32)lane < cnt ? 6144u + (u32)lane * 4u : 0x7FFFFFF0u, so, AUX_SYS);
#pragma unroll
                for (int i = 0; i < 6; ++i) pk[i] = ld128(rsSrc, vo16 + i * 1024, so);
                SCHED();
                VM_WAIT(7);
                if (pend) { if (lane == 0) st_flag32(cx.me + pend, cx.call); pend = 0; }
                SCHED();
                u32 H[16];
#pragma unroll
                for (int f = 0; f < 8; ++f) {
                    const u32 nn = pk[3 * (f >> 2) + 2][f & 3];
                    H[2 * f] = (nn & 0x87878787u) + base4;
                    H[2 * f + 1] = (__builtin_amdgcn_alignbit(nn, nn, 4) & 0x87878787u) + base4;
                }
                for (u32 e = 0; e < cnt; ++e) {
                    const u32 ent = rlane(ex, e);
                    const u32 m = ((ent >> 8) & 63u) == (u32)lane ? 0xFFu << (8 * ((ent >> 14) & 3u)) : 0u;
                    const u32 val = (ent & 0xFFu) * 0x01010101u, ii = (ent >> 16) & 15u;
#pragma unroll
                    for (int i = 0; i < 16; ++i) {
                        const u32 mi = ii == (u32)i ? m : 0u;
                        H[i] = (H[i] & ~mi) | (val & mi);
                    }
                }
#pragma unroll
                for (int f = 0; f < 8; ++f) {
                    const v4u lp = pk[3 * (f >> 2) + ((f >> 1) & 1)];
                    const u32 la = lp[2 * (f & 1)], lb = lp[2 * (f & 1) + 1];
                    st128(v4u{perm(H[2 * f], la, 0x05010400u), perm(H[2 * f], la, 0x07030602u),
                              perm(H[2 * f + 1], lb, 0x05010400u), perm(H[2 * f + 1], lb, 0x07030602u)},
                          rsMe, vo16 + f * 1024, dso);
                }
            }
            pend = rlane(gOff, n);
        }
        VM_WAIT(0);
        if (pend && !cx.dead) { if (lane == 0) st_flag32(cx.me + pend, cx.call); }
        pend = 0;
        STAMP(2);
        {
            const __amdgpu_buffer_rsrc_t rsOut = mkrs(p.out, (u32)MM * NN);
            const u32 id = jw + NPW * (lane < OPWV ? (u32)lane : 0u);
            const u32 tl = id >> 2, cn = id & 3;
            const u32 rb = row_block(sG, sU >> 1, 16 + 2 * (tl >> 4) + (cn >> 1)), cb = col_block(sG, sU & 1, 2 * (tl & 15) + (cn & 1));
            const u32 oOff = 64 * rb * NN + 64 * cb;
            pOff = pfb + (PF_O + id) * 8;
            ready = 0;
            const u32 ovo = (u32)(lane & 15) * NN + (u32)(lane >> 4) * 16u;
            for (u32 n = 0; n < (u32)OPWV; ++n) {
                wait_ready(n);
                if (n == 0) STAMP(3);
                const u32 so = OFF_EXPO + (jw + NPW * n) * 4096, oso = rlane(oOff, n);
                v4u r[4];
                if (!cx.dead) {
#pragma unroll
                    for (int i = 0; i < 4; ++i) r[i] = ld128(rsSrc, vo16 + i * 1024, so);
                } else {
#pragma unroll
                    for (int i = 0; i < 4; ++i) r[i] = v4u{0x80808080u, 0x80808080u, 0x80808080u, 0x80808080u};
                }
#pragma unroll
                for (int i = 0; i < 4; ++i) st128(r[i], rsOut, ovo + i * 16 * NN, oso);
            }
        }
        STAMP(4);
    }
}

}

extern "C" void kernel_launch(int rank, int nranks, ncclComm_t comm, hipStream_t stream, void* const* d_in,
                              const int* in_sizes, int n_in, void* d_out, int out_size, const int* in_dtypes,
                              int out_dtype, void* const* d_out_all, void* const* d_ws_all, size_t ws_bytes) {
    if (nranks != NR || (unsigned)rank >= (unsigned)NR || n_in < 2 || in_sizes[0] != MM * KS || in_sizes[1] != KS * NN ||
        out_size != MM * NN || ws_bytes < WS_USED) {
        fprintf(stderr, "kernel_launch: unexpected problem shape\n");
        return;
    }
    Params p;
    for (int r = 0; r < NR; ++r) p.ws[r] = (unsigned char*)d_ws_all[r];
    p.x = (const float*)d_in[0];
    p.w = (const float*)d_in[1];
    p.out = (signed char*)d_out;
    p.rank = rank;
    p.pad = 0;
    fused_gemm_ar_q8<<<dim3(NWG), dim3(512), 0, stream>>>(p);
    hipError_t e = hipGetLastError();
    if (e != hipSuccess) fprintf(stderr, "launch failed: %s\n", hipGetErrorString(e));
}
```

```cpp
#include <hip/hip_runtime.h>
#include <rccl/rccl.h>
#include <cstdio>
#include <cstdint>

#define STAMPS 0

namespace {

#define GAS __attribute__((address_space(1)))
#define LAS __attribute__((address_space(3)))
typedef unsigned u32;
typedef unsigned long long u64;
typedef unsigned v4u __attribute__((ext_vector_type(4)));
typedef unsigned v2u __attribute__((ext_vector_type(2)));
typedef float f32x4 __attribute__((ext_vector_type(4)));
typedef float f32x2 __attribute__((ext_vector_type(2)));
typedef short bf16x8 __attribute__((ext_vector_type(8)));
typedef __bf16 bf16x2 __attribute__((ext_vector_type(2)));

constexpr int NR = 8;
constexpr int MM = 4096, NN = 4096, KS = 512;
constexpr int NGEMM = 128;
constexpr int NSONLY = 64;
constexpr int MOV0 = NGEMM + NSONLY;
constexpr int NMOV = 56;
constexpr int NWG = 256;
#define PW1 2
#define PW2 3
#define NP1 13
#define NP2 20
#define SFIRST 2
constexpr int NTASK = 320;
constexpr int NOUT = 512;
constexpr int NWAVE = 12;
constexpr int NSTEP = 65;
static_assert(PW1 <= PW2 && PW2 <= NWAVE, "pulling waves");

constexpr u32 BLK = 8192;
constexpr u32 OFF_CNT = 0;
constexpr u32 OFF_TMO = 14336;
constexpr u32 OFF_TK = 24576;
constexpr u32 OFF_AM = 16384;
constexpr u32 OFF_GF = 32768;
constexpr u32 OFF_GP = OFF_GF + 8192;
constexpr u32 OFF_PF = 65536;
constexpr u32 PF_SRC = 16384;
constexpr u32 PF_P = 512, PF_O = 1024;
constexpr u32 OFF_EXP = 1u << 20;
constexpr u32 OFF_EXPP = OFF_EXP + 1024 * BLK;
constexpr u32 OFF_OP = OFF_EXPP + 512 * BLK;
constexpr u32 OFF_OPP = OFF_OP + 2048 * BLK;
constexpr u32 OFF_EXPO = OFF_OPP + 512 * BLK;
constexpr u32 QSLOT = 4608;
constexpr u32 WS_USED = OFF_EXPO + 512 * QSLOT;
constexpr u32 OFF_STAMP = WS_USED;
constexpr int NSTAMP = 192;
static_assert(OFF_GP + 128 * 128 <= OFF_PF && OFF_PF + 8 * PF_SRC <= OFF_EXP, "control area");

constexpr u64 DEADLINE = 100000000ull;
constexpr int AUX_SYS = 17;
constexpr int RSRC3 = 0x00020000;
constexpr u32 OOB = 0x7FFFFFF0u;
#define CONV_SLEEP 8

constexpr int NSLOT = 5;
constexpr u32 SLOT_B = 4 * BLK;
constexpr u32 LDS_BYTES = NSLOT * SLOT_B;

constexpr unsigned NOCONV = 0xFFFFu;
#define TSTEP 40
constexpr int conv_cap(int b) {
    if (b >= MOV0 && b < MOV0 + NMOV) return PW1;
    if ((b & 7) < 4) return b < NGEMM ? 0 : NWAVE;
    return 5;
}
struct Tables {
    unsigned pull[4][NTASK];
    unsigned conv[NWG * NWAVE];
    int nconv;
    unsigned char step[96][68];
};
constexpr Tables make_tables() {
    Tables t{};
    int posX[4][8][32] = {}, posW[4][8][32] = {}, posP[128] = {};
    for (int L = 0; L < 4; ++L) {
        int n = 0;
        auto putX = [&](int c, int so, int rot) {
            for (int i = 0; i < 16; ++i) {
                const int l = 16 * so + (i + rot) % 16;
                posX[L][c][l] = n;
                t.pull[L][n++] = (unsigned)((c << 5) | l);
            }
        };
        auto putW = [&](int c, int g) {
            for (int k = 0; k < 4; ++k) {
                const int l = (k >> 1) * 16 + 2 * g + (k & 1);
                posW[L][c][l] = n;
                t.pull[L][n++] = (unsigned)(256 | (c << 5) | l);
            }
        };
        if (L < 3) {
            for (int key = 0; key < 12; ++key) {
                if (key < 5) putX(key, 0, 0);
                for (int c = 4; c >= 0; --c)
                    if (key - c >= 0 && key - c < 8) putW(c, key - c);
            }
            for (int c = 0; c < 5; ++c) putX(c, 1, L == 0 ? 0 : L == 1 ? 6 : 10);
        } else {
            for (int c = 5; c < 8; ++c) {
                putX(c, 0, 0);
                for (int g = 0; g < 8; ++g) putW(c, g);
            }
            for (int c = 5; c < 8; ++c) putX(c, 1, 0);
            for (int g = 0; g < 8; ++g)
                for (int r = 0; r < 4; ++r)
                    for (int cn = 0; cn < 4; ++cn) {
                        const int id = (r << 5) | (g << 2) | cn;
                        posP[id] = n;
                        t.pull[L][n++] = (unsigned)(512 | id);
                    }
        }
    }
    {
        unsigned ord[1024] = {};
        int n = 0;
        for (int pos = 0; pos < NTASK; ++pos)
            for (int L = 0; L < 4; L += 3) {
                const int code = t.pull[L][pos];
                if (code & 512) continue;
                const int kind = code >> 8, c = (code >> 5) & 7, l = code & 31;
                for (int ij = 0; ij < 2; ++ij) ord[n++] = (unsigned)(kind * 512 + c * 64 + ij * 32 + l);
            }
        n = 0;
        for (int i = 0; i < NWG * NWAVE; ++i) t.conv[i] = NOCONV;
        for (int w = 0; w < PW1; ++w)
            for (int b = MOV0; b < MOV0 + NMOV; ++b) t.conv[b * NWAVE + w] = ord[n++];
        for (int r = 0; r < 60; ++r)
            for (int b = 0; b < NWG; ++b) {
                if (b >= MOV0 && b < MOV0 + NMOV) continue;
                const int cap = conv_cap(b);
                for (int w = 0; w < cap; ++w)
                    if (60 * w / cap == r && n < 1024) t.conv[b * NWAVE + w] = ord[n++] | (unsigned)(r / 5 + 1) << 16;
            }
        t.nconv = n;
    }
    constexpr int T_STEP = TSTEP, T_MARGIN = 100, T_GEMM0 = 700;
    auto arrive = [](int pos) { return 410 + (pos + 16) * 1035 / 100 + 200; };
    for (int li = 0; li < 96; ++li) {
        const int rt = li < 64 ? li >> 3 : (li - 64) >> 2, g = li < 64 ? li & 7 : 4 + ((li - 64) & 3);
        const bool hasS = li >= 64 || g < 4, hasO = li < 64;
        int av[NSTEP] = {};
        bool used[NSTEP] = {};
        for (int so = 0; so < 2; ++so)
            for (int kc = 0; kc < 32; ++kc) {
                int k = T_GEMM0 - T_MARGIN;
                if (kc >= 5) {
                    const int L = kc < 20 ? kc / 5 - 1 : 3;
                    const int c = kc < 20 ? kc % 5 : 5 + (kc - 20) % 3;
                    const int a = posX[L][c][so * 16 + 2 * rt + 1], b = posW[L][c][16 + 2 * g + 1];
                    k = arrive(a > b ? a : b);
                }
                av[so * 32 + kc] = k;
                used[so * 32 + kc] = so ? !hasO : !hasS;
            }
        av[64] = arrive(posP[((rt & 3) << 5) | (g << 2) | 3]) + T_MARGIN;
        used[64] = !hasO;
        const int ns = (hasS ? 32 : 0) + (hasO ? 33 : 0);
        int code[NSTEP] = {};
        int now = T_GEMM0;
        for (int n = 0; n < ns; ++n) {
            int bs = -1, bo = -1;
            for (int i = 0; i < 32; ++i) if (!used[i] && (bs < 0 || av[i] < av[bs])) bs = i;
            for (int i = 32; i < (bs >= 0 ? 64 : 65); ++i) if (!used[i] && (bo < 0 || av[i] < av[bo])) bo = i;
            int pick = bs;
            if (bs < 0) pick = bo;
            else if (g >= SFIRST && av[bs] > now && bo >= 0 && av[bo] + T_MARGIN < av[bs]) pick = bo;
            const int ready = pick >= 32 && bs >= 0 ? av[pick] + T_MARGIN : av[pick];
            now = (ready > now ? ready : now) + T_STEP;
            used[pick] = true;
            code[n] = pick;
        }
        for (int a = 1; a + 1 < ns; ++a)
            if ((code[a] >> 5) == 2 && (code[a - 1] >> 5) != 1) { const int cc = code[a]; code[a] = code[a + 1]; code[a + 1] = cc; }
        for (int a = 0; a < 67; ++a) t.step[li][a] = (unsigned char)(a < ns ? code[a] : 3 << 5);
        t.step[li][67] = (unsigned char)ns;
    }
    return t;
}
constexpr bool tables_ok() {
    const Tables t = make_tables();
    if (t.nconv != 1024) return false;
    for (int w = 0; w < 64; ++w) {
        int lastS = -1, sP = -1;
        for (int s = 0; s < NSTEP; ++s) {
            if ((t.step[w][s] >> 5) == 0) lastS = s;
            if ((t.step[w][s] >> 5) == 2) sP = s;
        }
        if (lastS > 62 || sP <= lastS || sP < 1 || (t.step[w][sP - 1] >> 5) != 1) return false;
    }
    return true;
}
static_assert(tables_ok(), "every block is converted; the partial step follows an O step and the last S step");
__constant__ const Tables TB = make_tables();

struct Params {
    unsigned char* ws[NR];
    const float* x;
    const float* w;
    signed char* out;
    int rank;
    int pad;
};

__device__ __forceinline__ u64 rt_now() { return __builtin_amdgcn_s_memrealtime(); }
__device__ __forceinline__ u32 rfl(u32 v) { return __builtin_amdgcn_readfirstlane(v); }
__device__ __forceinline__ u32 rlane(u32 v, u32 l) { return __builtin_amdgcn_readlane(v, l); }
__device__ __forceinline__ __amdgpu_buffer_rsrc_t mkrs(const void* p, u32 bytes) {
    return __builtin_amdgcn_make_buffer_rsrc((void*)p, 0, (int)bytes, RSRC3);
}
__device__ __forceinline__ v4u ld128(__amdgpu_buffer_rsrc_t rs, u32 voff, u32 soff) {
    return __builtin_amdgcn_raw_buffer_load_b128(rs, voff, soff, AUX_SYS);
}
__device__ __forceinline__ void st128(v4u v, __amdgpu_buffer_rsrc_t rs, u32 voff, u32 soff) {
    __builtin_amdgcn_raw_buffer_store_b128(v, rs, voff + soff, 0, AUX_SYS);
}
#define VM_WAIT(n) asm volatile("s_waitcnt vmcnt(" #n ")" ::: "memory")
#define LGKM0() asm volatile("s_waitcnt lgkmcnt(0)" ::: "memory")
#define SCHED() __builtin_amdgcn_sched_barrier(0)
__device__ __forceinline__ u32 cvt2(float a, float b) {
    return __builtin_bit_cast(u32, __builtin_convertvector(f32x2{a, b}, bf16x2));
}
__device__ __forceinline__ u32 perm(u32 s0, u32 s1, u32 sel) { return __builtin_amdgcn_perm(s0, s1, sel); }
__device__ __forceinline__ u32 wave_max_u32(u32 v) {
    u32 t;
    t = __builtin_amdgcn_update_dpp(0u, v, 0x111, 0xf, 0xf, true); v = v > t ? v : t;
    t = __builtin_amdgcn_update_dpp(0u, v, 0x112, 0xf, 0xf, true); v = v > t ? v : t;
    t = __builtin_amdgcn_update_dpp(0u, v, 0x114, 0xf, 0xf, true); v = v > t ? v : t;
    t = __builtin_amdgcn_update_dpp(0u, v, 0x118, 0xf, 0xf, true); v = v > t ? v : t;
    t = __builtin_amdgcn_update_dpp(0u, v, 0x142, 0xa, 0xf, false); v = v > t ? v : t;
    t = __builtin_amdgcn_update_dpp(0u, v, 0x143, 0xc, 0xf, false); v = v > t ? v : t;
    return rlane(v, 63);
}
__device__ __forceinline__ void lds_st(u32 addr, u32 v) { asm volatile("ds_write_b32 %0, %1" ::"v"(addr), "v"(v) : "memory"); }
__device__ __forceinline__ v4u lds_ld4(u32 addr) {
    v4u v;
    asm volatile("ds_read_b128 %0, %1\n\ts_waitcnt lgkmcnt(0)" : "=&v"(v) : "v"(addr) : "memory");
    return v;
}
__device__ __forceinline__ u32 lds_ld(u32 addr) {
    u32 v;
    asm volatile("ds_read_b32 %0, %1\n\ts_waitcnt lgkmcnt(0)" : "=&v"(v) : "v"(addr) : "memory");
    return v;
}
__device__ __forceinline__ void glds4(const void* gsrc, u32 lds_dst) {
    u32 keep;
    asm volatile(
        "s_mov_b32 %0, m0\n\ts_mov_b32 m0, %2\n\ts_nop 0\n\t"
        "global_load_lds_dwordx4 %1, off sc0 sc1\n\t"
        "global_load_lds_dwordx4 %1, off offset:1024 sc0 sc1\n\t"
        "global_load_lds_dwordx4 %1, off offset:2048 sc0 sc1\n\t"
        "global_load_lds_dwordx4 %1, off offset:3072 sc0 sc1\n\t"
        "s_mov_b32 m0, %0"
        : "=&s"(keep) : "v"(gsrc), "s"(lds_dst) : "memory");
}

struct Ctx {
    unsigned char* me;
    u64 t0;
    u32 call;
    int lane;
    bool dead;
};
__device__ __forceinline__ bool give_up(Ctx& c) {
    GAS u32* tmo = (GAS u32*)(c.me + OFF_TMO);
    const u32 t = rfl(__hip_atomic_load(tmo, __ATOMIC_RELAXED, __HIP_MEMORY_SCOPE_AGENT));
    if (t == c.call || rt_now() - c.t0 > DEADLINE) {
        __hip_atomic_store(tmo, c.call, __ATOMIC_RELAXED, __HIP_MEMORY_SCOPE_AGENT);
        c.dead = true;
        return true;
    }
    return false;
}

#define STAMP(i) do {} while (0)
#define RSTAMP(V, i) do {} while (0)
#define RDUMP(V, o) do {} while (0)
#define BAR() asm volatile("s_barrier" ::: "memory")

__device__ __forceinline__ u32 bfi(u32 m, u32 a, u32 b) { return (a & m) | (b & ~m); }
__device__ __forceinline__ u32 mbcnt(u64 m) { return __builtin_amdgcn_mbcnt_hi((u32)(m >> 32), __builtin_amdgcn_mbcnt_lo((u32)m, 0u)); }

__device__ __forceinline__ u32 encode_block(const u32 (&D)[32], u32 hint, __amdgpu_buffer_rsrc_t rs, u32 soff, int lane) {
    const u32 vo = (u32)lane * 16u;
    const u32 e7 = wave_max_u32(hint) >> 24;
    const u32 base = e7 > 7u ? e7 - 7u : 0u;
    const u32 base4 = base * 0x01010101u;
#pragma unroll
    for (int k = 0; k < 4; ++k) {
        v4u lo;
#pragma unroll
        for (int e = 0; e < 4; ++e) lo[e] = perm(D[8 * k + 2 * e + 1], D[8 * k + 2 * e], 0x06040200u);
        st128(lo, rs, vo + (k >> 1) * 3072 + (k & 1) * 1024, soff);
    }
    u32 U[16], bad[4] = {0u, 0u, 0u, 0u};
#pragma unroll
    for (int i = 0; i < 16; ++i) {
        U[i] = perm(D[2 * i + 1], D[2 * i], 0x07050301u) - base4;
        bad[i >> 2] |= U[i] & 0x78787878u;
    }
#pragma unroll
    for (int u = 0; u < 2; ++u) {
        v4u nn;
#pragma unroll
        for (int e = 0; e < 4; ++e) nn[e] = bfi(0x87878787u, U[8 * u + 2 * e], __builtin_amdgcn_alignbit(U[8 * u + 2 * e + 1], U[8 * u + 2 * e + 1], 28));
        st128(nn, rs, vo + u * 3072 + 2048, soff);
    }
    if (__builtin_amdgcn_ballot_w64((bad[0] | bad[1] | bad[2] | bad[3]) != 0) == 0) return base;
    u32 cnt = 0;
#pragma unroll
    for (int q = 0; q < 4; ++q)
        if (__builtin_amdgcn_ballot_w64(bad[q] != 0) != 0) {
#pragma unroll
            for (int i = 4 * q; i < 4 * q + 4; ++i) {
                const bool b = (U[i] & 0x78787878u) != 0;
                const u64 m = __builtin_amdgcn_ballot_w64(b);
                const u32 idx = cnt + mbcnt(m);
                st128(v4u{D[2 * i], D[2 * i + 1], vo + (u32)((i >> 1) * 1024 + (i & 1) * 8), 0u}, rs,
                      b && idx < 64u ? 6144u + idx * 16u : OOB, soff);
                cnt += (u32)__builtin_popcountll(m);
            }
        }
    if (cnt <= 64u) return base | (cnt << 7);
    VM_WAIT(0);
#pragma unroll
    for (int f = 0; f < 8; ++f) st128(v4u{D[4 * f], D[4 * f + 1], D[4 * f + 2], D[4 * f + 3]}, rs, vo + f * 1024, soff);
    return 1u << 14;
}
__device__ __forceinline__ void decode_block(const v4u (&pk)[6], u32 info, v4u (&out)[8]) {
    const u32 base4 = (info & 127u) * 0x01010101u;
#pragma unroll
    for (int f = 0; f < 8; ++f) {
        const u32 nn = pk[3 * (f >> 2) + 2][f & 3];
        const u32 ha = (nn & 0x87878787u) + base4, hb = (__builtin_amdgcn_alignbit(nn, nn, 4) & 0x87878787u) + base4;
        const v4u lp = pk[3 * (f >> 2) + ((f >> 1) & 1)];
        const u32 la = lp[2 * (f & 1)], lb = lp[2 * (f & 1) + 1];
        out[f] = v4u{perm(ha, la, 0x05010400u), perm(ha, la, 0x07030602u), perm(hb, lb, 0x05010400u), perm(hb, lb, 0x07030602u)};
    }
}

__device__ __forceinline__ u32 encode_q(const v4u (&q)[4], __amdgpu_buffer_rsrc_t rs, u32 soff, int lane) {
    u32 P[14], M[2];
#pragma unroll
    for (int g = 0; g < 2; ++g) {
        const u32 top = q[2 * g + 1][3];
        u32 m = 0;
#pragma unroll
        for (int i = 0; i < 8; ++i) {
            const u32 x = q[2 * g + (i >> 2)][i & 3];
            if (i < 7) P[7 * g + i] = bfi(0x80808080u, top << (7 - i), x);
            m |= ((x ^ (x << 1)) & 0x80808080u) >> i;
        }
        M[g] = m;
    }
    const u32 vo = (u32)lane * 16u;
#pragma unroll
    for (int k = 0; k < 3; ++k) st128(v4u{P[4 * k], P[4 * k + 1], P[4 * k + 2], P[4 * k + 3]}, rs, vo + k * 1024, soff);
    __builtin_amdgcn_raw_buffer_store_b64(v2u{P[12], P[13]}, rs, 3072u + (u32)lane * 8u, soff, AUX_SYS);
    const u64 b0 = __builtin_amdgcn_ballot_w64(M[0] != 0), b1 = __builtin_amdgcn_ballot_w64(M[1] != 0);
    const u32 n0 = (u32)__builtin_popcountll(b0);
    st128(v4u{(u32)b0, (u32)(b0 >> 32), (u32)b1, (u32)(b1 >> 32)}, rs, lane == 0 ? 3584u : OOB, soff);
    __builtin_amdgcn_raw_buffer_store_b32(M[0], rs, M[0] != 0 ? 3600u + mbcnt(b0) * 4u : OOB, soff, AUX_SYS);
    __builtin_amdgcn_raw_buffer_store_b32(M[1], rs, M[1] != 0 ? 3600u + (n0 + mbcnt(b1)) * 4u : OOB, soff, AUX_SYS);
    return n0 + (u32)__builtin_popcountll(b1);
}
__device__ __forceinline__ void decode_q(const v4u (&pk)[3], v2u pk3, v4u ex, LAS unsigned char* scr, int lane, v4u (&q)[4]) {
    *(LAS v4u*)(scr + lane * 16) = ex;
    const u64 b0 = (u64)rlane(ex.x, 0) | (u64)rlane(ex.y, 0) << 32, b1 = (u64)rlane(ex.z, 0) | (u64)rlane(ex.w, 0) << 32;
    const u32 n0 = (u32)__builtin_popcountll(b0);
    u32 M[2];
    M[0] = *(const LAS u32*)(scr + 16 + mbcnt(b0) * 4);
    M[1] = *(const LAS u32*)(scr + 16 + (n0 + mbcnt(b1)) * 4);
    const u32 P[14] = {pk[0][0], pk[0][1], pk[0][2], pk[0][3], pk[1][0], pk[1][1], pk[1][2], pk[1][3],
                       pk[2][0], pk[2][1], pk[2][2], pk[2][3], pk3.x, pk3.y};
    u32 top[2];
#pragma unroll
    for (int g = 0; g < 2; ++g) {
        u32 t = 0;
#pragma unroll
        for (int i = 0; i < 7; ++i) t = bfi(0x01010101u << i, P[7 * g + i] >> (7 - i), t);
        top[g] = t;
    }
    M[0] = (b0 >> lane) & 1ull ? M[0] : 0u;
    M[1] = (b1 >> lane) & 1ull ? M[1] : 0u;
#pragma unroll
    for (int g = 0; g < 2; ++g)
#pragma unroll
        for (int i = 0; i < 8; ++i) {
            const u32 x = i < 7 ? P[7 * g + i] : top[g];
            q[2 * g + (i >> 2)][i & 3] = bfi(0x80808080u, (x << 1) ^ (M[g] << i), x);
        }
}

__device__ __forceinline__ void st_flag64(unsigned char* p, u32 call, u32 info) {
    __hip_atomic_store((GAS u64*)p, ((u64)call << 32) | info, __ATOMIC_RELAXED, __HIP_MEMORY_SCOPE_SYSTEM);
}
__device__ __forceinline__ void st_flag32(unsigned char* p, u32 call) {
    __hip_atomic_store((GAS u32*)p, call, __ATOMIC_RELAXED, __HIP_MEMORY_SCOPE_SYSTEM);
}
__device__ __forceinline__ u32 row_block(u32 G, u32 i, u32 l) {
    const u32 so = l >> 4, q = (l >> 3) & 1, idx = l & 7;
    const u32 a = G ? i + 2 * q : 2 * i + q, hf = G ? 1 - so : so;
    return 16 * a + 8 * hf + idx;
}
__device__ __forceinline__ u32 col_block(u32 G, u32 j, u32 l) {
    const u32 q = l >> 4, idx = l & 15;
    return 16 * (G ? j + 2 * q : 2 * j + q) + idx;
}

__device__ __forceinline__ void convert_block(const Params& p, Ctx& cx, u32 slot, u32 cnt_v, __amdgpu_buffer_rsrc_t rsMe, u32 wave) {
    const int lane = cx.lane;
    const u32 rank = p.rank, G = rank >> 2, myi = (rank >> 1) & 1, myj = rank & 1;
    const u32 kind = slot >> 9, c = (slot >> 6) & 7, ij = (slot >> 5) & 1, l = slot & 31;
    const u32 dG = c < 5 ? G : 1 - G;
    const u32 d0 = (dG * 4 + (kind ? ij : ij * 2)) & 7, d1 = (d0 + (kind ? 2 : 1)) & 7;
    unsigned char* const w0 = p.ws[d0];
    unsigned char* const w1 = p.ws[d1];
    u32 D[32];
    float mx = 0.f;
    const u32 r16 = lane & 15, g4 = lane >> 4;
    if (kind == 0) {
        const u32 rb = row_block(dG, ij, l);
        const GAS f32x4* src = (const GAS f32x4*)(p.x + (size_t)(64 * rb + r16) * KS + 64 * c + 4 * g4);
        f32x4 v[16];
#pragma unroll
        for (int mt = 0; mt < 4; ++mt)
#pragma unroll
            for (int i = 0; i < 4; ++i) v[mt * 4 + i] = src[mt * 16 * (KS / 4) + i * 4];
        SCHED();
#pragma unroll
        for (int mt = 0; mt < 4; ++mt)
#pragma unroll
            for (int i = 0; i < 4; ++i) {
                const f32x4 a = v[mt * 4 + i];
                const int f = (i >> 1) * 4 + mt, o = 4 * f + 2 * (i & 1);
                D[o] = cvt2(a.x, a.y);
                D[o + 1] = cvt2(a.z, a.w);
                if (i == 0) mx = fmaxf(fmaxf(fmaxf(fmaxf(mx, fabsf(a.x)), fabsf(a.y)), fabsf(a.z)), fabsf(a.w));
            }
    } else {
        const u32 cb = col_block(dG, ij, l);
        const GAS f32x4* src = (const GAS f32x4*)(p.w + (size_t)(64 * c + 4 * g4) * NN + 64 * cb + 4 * r16);
        f32x4 v[16];
#pragma unroll
        for (int j = 0; j < 16; ++j) v[j] = src[((j >> 2) * 16 + (j & 3)) * (NN / 4)];
        SCHED();
#pragma unroll
        for (int q = 0; q < 2; ++q)
#pragma unroll
            for (int pp = 0; pp < 4; ++pp) {
                const f32x4 a = v[q * 8 + 2 * pp], b = v[q * 8 + 2 * pp + 1];
#pragma unroll
                for (int t = 0; t < 4; ++t) {
                    D[4 * (q * 4 + t) + pp] = cvt2(a[t], b[t]);
                    if (pp == 0) mx = fmaxf(fmaxf(mx, fabsf(a[t])), fabsf(b[t]));
                }
            }
    }
    STAMP(28);
    cx.call = rfl(cnt_v) + 1u;
    const u32 info = encode_block(D, __builtin_bit_cast(u32, mx), rsMe, OFF_EXP + slot * BLK, lane);
    STAMP(29);
    VM_WAIT(0);
    const u32 fo = OFF_PF + rank * PF_SRC + (kind * 256 + c * 32 + l) * 8;
    if (lane == 0) {
        if (d0 != rank) st_flag64(w0 + fo, cx.call, info);
        if (d1 != rank) st_flag64(w1 + fo, cx.call, info);
    }
    if (d0 == rank || d1 == rank) {
        const u32 so = OFF_OP + ((kind * 32 + l) * 32 + c) * BLK;
#pragma unroll
        for (int f = 0; f < 8; ++f) st128(v4u{D[4 * f], D[4 * f + 1], D[4 * f + 2], D[4 * f + 3]}, rsMe, (u32)lane * 16u + f * 1024, so);
        VM_WAIT(0);
        if (lane == 0) st_flag32(cx.me + OFF_GF + (kind * 32 + l) * 128 + c * 4, cx.call);
    }
}

__global__ void __launch_bounds__(768, 3) fused_gemm_ar_q8(Params p) {
    __shared__ __attribute__((aligned(16))) unsigned char smem[LDS_BYTES];
    const int tid = threadIdx.x, lane = tid & 63;
    const u32 wave = rfl(tid >> 6);
    const u32 bid = blockIdx.x;
    const bool is_gemm = bid < (u32)MOV0;
    if (!is_gemm && bid < (u32)(MOV0 + NMOV) && wave >= (u32)PW2) return;
    const u32 cslot = TB.conv[bid * NWAVE + wave];
    const u32 rank = p.rank, G = rank >> 2, U = rank & 3, myi = U >> 1, myj = U & 1;
    Ctx cx;
    cx.me = p.ws[rank];
    cx.t0 = rt_now();
    cx.lane = lane;
    cx.dead = false;
    GAS u32* cntp = (GAS u32*)(cx.me + OFF_CNT) + bid * NWAVE + wave;
    const u32 cnt_v = __hip_atomic_load(cntp, __ATOMIC_RELAXED, __HIP_MEMORY_SCOPE_AGENT);
    const __amdgpu_buffer_rsrc_t rsMe = mkrs(cx.me, WS_USED);
    const u32 vo16 = (u32)lane * 16u;
    STAMP(0);
    if (cslot != NOCONV) {
        for (u32 s = cslot >> 16; s > 0; --s) __builtin_amdgcn_s_sleep(CONV_SLEEP);
        convert_block(p, cx, cslot & 1023u, cnt_v, rsMe, wave);
    }
    cx.call = rfl(cnt_v) + 1u;
    if (lane == 0) *cntp = cx.call;
    STAMP(30);
    if (bid >= (u32)(MOV0 + NMOV)) return;

    if (is_gemm) {
        const u32 lds0 = (u32)(size_t)(LAS unsigned char*)smem;
        const u32 xc = bid & 7, k = bid >> 3;
        const bool owner = bid < (u32)NGEMM;
        const u32 rt = 4 * (xc & 1) + (k & 3);
        const u32 g = owner ? 2 * (xc >> 1) + ((k >> 2) & 1) : 4 + (xc >> 1);
        const u32 ct = g + 8 * (owner ? k >> 3 : (k >> 2) & 1);
        const u32 tile = rt * 16 + ct;
        if (wave < 8 && (wave < 4 ? !owner : owner && g >= 4)) return;
        const u32 li = owner ? rt * 8 + g : 64 + rt * 4 + g - 4;
        const u32 code0 = rfl(TB.step[li][0]), nstep = rfl(TB.step[li][67]);
        const u32 codeV = TB.step[li][1 + lane];

        if (wave >= 8) {
            const u32 j = wave - 8;
            auto blk_of = [&](u32 code, u32& boff, u32& foff) {
                const u32 ty = code >> 5, kc = code & 31;
                const u32 row = j < 2 ? (ty & 1) * 16 + 2 * rt + j : 32 + 2 * ct + (j - 2);
                boff = ty == 2 ? OFF_OPP + (tile * 4 + j) * BLK : OFF_OP + (row * 32 + kc) * BLK;
                foff = ty == 2 ? OFF_GP + tile * 128 + j * 4 : OFF_GF + row * 128 + kc * 4;
            };
            u32 b0, f0, bV, fV;
            blk_of(code0, b0, f0);
            blk_of(codeV, bV, fV);
            const u32 ringj = lds0 + j * BLK;
            const unsigned char* gl = cx.me + vo16;
            GAS u32* fp = (GAS u32*)(cx.me + fV);
            {
                u32 spins = 0;
                while (rfl(__hip_atomic_load((GAS u32*)(cx.me + f0), __ATOMIC_RELAXED, __HIP_MEMORY_SCOPE_SYSTEM)) < cx.call) {
                    __builtin_amdgcn_s_sleep(2);
                    if ((++spins & 63u) == 0u && give_up(cx)) break;
                }
                VM_WAIT(0);
                glds4(gl + b0, ringj);
                glds4(gl + b0 + 4096, ringj + 4096);
            }
            u64 ready = cx.dead ? ~0ull : 0ull;
            u32 I = 1, dst = SLOT_B;
            auto issue_upto = [&](u32 last) {
                while (I <= last && I < nstep && ((ready >> (I - 1)) & 1ull)) {
                    if (STAMPS > 1) RSTAMP(stA, I);
                    const u32 bo = rlane(bV, I - 1);
                    glds4(gl + bo, ringj + dst);
                    glds4(gl + bo + 4096, ringj + dst + 4096);
                    ++I;
                    dst = dst == (NSLOT - 1) * SLOT_B ? 0u : dst + SLOT_B;
                }
            };
            bool pollp = false;
            u32 fv = 0;
            for (u32 s = 0; s < nstep; ++s) {
                if (I <= s) {
                    u32 spins = 0;
                    for (;;) {
                        if (!pollp) fv = __hip_atomic_load(fp, __ATOMIC_RELAXED, __HIP_MEMORY_SCOPE_SYSTEM);
                        pollp = false;
                        ready = __builtin_amdgcn_ballot_w64(fv >= cx.call);
                        if ((ready >> (s - 1)) & 1ull) break;
                        __builtin_amdgcn_s_sleep(1);
                        if ((++spins & 63u) == 0u && give_up(cx)) { ready = ~0ull; break; }
                    }
                    VM_WAIT(0);
                    issue_upto(s + NSLOT - 2);
                }
                const u32 ahead = I - s - 1;
                if (pollp) {
                    if (ahead == 0) VM_WAIT(1); else if (ahead == 1) VM_WAIT(9); else if (ahead == 2) VM_WAIT(17); else VM_WAIT(25);
                } else {
                    if (ahead == 0) VM_WAIT(0); else if (ahead == 1) VM_WAIT(8); else if (ahead == 2) VM_WAIT(16); else VM_WAIT(24);
                }
                if (STAMPS > 1) RSTAMP(stB, s);
                BAR();
                if (pollp) { ready = __builtin_amdgcn_ballot_w64(fv >= cx.call); pollp = false; }
                issue_upto(s + NSLOT - 1);
                if (I <= s + NSLOT - 1 && I < nstep) {
                    fv = __hip_atomic_load(fp, __ATOMIC_RELAXED, __HIP_MEMORY_SCOPE_SYSTEM);
                    pollp = true;
                }
            }
            STAMP(2);
            RDUMP(stA, 32);
            RDUMP(stB, 112);
            return;
        }

        const u32 cw = wave & 3, wm = cw >> 1, wn = cw & 1;
        const u64 smask = (__builtin_amdgcn_ballot_w64((codeV >> 5) == 0) << 1) | (u64)((code0 >> 5) == 0);
        unsigned char* const peer = p.ws[lane & 7];
        f32x4 acc[4][4];
#pragma unroll
        for (int a = 0; a < 4; ++a)
#pragma unroll
            for (int b = 0; b < 4; ++b) acc[a][b] = f32x4{0.f, 0.f, 0.f, 0.f};
        const LAS unsigned char* ring = (const LAS unsigned char*)smem + vo16;
        u32 s = 0, so = 0;
#define NEXT() do { ++s; so = so == (NSLOT - 1) * SLOT_B ? 0u : so + SLOT_B; } while (0)
#define RD(XF, WF, q) do { \
        _Pragma("unroll") for (int t = 0; t < 4; ++t) WF[t] = *(const LAS v4u*)(ring + so + (2 + wn) * BLK + ((q) * 4 + t) * 1024); \
        _Pragma("unroll") for (int t = 0; t < 4; ++t) XF[t] = *(const LAS v4u*)(ring + so + wm * BLK + ((q) * 4 + t) * 1024); \
    } while (0)
#define MIX() _Pragma("unroll") for (int i_ = 0; i_ < 8; ++i_) { __builtin_amdgcn_sched_group_barrier(0x100, 1, 0); __builtin_amdgcn_sched_group_barrier(0x8, 1, 0); }
#define MMA(XF, WF) \
    _Pragma("unroll") for (int mt = 0; mt < 4; ++mt) _Pragma("unroll") for (int t = 0; t < 4; ++t) \
        acc[mt][t] = __builtin_amdgcn_mfma_f32_16x16x32_bf16(__builtin_bit_cast(bf16x8, WF[t]), __builtin_bit_cast(bf16x8, XF[mt]), acc[mt][t], 0, 0, 0)
        {
            const u64 mine = __builtin_amdgcn_ballot_w64((codeV >> 5) == (wave < 4 ? 1u : 0u));
            const u32 sP = wave < 4 ? 1u + (u32)__builtin_ctzll(__builtin_amdgcn_ballot_w64((codeV >> 5) == 2)) : 999u;
            const u32 send = wave < 4 ? nstep : 64u - (u32)__builtin_clzll(smask);
            auto add_partial = [&]() {
                BAR();
                RSTAMP(stA, s);
                v4u pv[8];
#pragma unroll
                for (int f = 0; f < 8; ++f) pv[f] = *(const LAS v4u*)(ring + so + cw * BLK + f * 1024);
#pragma unroll
                for (int i = 0; i < 16; ++i) {
                    const u32 u0 = pv[i >> 1][2 * (i & 1)], u1 = pv[i >> 1][2 * (i & 1) + 1];
                    acc[i >> 2][i & 3] += __builtin_bit_cast(f32x4, v4u{u0 << 16, u0 & 0xFFFF0000u, u1 << 16, u1 & 0xFFFF0000u});
                }
                LGKM0();
                NEXT();
            };
            do {
                while ((((smask >> (s & 63u)) & 1ull) != 0 && s < 64u) != (wave >= 4)) { BAR(); NEXT(); }
                BAR();
                RSTAMP(stA, s);
                const u64 rest = s < 64u ? ~(mine >> s) : ~0ull;
                const u32 e = s + 1 + (rest ? (u32)__builtin_ctzll(rest) : 64u);
                v4u xa[4], wa[4], xb[4], wb[4];
                RD(xa, wa, 0); RD(xb, wb, 1); SCHED();
                MMA(xa, wa); SCHED();
                LGKM0(); NEXT();
                while (s < e) {
                    BAR();
                    RSTAMP(stA, s);
                    RD(xa, wa, 0); MMA(xb, wb); MIX(); SCHED();
                    RD(xb, wb, 1); MMA(xa, wa); MIX(); SCHED();
                    LGKM0(); NEXT();
                }
                MMA(xb, wb); SCHED();
                if (s == sP) add_partial();
            } while (s < send);
        }
#pragma unroll
        for (int i = 0; i < 16; ++i) asm volatile("" : "+v"(acc[i >> 2][i & 3]));
        if (wave >= 4) {
            STAMP(3);
            u32 D[32];
            float mx = 0.f;
#pragma unroll
            for (int i = 0; i < 32; ++i) {
                const int mt = i >> 3, t = (i >> 1) & 3, h = i & 1;
                const float a = acc[mt][t][2 * h], b = acc[mt][t][2 * h + 1];
                D[i] = cvt2(a, b);
                if (t == 0) mx = fmaxf(fmaxf(mx, fabsf(a)), fabsf(b));
            }
            const u32 d = (1 - G) * 4 + (rt >> 2) * 2 + (ct >> 3);
            unsigned char* const wd = p.ws[d & 7];
            const u32 info = encode_block(D, __builtin_bit_cast(u32, mx), rsMe, OFF_EXPP + (tile * 4 + cw) * BLK, lane);
            VM_WAIT(0);
            if (lane == 0) st_flag64(wd + OFF_PF + rank * PF_SRC + (PF_P + (((rt & 3) << 5) | (g << 2) | cw)) * 8, cx.call, info);
            STAMP(4);
            for (; s < nstep; ++s) BAR();
            RDUMP(stA, 32);
            return;
        }
        STAMP(5);
        u32 amax_bits;
        {
            float mx = 0.f;
#pragma unroll
            for (int a = 0; a < 4; ++a)
#pragma unroll
                for (int b = 0; b < 4; ++b) {
                    mx = fmaxf(fmaxf(mx, fabsf(acc[a][b][0])), fabsf(acc[a][b][1]));
                    mx = fmaxf(fmaxf(mx, fabsf(acc[a][b][2])), fabsf(acc[a][b][3]));
                }
            const u32 post = lds0 + so;
            lds_st(post + wave * 4, wave_max_u32(__builtin_bit_cast(u32, mx)));
            LGKM0();
            BAR();
            if (wave == 0) {
                const v4u m4 = lds_ld4(post);
                const u32 m0 = m4.x > m4.y ? m4.x : m4.y, m1 = m4.z > m4.w ? m4.z : m4.w;
                if (lane < NR) st_flag64(peer + OFF_AM + (rank * NGEMM + bid) * 8, cx.call, m0 > m1 ? m0 : m1);
            }
            u32 spins = 0, m = 0;
            for (;;) {
                const v4u a = ld128(rsMe, vo16, OFF_AM + wave * 2048), b = ld128(rsMe, vo16 + 1024, OFF_AM + wave * 2048);
                const u32 tm = __builtin_amdgcn_raw_buffer_load_b32(rsMe, 0, OFF_TMO, AUX_SYS);
                const bool ok = a.y >= cx.call && a.w >= cx.call && b.y >= cx.call && b.w >= cx.call;
                const u32 m0 = a.x > a.z ? a.x : a.z, m1 = b.x > b.z ? b.x : b.z;
                m = m0 > m1 ? m0 : m1;
                if (rfl(tm) == cx.call) cx.dead = true;
                if (__builtin_amdgcn_ballot_w64(!ok) == 0 || cx.dead) break;
                __builtin_amdgcn_s_sleep(1);
                if ((++spins & 63u) == 0u && give_up(cx)) break;
            }
            lds_st(post + 16 + wave * 4, cx.dead ? 0xFFFFFFFFu : wave_max_u32(m));
            LGKM0();
            BAR();
            const v4u m4 = lds_ld4(post + 16);
            const u32 m0 = m4.x > m4.y ? m4.x : m4.y, m1 = m4.z > m4.w ? m4.z : m4.w;
            amax_bits = rfl(m0 > m1 ? m0 : m1);
            if (amax_bits == 0xFFFFFFFFu) cx.dead = true;
        }
        STAMP(6);
        {
            const float amax = __builtin_bit_cast(float, amax_bits);
            const float scale = amax / 127.0f;
            const float inv = scale > 0.f ? 1.0f / scale : 0.f;
            const float MAGIC = 12582912.0f;
            const __amdgpu_buffer_rsrc_t rsOut = mkrs(p.out, (u32)MM * NN);
            const u32 rb = row_block(G, myi, 16 + 2 * rt + wm), cb = col_block(G, myj, 2 * ct + wn);
            const u32 ovo = (u32)(lane & 15) * NN + (u32)(lane >> 4) * 16u, oso = 64 * rb * NN + 64 * cb;
            v4u qv[4];
#pragma unroll
            for (int mt = 0; mt < 4; ++mt) {
#pragma unroll
                for (int r = 0; r < 4; ++r) {
                    u32 tq[4];
#pragma unroll
                    for (int t = 0; t < 4; ++t) {
                        const float z = __builtin_amdgcn_fmed3f(__builtin_fmaf(acc[mt][t][r], inv, MAGIC), MAGIC - 127.0f, MAGIC + 127.0f);
                        tq[t] = __builtin_bit_cast(u32, z);
                    }
                    const u32 lo = perm(tq[1], tq[0], 0x0c0c0400u), hi = perm(tq[3], tq[2], 0x04000c0cu);
                    qv[mt][r] = cx.dead ? 0x80808080u : (lo | hi);
                }
            }
            const u32 nz = encode_q(qv, rsMe, OFF_EXPO + (tile * 4 + wave) * QSLOT, lane);
            VM_WAIT(0);
            if (lane < NR && (u32)lane != rank) st_flag64(peer + OFF_PF + rank * PF_SRC + (PF_O + tile * 4 + wave) * 8, cx.call, nz);
#pragma unroll
            for (int mt = 0; mt < 4; ++mt) st128(qv[mt], rsOut, ovo + mt * 16 * NN, oso);
        }
        STAMP(7);
        RDUMP(stA, 32);
        return;
    }

    {
        const u32 mid = bid - MOV0, L = mid % 7u, cu = mid / 7u;
        const u32 v = L < 3 ? L + 1 : L - 3;
        const u32 sG = L < 3 ? G : 1 - G, sU = U ^ v, src = (sG * 4 + sU) & 7;
        const __amdgpu_buffer_rsrc_t rsSrc = mkrs(p.ws[src], WS_USED);
        const u32 pfb = OFF_PF + src * PF_SRC;
        const u32 tk = OFF_TK + ((cx.call & 1u) * 16 + L) * 128;
        const u32 l0 = lane == 0 ? 0u : OOB;
        if (cu == 0 && wave == 0 && lane < 2)
            __builtin_amdgcn_raw_buffer_store_b32(0u, rsMe, (u32)lane * 1024u, tk ^ 2048u, AUX_SYS);
        u32 pend = OOB;
        auto publish = [&]() {
            __builtin_amdgcn_raw_buffer_store_b32(cx.call, rsMe, lane == 0 ? pend : OOB, 0, AUX_SYS);
            pend = OOB;
        };
        auto poll = [&](u32 po) -> u32 {
            VM_WAIT(0);
            publish();
            u32 spins = 0;
            while (!cx.dead) {
                const v2u f = __builtin_amdgcn_raw_buffer_load_b64(rsMe, 0, po, AUX_SYS);
                if (rfl(f.y) >= cx.call) return rfl(f.x);
                __builtin_amdgcn_s_sleep(1);
                if ((++spins & 63u) == 0u) give_up(cx);
            }
            return 0u;
        };
        const u32 jw = wave * 8 + cu;
        if (jw < (u32)NP1) {
            const u32* list = TB.pull[L < 3 ? L : 3];
            auto desc = [&](u32 n, u32& so, u32& dso, u32& go, u32& po) {
                const u32 code = list[n < (u32)NTASK ? n : 0u];
                const u32 kind = code >> 8, c = (code >> 5) & 7, l = code & 31;
                if (kind < 2) {
                    const u32 kc = L < 3 ? 5 * v + c : 20 + 3 * v + (c - 5);
                    so = OFF_EXP + (kind * 512 + c * 64 + (kind ? myj : myi) * 32 + l) * BLK;
                    dso = OFF_OP + ((kind * 32 + l) * 32 + kc) * BLK;
                    go = OFF_GF + (kind * 32 + l) * 128 + kc * 4;
                    po = pfb + (kind * 256 + c * 32 + l) * 8;
                } else {
                    const u32 id = code & 127, r = id >> 5, gg = (id >> 2) & 7, cn = id & 3;
                    const u32 tS = (4 * myi + r) * 16 + 8 * myj + gg, tO = (4 * (sU >> 1) + r) * 16 + 8 * (sU & 1) + gg;
                    so = OFF_EXPP + (tS * 4 + cn) * BLK;
                    dso = OFF_OPP + (tO * 4 + cn) * BLK;
                    go = OFF_GP + tO * 128 + cn * 4;
                    po = pfb + (PF_P + id) * 8;
                }
            };
            u32 so, dso, go, po;
            desc(jw, so, dso, go, po);
            u32 info = poll(po);
            STAMP(1);
            while (!cx.dead) {
                u32 n2, so2, dso2, go2, po2;
                v2u fl;
                if (info & (1u << 14)) {
                    const u32 tv = (u32)__builtin_amdgcn_raw_ptr_buffer_atomic_add_i32(1, rsMe, l0, tk, 16);
                    v4u r[8];
#pragma unroll
                    for (int f = 0; f < 8; ++f) r[f] = ld128(rsSrc, vo16 + f * 1024, so);
                    VM_WAIT(0);
                    publish();
                    n2 = rfl(tv) + NP1;
                    desc(n2, so2, dso2, go2, po2);
                    fl = __builtin_amdgcn_raw_buffer_load_b64(rsMe, 0, po2, AUX_SYS);
#pragma unroll
                    for (int f = 0; f < 8; ++f) st128(r[f], rsMe, vo16 + f * 1024, dso);
                    __builtin_amdgcn_raw_buffer_store_b32(0u, rsMe, OOB, 0, AUX_SYS);
                } else {
                    const u32 cnt = (info >> 7) & 127u;
                    v4u pk[6];
                    const u32 tv = (u32)__builtin_amdgcn_raw_ptr_buffer_atomic_add_i32(1, rsMe, l0, tk, 16);
                    SCHED();
                    const v4u ex = ld128(rsSrc, (u32)lane < cnt ? 6144u + vo16 : OOB, so);
#pragma unroll
                    for (int i = 0; i < 6; ++i) pk[i] = ld128(rsSrc, vo16 + i * 1024, so);
                    SCHED();
                    VM_WAIT(8);
                    publish();
                    SCHED();
                    n2 = rfl(tv) + NP1;
                    desc(n2, so2, dso2, go2, po2);
                    SCHED();
                    VM_WAIT(1);
                    fl = __builtin_amdgcn_raw_buffer_load_b64(rsMe, 0, po2, AUX_SYS);
                    SCHED();
                    v4u o[8];
                    decode_block(pk, info, o);
#pragma unroll
                    for (int f = 0; f < 8; ++f) st128(o[f], rsMe, vo16 + f * 1024, dso);
                    SCHED();
                    __builtin_amdgcn_raw_buffer_store_b64(v2u{ex.x, ex.y}, rsMe, (u32)lane < cnt ? (ex.z & (BLK - 8)) : OOB, dso, AUX_SYS);
                }
                pend = go;
                if (n2 >= (u32)NTASK) break;
                info = rfl(fl.y) >= cx.call ? rfl(fl.x) : poll(po2);
                so = so2; dso = dso2; go = go2;
            }
            VM_WAIT(0);
            publish();
            STAMP(2);
        }
        if (jw < (u32)NP2) {
            const __amdgpu_buffer_rsrc_t rsOut = mkrs(p.out, (u32)MM * NN);
            const u32 ovo = (u32)(lane & 15) * NN + (u32)(lane >> 4) * 16u;
            u32 n = jw;
            u32 nz = poll(pfb + (PF_O + n) * 8);
            LAS unsigned char* scr = (LAS unsigned char*)smem + wave * 1024;
            STAMP(3);
            for (;;) {
                const u32 tv = (u32)__builtin_amdgcn_raw_ptr_buffer_atomic_add_i32(1, rsMe, l0, tk + 1024, 16);
                SCHED();
                const u32 so = OFF_EXPO + n * QSLOT;
                const v4u ex = ld128(rsSrc, (u32)lane * 4u < 4u + (nz & 255u) ? 3584u + vo16 : OOB, so);
                v4u pk[3];
#pragma unroll
                for (int i = 0; i < 3; ++i) pk[i] = ld128(rsSrc, vo16 + i * 1024, so);
                const v2u pk3 = __builtin_amdgcn_raw_buffer_load_b64(rsSrc, 3072u + (u32)lane * 8u, so, AUX_SYS);
                SCHED();
                const u32 n2 = rfl(tv) + NP2;
                const u32 po2 = pfb + (PF_O + (n2 < (u32)NOUT ? n2 : 0u)) * 8;
                const v2u fl = __builtin_amdgcn_raw_buffer_load_b64(rsMe, 0, po2, AUX_SYS);
                SCHED();
                const u32 tl = n >> 2, cn = n & 3;
                const u32 rb = row_block(sG, sU >> 1, 16 + 2 * (tl >> 4) + (cn >> 1)), cb = col_block(sG, sU & 1, 2 * (tl & 15) + (cn & 1));
                v4u r[4];
                decode_q(pk, pk3, ex, scr, lane, r);
#pragma unroll
                for (int i = 0; i < 4; ++i) {
                    if (cx.dead) r[i] = v4u{0x80808080u, 0x80808080u, 0x80808080u, 0x80808080u};
                    st128(r[i], rsOut, ovo + i * 16 * NN, 64 * rb * NN + 64 * cb);
                }
                if (n2 >= (u32)NOUT) break;
                nz = rfl(fl.y) >= cx.call ? rfl(fl.x) : poll(po2);
                n = n2;
            }
        }
        STAMP(4);
    }
}

}

extern "C" void kernel_launch(int rank, int nranks, ncclComm_t comm, hipStream_t stream, void* const* d_in,
                              const int* in_sizes, int n_in, void* d_out, int out_size, const int* in_dtypes,
                              int out_dtype, void* const* d_out_all, void* const* d_ws_all, size_t ws_bytes) {
    if (nranks != NR || (unsigned)rank >= (unsigned)NR || n_in < 2 || in_sizes[0] != MM * KS || in_sizes[1] != KS * NN ||
        out_size != MM * NN || ws_bytes < WS_USED) {
        fprintf(stderr, "kernel_launch: unexpected problem shape\n");
        return;
    }
    Params p;
    for (int r = 0; r < NR; ++r) p.ws[r] = (unsigned char*)d_ws_all[r];
    p.x = (const float*)d_in[0];
    p.w = (const float*)d_in[1];
    p.out = (signed char*)d_out;
    p.rank = rank;
    p.pad = 0;
    fused_gemm_ar_q8<<<dim3(NWG), dim3(64 * NWAVE), 0, stream>>>(p);
    hipError_t e = hipGetLastError();
    if (e != hipSuccess) fprintf(stderr, "launch failed: %s\n", hipGetErrorString(e));
}
```

```cpp
#include <hip/hip_runtime.h>
#include <rccl/rccl.h>
#include <cstdio>
#include <cstdint>

#define STAMPS 0

namespace {

#define GAS __attribute__((address_space(1)))
#define LAS __attribute__((address_space(3)))
typedef unsigned u32;
typedef unsigned long long u64;
typedef unsigned v4u __attribute__((ext_vector_type(4)));
typedef unsigned v2u __attribute__((ext_vector_type(2)));
typedef float f32x4 __attribute__((ext_vector_type(4)));
typedef float f32x2 __attribute__((ext_vector_type(2)));
typedef short bf16x8 __attribute__((ext_vector_type(8)));
typedef __bf16 bf16x2 __attribute__((ext_vector_type(2)));

constexpr int NR = 8;
constexpr int MM = 4096, NN = 4096, KS = 512;
constexpr int NGEMM = 128;
constexpr int NSONLY = 64;
constexpr int MOV0 = NGEMM + NSONLY;
constexpr int NMOV = 56;
constexpr int NWG = 256;
#define PW1 2
#define PW2 3
#define NP1 13
#define NP2 19
#define SFIRST 2
#define YG 4
#define YEARLY 3
#define YMIX 1
constexpr int NYB = 16 * YG;
constexpr int NTASK = 320 + NYB;
constexpr int NOUT = 512 - NYB;
constexpr int NWAVE = 12;
constexpr int NSTEP = 65;
static_assert(PW1 <= PW2 && PW2 <= NWAVE, "pulling waves");

constexpr u32 BLK = 8192;
constexpr u32 OFF_CNT = 0;
constexpr u32 OFF_TMO = 14336;
constexpr u32 OFF_TK = 24576;
constexpr u32 OFF_ACC = 16384;
constexpr u32 OFF_AM = 262144;
constexpr u32 OFF_GF = 32768;
constexpr u32 OFF_GP = OFF_GF + 8192;
constexpr u32 OFF_GY = OFF_GP + 16384;
constexpr u32 OFF_PF = 65536;
constexpr u32 PF_SRC = 16384;
constexpr u32 PF_P = 512, PF_Y = 640, PF_O = 1024;
constexpr u32 OFF_EXP = 1u << 20;
constexpr u32 OFF_EXPP = OFF_EXP + 1024 * BLK;
constexpr u32 OFF_OP = OFF_EXPP + 512 * BLK;
constexpr u32 OFF_OPP = OFF_OP + 2048 * BLK;
constexpr u32 OFF_EXPY = OFF_OPP + 512 * BLK;
constexpr u32 OFF_OPY = OFF_EXPY + 128 * BLK;
constexpr u32 OFF_EXPO = OFF_OPY + 1024 * BLK;
constexpr u32 QSLOT = 4608;
constexpr u32 WS_USED = OFF_EXPO + 512 * QSLOT;
constexpr u32 OFF_STAMP = WS_USED;
constexpr int NSTAMP = 192;
static_assert(OFF_GY + 4096 <= OFF_PF && OFF_PF + 8 * PF_SRC <= OFF_AM && OFF_AM + 65536 <= OFF_EXP, "control area");

constexpr u64 DEADLINE = 100000000ull;
constexpr int AUX_SYS = 17;
constexpr int RSRC3 = 0x00020000;
constexpr u32 OOB = 0x7FFFFFF0u;
#define LD_SLEEP 1
#define PL_SLEEP 1
#define CONV_SLEEP 10

#define NSLOT 5
#define LCAP 1
#define LOOK 4
constexpr u32 SLOT_B = 4 * BLK;
constexpr u32 LDS_BYTES = NSLOT * SLOT_B;

constexpr unsigned NOCONV = 0xFFFFu;
#define PMARGIN 200
#define MOVCONV 0
#define QSTAG 8
#define TSTEP 40
constexpr int conv_cap(int b) {
    if (b >= MOV0 && b < MOV0 + NMOV) return PW1;
    if ((b & 7) < 4) return b < NGEMM ? 0 : NWAVE;
    return 5;
}
constexpr bool owner_runs_s(int rt, int g) { return g < 2 || (g < 4 && (rt & 3) != 0); }
struct Tables {
    unsigned pull[4][NTASK];
    unsigned conv[NWG * NWAVE];
    int nconv;
    unsigned outb[512];
    unsigned char step[128][68];
};
constexpr Tables make_tables() {
    Tables t{};
    int posX[4][8][32] = {}, posW[4][8][32] = {}, posP[128] = {};
    for (int L = 0; L < 4; ++L) {
        int n = 0;
        auto putX = [&](int c, int so, int rot) {
            for (int i = 0; i < 16; ++i) {
                const int l = 16 * so + (i + rot) % 16;
                posX[L][c][l] = n;
                t.pull[L][n++] = (unsigned)((c << 5) | l);
            }
        };
        auto putW = [&](int c, int g) {
            for (int k = 0; k < 4; ++k) {
                const int l = (k >> 1) * 16 + 2 * g + (k & 1);
                posW[L][c][l] = n;
                t.pull[L][n++] = (unsigned)(256 | (c << 5) | l);
            }
        };
        auto putXO = [&](int c, bool yrows, int rot) {
            for (int i = 0; i < (yrows ? 4 : 12); ++i) {
                const int k = yrows ? i : (i + rot) % 12;
                const int l = 16 + (yrows ? (k >> 1) * 8 + (k & 1) : (k / 6) * 8 + 2 + k % 6);
                posX[L][c][l] = n;
                t.pull[L][n++] = (unsigned)((c << 5) | l);
            }
        };
        if (L < 3) {
            for (int key = 0; key < 12; ++key) {
                if (key < 5) putX(key, 0, 0);
                for (int c = 4; c >= 0; --c)
                    if (key - c >= 0 && key - c < 8) putW(c, key - c);
                if (YG > 0 && key >= 11 - YEARLY && key < 11) putXO(key - 11 + YEARLY, true, 0);
            }
            if (YG > 0) {
                int cn = 0;
                for (int c = YEARLY; c < 5; ++c) {
                    if (YMIX && c == 4 && YEARLY < 4) putXO(cn++, false, 4 * L);
                    putXO(c, true, 0);
                }
                for (; cn < 5; ++cn) putXO(cn, false, 4 * L);
            } else
                for (int c = 0; c < 5; ++c) putX(c, 1, L == 0 ? 0 : L == 1 ? 6 : 10);
        } else {
            for (int c = 5; c < 8; ++c) {
                putX(c, 0, 0);
                for (int g = 0; g < 8; ++g) putW(c, g);
            }
            for (int c = 5; c < 8; ++c) putX(c, 1, 0);
            for (int g = 0; g < 8; ++g)
                for (int r = 0; r < 4; ++r)
                    for (int cn = 0; cn < 4; ++cn) {
                        const int id = (r << 5) | (g << 2) | cn;
                        posP[id] = n;
                        t.pull[L][n++] = (unsigned)(512 | id);
                    }
        }
        constexpr int yorder[8] = {2, 3, 4, 0, 5, 1, 6, 7};
        for (int k = 0; k < 8; ++k)
            if (yorder[k] < YG)
                for (int i = 0; i < 16; ++i) t.pull[L][n++] = (unsigned)(768 | (yorder[k] << 4) | i);
    }
    {
        int n = 0;
        for (int g = 0; g < 8; ++g)
            for (int rt = 0; rt < 8; ++rt)
                for (int q = 0; q < 2; ++q)
                    if (!((rt & 3) == 0 && g < YG))
                        for (int cn = 0; cn < 4; ++cn) t.outb[n++] = (unsigned)((rt * 16 + q * 8 + g) * 4 + cn);
    }
    {
        unsigned ord[1024] = {};
        int n = 0;
        for (int pos = 0; pos < NTASK; ++pos)
            for (int L = 0; L < 4; L += 3) {
                const int code = t.pull[L][pos];
                if (code & 512) continue;
                const int kind = code >> 8, c = (code >> 5) & 7, l = code & 31;
                for (int ij = 0; ij < 2; ++ij) ord[n++] = (unsigned)(kind * 512 + c * 64 + ij * 32 + l);
            }
        n = 0;
        for (int i = 0; i < NWG * NWAVE; ++i) t.conv[i] = NOCONV;
        if (MOVCONV)
            for (int w = 0; w < PW1; ++w)
                for (int b = MOV0; b < MOV0 + NMOV; ++b) t.conv[b * NWAVE + w] = ord[n++];
        else
            for (int w = 0; w < 2; ++w)
                for (int b = 0; b < NGEMM; ++b)
                    if (conv_cap(b) == 0) t.conv[b * NWAVE + w] = ord[n++];
        for (int r = 0; r < 60; ++r)
            for (int b = 0; b < NWG; ++b) {
                if (b >= MOV0 && b < MOV0 + NMOV) continue;
                const int cap = conv_cap(b);
                for (int w = 0; w < cap; ++w)
                    if (60 * w / cap == r && n < 1024) t.conv[b * NWAVE + w] = ord[n++] | (unsigned)(r / 5 + 1) << 16;
            }
        t.nconv = n;
    }
    constexpr int T_STEP = TSTEP, T_MARGIN = 100, T_GEMM0 = 700;
    auto arrive = [](int pos) { return 410 + (pos + 16) * 1035 / 100 + 200; };
    for (int li = 0; li < 128; ++li) {
        const int rt = (li >> 3) & 7, g = li & 7;
        const bool hasS = li >= 64 || owner_runs_s(rt, g), hasO = li < 64;
        int av[NSTEP] = {};
        bool used[NSTEP] = {};
        for (int so = 0; so < 2; ++so)
            for (int kc = 0; kc < 32; ++kc) {
                int k = T_GEMM0 - T_MARGIN;
                if (kc >= 5) {
                    const int L = kc < 20 ? kc / 5 - 1 : 3;
                    const int c = kc < 20 ? kc % 5 : 5 + (kc - 20) % 3;
                    const int a = posX[L][c][so * 16 + 2 * rt + 1], b = posW[L][c][16 + 2 * g + 1];
                    k = arrive(a > b ? a : b);
                }
                av[so * 32 + kc] = k;
                used[so * 32 + kc] = so ? !hasO : !hasS;
            }
        av[64] = arrive(posP[((rt & 3) << 5) | (g << 2) | 3]) + ((rt & 3) == 0 && g >= 2 ? 1000 : PMARGIN);
        used[64] = !hasO;
        const int ns = (hasS ? 32 : 0) + (hasO ? 33 : 0);
        int code[NSTEP] = {};
        int now = T_GEMM0;
        for (int n = 0; n < ns; ++n) {
            int bs = -1, bo = -1;
            for (int i = 0; i < 32; ++i) if (!used[i] && (bs < 0 || av[i] < av[bs])) bs = i;
            for (int i = 32; i < (bs >= 0 ? 64 : 65); ++i) if (!used[i] && (bo < 0 || av[i] < av[bo])) bo = i;
            int pick = bs;
            if (bs < 0) pick = bo;
            else if (g >= SFIRST && av[bs] > now && bo >= 0 && av[bo] + T_MARGIN < av[bs]) pick = bo;
            const int ready = pick >= 32 && bs >= 0 ? av[pick] + T_MARGIN : av[pick];
            now = (ready > now ? ready : now) + T_STEP;
            used[pick] = true;
            code[n] = pick;
        }
        for (int a = 1; a + 1 < ns; ++a)
            if ((code[a] >> 5) == 2 && (code[a - 1] >> 5) != 1) { const int cc = code[a]; code[a] = code[a + 1]; code[a + 1] = cc; }
        for (int a = 0; a < 67; ++a) t.step[li][a] = (unsigned char)(a < ns ? code[a] : 3 << 5);
        t.step[li][67] = (unsigned char)ns;
    }
    return t;
}
constexpr bool tables_ok() {
    const Tables t = make_tables();
    if (t.nconv != 1024) return false;
    for (int w = 0; w < 64; ++w) {
        int lastS = -1, sP = -1;
        for (int s = 0; s < NSTEP; ++s) {
            if ((t.step[w][s] >> 5) == 0) lastS = s;
            if ((t.step[w][s] >> 5) == 2) sP = s;
        }
        if (lastS > 62 || sP <= lastS || sP < 1 || (t.step[w][sP - 1] >> 5) != 1) return false;
    }
    return true;
}
static_assert(tables_ok(), "every block is converted; the partial step follows an O step and the last S step");
__constant__ const Tables TB = make_tables();

struct Params {
    unsigned char* ws[NR];
    const float* x;
    const float* w;
    signed char* out;
    int rank;
    int pad;
};

__device__ __forceinline__ u64 rt_now() { return __builtin_amdgcn_s_memrealtime(); }
__device__ __forceinline__ u32 rfl(u32 v) { return __builtin_amdgcn_readfirstlane(v); }
__device__ __forceinline__ u32 rlane(u32 v, u32 l) { return __builtin_amdgcn_readlane(v, l); }
__device__ __forceinline__ __amdgpu_buffer_rsrc_t mkrs(const void* p, u32 bytes) {
    return __builtin_amdgcn_make_buffer_rsrc((void*)p, 0, (int)bytes, RSRC3);
}
__device__ __forceinline__ v4u ld128(__amdgpu_buffer_rsrc_t rs, u32 voff, u32 soff) {
    return __builtin_amdgcn_raw_buffer_load_b128(rs, voff, soff, AUX_SYS);
}
__device__ __forceinline__ void st128(v4u v, __amdgpu_buffer_rsrc_t rs, u32 voff, u32 soff) {
    __builtin_amdgcn_raw_buffer_store_b128(v, rs, voff + soff, 0, AUX_SYS);
}
#define VM_WAIT(n) asm volatile("s_waitcnt vmcnt(" #n ")" ::: "memory")
#define LGKM0() asm volatile("s_waitcnt lgkmcnt(0)" ::: "memory")
#define SCHED() __builtin_amdgcn_sched_barrier(0)
__device__ __forceinline__ u32 cvt2(float a, float b) {
    return __builtin_bit_cast(u32, __builtin_convertvector(f32x2{a, b}, bf16x2));
}
__device__ __forceinline__ u32 perm(u32 s0, u32 s1, u32 sel) { return __builtin_amdgcn_perm(s0, s1, sel); }
__device__ __forceinline__ u32 wave_max_u32(u32 v) {
    u32 t;
    t = __builtin_amdgcn_update_dpp(0u, v, 0x111, 0xf, 0xf, true); v = v > t ? v : t;
    t = __builtin_amdgcn_update_dpp(0u, v, 0x112, 0xf, 0xf, true); v = v > t ? v : t;
    t = __builtin_amdgcn_update_dpp(0u, v, 0x114, 0xf, 0xf, true); v = v > t ? v : t;
    t = __builtin_amdgcn_update_dpp(0u, v, 0x118, 0xf, 0xf, true); v = v > t ? v : t;
    t = __builtin_amdgcn_update_dpp(0u, v, 0x142, 0xa, 0xf, false); v = v > t ? v : t;
    t = __builtin_amdgcn_update_dpp(0u, v, 0x143, 0xc, 0xf, false); v = v > t ? v : t;
    return rlane(v, 63);
}
__device__ __forceinline__ void lds_st(u32 addr, u32 v) { asm volatile("ds_write_b32 %0, %1" ::"v"(addr), "v"(v) : "memory"); }
__device__ __forceinline__ v4u lds_ld4(u32 addr) {
    v4u v;
    asm volatile("ds_read_b128 %0, %1\n\ts_waitcnt lgkmcnt(0)" : "=&v"(v) : "v"(addr) : "memory");
    return v;
}
__device__ __forceinline__ u32 lds_ld(u32 addr) {
    u32 v;
    asm volatile("ds_read_b32 %0, %1\n\ts_waitcnt lgkmcnt(0)" : "=&v"(v) : "v"(addr) : "memory");
    return v;
}
__device__ __forceinline__ void glds4(const void* gsrc, u32 lds_dst) {
    u32 keep;
    asm volatile(
        "s_mov_b32 %0, m0\n\ts_mov_b32 m0, %2\n\ts_nop 0\n\t"
        "global_load_lds_dwordx4 %1, off sc0 sc1\n\t"
        "global_load_lds_dwordx4 %1, off offset:1024 sc0 sc1\n\t"
        "global_load_lds_dwordx4 %1, off offset:2048 sc0 sc1\n\t"
        "global_load_lds_dwordx4 %1, off offset:3072 sc0 sc1\n\t"
        "s_mov_b32 m0, %0"
        : "=&s"(keep) : "v"(gsrc), "s"(lds_dst) : "memory");
}

struct Ctx {
    unsigned char* me;
    u64 t0;
    u32 call;
    int lane;
    bool dead;
};
__device__ __forceinline__ bool give_up(Ctx& c) {
    GAS u32* tmo = (GAS u32*)(c.me + OFF_TMO);
    const u32 t = rfl(__hip_atomic_load(tmo, __ATOMIC_RELAXED, __HIP_MEMORY_SCOPE_AGENT));
    if (t == c.call || rt_now() - c.t0 > DEADLINE) {
        __hip_atomic_store(tmo, c.call, __ATOMIC_RELAXED, __HIP_MEMORY_SCOPE_AGENT);
        c.dead = true;
        return true;
    }
    return false;
}

#define STAMP(i) do {} while (0)
#define RSTAMP(V, i) do {} while (0)
#define RDUMP(V, o) do {} while (0)
#define BAR() asm volatile("s_barrier" ::: "memory")

__device__ __forceinline__ u32 bfi(u32 m, u32 a, u32 b) { return (a & m) | (b & ~m); }
__device__ __forceinline__ u32 mbcnt(u64 m) { return __builtin_amdgcn_mbcnt_hi((u32)(m >> 32), __builtin_amdgcn_mbcnt_lo((u32)m, 0u)); }

__device__ __forceinline__ u32 encode_block(const u32 (&D)[32], u32 hint, __amdgpu_buffer_rsrc_t rs, u32 soff, int lane) {
    const u32 vo = (u32)lane * 16u;
    const u32 e7 = wave_max_u32(hint) >> 24;
    const u32 base = e7 > 7u ? e7 - 7u : 0u;
    const u32 base4 = base * 0x01010101u;
#pragma unroll
    for (int k = 0; k < 4; ++k) {
        v4u lo;
#pragma unroll
        for (int e = 0; e < 4; ++e) lo[e] = perm(D[8 * k + 2 * e + 1], D[8 * k + 2 * e], 0x06040200u);
        st128(lo, rs, vo + (k >> 1) * 3072 + (k & 1) * 1024, soff);
    }
    u32 U[16], bad[4] = {0u, 0u, 0u, 0u};
#pragma unroll
    for (int i = 0; i < 16; ++i) {
        U[i] = perm(D[2 * i + 1], D[2 * i], 0x07050301u) - base4;
        bad[i >> 2] |= U[i] & 0x78787878u;
    }
#pragma unroll
    for (int u = 0; u < 2; ++u) {
        v4u nn;
#pragma unroll
        for (int e = 0; e < 4; ++e) nn[e] = bfi(0x87878787u, U[8 * u + 2 * e], __builtin_amdgcn_alignbit(U[8 * u + 2 * e + 1], U[8 * u + 2 * e + 1], 28));
        st128(nn, rs, vo + u * 3072 + 2048, soff);
    }
    if (__builtin_amdgcn_ballot_w64((bad[0] | bad[1] | bad[2] | bad[3]) != 0) == 0) return base;
    u32 cnt = 0;
#pragma unroll
    for (int q = 0; q < 4; ++q)
        if (__builtin_amdgcn_ballot_w64(bad[q] != 0) != 0) {
#pragma unroll
            for (int i = 4 * q; i < 4 * q + 4; ++i) {
                const bool b = (U[i] & 0x78787878u) != 0;
                const u64 m = __builtin_amdgcn_ballot_w64(b);
                const u32 idx = cnt + mbcnt(m);
                st128(v4u{D[2 * i], D[2 * i + 1], vo + (u32)((i >> 1) * 1024 + (i & 1) * 8), 0u}, rs,
                      b && idx < 64u ? 6144u + idx * 16u : OOB, soff);
                cnt += (u32)__builtin_popcountll(m);
            }
        }
    if (cnt <= 64u) return base | (cnt << 7);
    VM_WAIT(0);
#pragma unroll
    for (int f = 0; f < 8; ++f) st128(v4u{D[4 * f], D[4 * f + 1], D[4 * f + 2], D[4 * f + 3]}, rs, vo + f * 1024, soff);
    return 1u << 14;
}
__device__ __forceinline__ void decode_block(const v4u (&pk)[6], u32 info, v4u (&out)[8]) {
    const u32 base4 = (info & 127u) * 0x01010101u;
#pragma unroll
    for (int f = 0; f < 8; ++f) {
        const u32 nn = pk[3 * (f >> 2) + 2][f & 3];
        const u32 ha = (nn & 0x87878787u) + base4, hb = (__builtin_amdgcn_alignbit(nn, nn, 4) & 0x87878787u) + base4;
        const v4u lp = pk[3 * (f >> 2) + ((f >> 1) & 1)];
        const u32 la = lp[2 * (f & 1)], lb = lp[2 * (f & 1) + 1];
        out[f] = v4u{perm(ha, la, 0x05010400u), perm(ha, la, 0x07030602u), perm(hb, lb, 0x05010400u), perm(hb, lb, 0x07030602u)};
    }
}

__device__ __forceinline__ u32 encode_q(const v4u (&q)[4], __amdgpu_buffer_rsrc_t rs, u32 soff, int lane) {
    u32 P[14], M[2];
#pragma unroll
    for (int g = 0; g < 2; ++g) {
        const u32 top = q[2 * g + 1][3];
        u32 m = 0;
#pragma unroll
        for (int i = 0; i < 8; ++i) {
            const u32 x = q[2 * g + (i >> 2)][i & 3];
            if (i < 7) P[7 * g + i] = bfi(0x80808080u, top << (7 - i), x);
            m |= ((x ^ (x << 1)) & 0x80808080u) >> i;
        }
        M[g] = m;
    }
    const u32 vo = (u32)lane * 16u;
#pragma unroll
    for (int k = 0; k < 3; ++k) st128(v4u{P[4 * k], P[4 * k + 1], P[4 * k + 2], P[4 * k + 3]}, rs, vo + k * 1024, soff);
    __builtin_amdgcn_raw_buffer_store_b64(v2u{P[12], P[13]}, rs, 3072u + (u32)lane * 8u, soff, AUX_SYS);
    const u64 b0 = __builtin_amdgcn_ballot_w64(M[0] != 0), b1 = __builtin_amdgcn_ballot_w64(M[1] != 0);
    const u32 n0 = (u32)__builtin_popcountll(b0);
    st128(v4u{(u32)b0, (u32)(b0 >> 32), (u32)b1, (u32)(b1 >> 32)}, rs, lane == 0 ? 3584u : OOB, soff);
    __builtin_amdgcn_raw_buffer_store_b32(M[0], rs, M[0] != 0 ? 3600u + mbcnt(b0) * 4u : OOB, soff, AUX_SYS);
    __builtin_amdgcn_raw_buffer_store_b32(M[1], rs, M[1] != 0 ? 3600u + (n0 + mbcnt(b1)) * 4u : OOB, soff, AUX_SYS);
    return n0 + (u32)__builtin_popcountll(b1);
}
__device__ __forceinline__ void decode_q(const v4u (&pk)[3], v2u pk3, v4u ex, LAS unsigned char* scr, int lane, v4u (&q)[4]) {
    *(LAS v4u*)(scr + lane * 16) = ex;
    const u64 b0 = (u64)rlane(ex.x, 0) | (u64)rlane(ex.y, 0) << 32, b1 = (u64)rlane(ex.z, 0) | (u64)rlane(ex.w, 0) << 32;
    const u32 n0 = (u32)__builtin_popcountll(b0);
    u32 M[2];
    M[0] = *(const LAS u32*)(scr + 16 + mbcnt(b0) * 4);
    M[1] = *(const LAS u32*)(scr + 16 + (n0 + mbcnt(b1)) * 4);
    const u32 P[14] = {pk[0][0], pk[0][1], pk[0][2], pk[0][3], pk[1][0], pk[1][1], pk[1][2], pk[1][3],
                       pk[2][0], pk[2][1], pk[2][2], pk[2][3], pk3.x, pk3.y};
    u32 top[2];
#pragma unroll
    for (int g = 0; g < 2; ++g) {
        u32 t = 0;
#pragma unroll
        for (int i = 0; i < 7; ++i) t = bfi(0x01010101u << i, P[7 * g + i] >> (7 - i), t);
        top[g] = t;
    }
    M[0] = (b0 >> lane) & 1ull ? M[0] : 0u;
    M[1] = (b1 >> lane) & 1ull ? M[1] : 0u;
#pragma unroll
    for (int g = 0; g < 2; ++g)
#pragma unroll
        for (int i = 0; i < 8; ++i) {
            const u32 x = i < 7 ? P[7 * g + i] : top[g];
            q[2 * g + (i >> 2)][i & 3] = bfi(0x80808080u, (x << 1) ^ (M[g] << i), x);
        }
}

__device__ __forceinline__ void st_flag64(unsigned char* p, u32 call, u32 info) {
    __hip_atomic_store((GAS u64*)p, ((u64)call << 32) | info, __ATOMIC_RELAXED, __HIP_MEMORY_SCOPE_SYSTEM);
}
__device__ __forceinline__ void st_flag32(unsigned char* p, u32 call) {
    __hip_atomic_store((GAS u32*)p, call, __ATOMIC_RELAXED, __HIP_MEMORY_SCOPE_SYSTEM);
}
__device__ __forceinline__ u32 row_block(u32 G, u32 i, u32 l) {
    const u32 so = l >> 4, q = (l >> 3) & 1, idx = l & 7;
    const u32 a = G ? i + 2 * q : 2 * i + q, hf = G ? 1 - so : so;
    return 16 * a + 8 * hf + idx;
}
__device__ __forceinline__ u32 col_block(u32 G, u32 j, u32 l) {
    const u32 q = l >> 4, idx = l & 15;
    return 16 * (G ? j + 2 * q : 2 * j + q) + idx;
}

__device__ __forceinline__ void convert_block(const Params& p, Ctx& cx, u32 slot, u32 cnt_v, __amdgpu_buffer_rsrc_t rsMe, u32 wave) {
    const int lane = cx.lane;
    const u32 rank = p.rank, G = rank >> 2, myi = (rank >> 1) & 1, myj = rank & 1;
    const u32 kind = slot >> 9, c = (slot >> 6) & 7, ij = (slot >> 5) & 1, l = slot & 31;
    const u32 dG = c < 5 ? G : 1 - G;
    const u32 d0 = (dG * 4 + (kind ? ij : ij * 2)) & 7, d1 = (d0 + (kind ? 2 : 1)) & 7;
    unsigned char* const w0 = p.ws[d0];
    unsigned char* const w1 = p.ws[d1];
    u32 D[32];
    float mx = 0.f;
    const u32 r16 = lane & 15, g4 = lane >> 4;
    if (kind == 0) {
        const u32 rb = row_block(dG, ij, l);
        const GAS f32x4* src = (const GAS f32x4*)(p.x + (size_t)(64 * rb + r16) * KS + 64 * c + 4 * g4);
        f32x4 v[16];
#pragma unroll
        for (int mt = 0; mt < 4; ++mt)
#pragma unroll
            for (int i = 0; i < 4; ++i) v[mt * 4 + i] = src[mt * 16 * (KS / 4) + i * 4];
        SCHED();
#pragma unroll
        for (int mt = 0; mt < 4; ++mt)
#pragma unroll
            for (int i = 0; i < 4; ++i) {
                const f32x4 a = v[mt * 4 + i];
                const int f = (i >> 1) * 4 + mt, o = 4 * f + 2 * (i & 1);
                D[o] = cvt2(a.x, a.y);
                D[o + 1] = cvt2(a.z, a.w);
                if (i == 0) mx = fmaxf(fmaxf(fmaxf(fmaxf(mx, fabsf(a.x)), fabsf(a.y)), fabsf(a.z)), fabsf(a.w));
            }
    } else {
        const u32 cb = col_block(dG, ij, l);
        const GAS f32x4* src = (const GAS f32x4*)(p.w + (size_t)(64 * c + 4 * g4) * NN + 64 * cb + 4 * r16);
        f32x4 v[16];
#pragma unroll
        for (int j = 0; j < 16; ++j) v[j] = src[((j >> 2) * 16 + (j & 3)) * (NN / 4)];
        SCHED();
#pragma unroll
        for (int q = 0; q < 2; ++q)
#pragma unroll
            for (int pp = 0; pp < 4; ++pp) {
                const f32x4 a = v[q * 8 + 2 * pp], b = v[q * 8 + 2 * pp + 1];
#pragma unroll
                for (int t = 0; t < 4; ++t) {
                    D[4 * (q * 4 + t) + pp] = cvt2(a[t], b[t]);
                    if (pp == 0) mx = fmaxf(fmaxf(mx, fabsf(a[t])), fabsf(b[t]));
                }
            }
    }
    STAMP(28);
    cx.call = rfl(cnt_v) + 1u;
    const u32 info = encode_block(D, __builtin_bit_cast(u32, mx), rsMe, OFF_EXP + slot * BLK, lane);
    STAMP(29);
    VM_WAIT(0);
    const u32 fo = OFF_PF + rank * PF_SRC + (kind * 256 + c * 32 + l) * 8;
    if (lane == 0) {
        if (d0 != rank) st_flag64(w0 + fo, cx.call, info);
        if (d1 != rank) st_flag64(w1 + fo, cx.call, info);
    }
    if (d0 == rank || d1 == rank) {
        const u32 so = OFF_OP + ((kind * 32 + l) * 32 + c) * BLK;
#pragma unroll
        for (int f = 0; f < 8; ++f) st128(v4u{D[4 * f], D[4 * f + 1], D[4 * f + 2], D[4 * f + 3]}, rsMe, (u32)lane * 16u + f * 1024, so);
        VM_WAIT(0);
        if (lane == 0) st_flag32(cx.me + OFF_GF + (kind * 32 + l) * 128 + c * 4, cx.call);
    }
}

__global__ void __launch_bounds__(768, 3) fused_gemm_ar_q8(Params p) {
    __shared__ __attribute__((aligned(16))) unsigned char smem[LDS_BYTES];
    const int tid = threadIdx.x, lane = tid & 63;
    const u32 wave = rfl(tid >> 6);
    const u32 bid = blockIdx.x;
    const bool is_gemm = bid < (u32)MOV0 || bid >= (u32)(MOV0 + NMOV);
    if (!is_gemm && wave >= (u32)PW2) return;
    const u32 cslot = TB.conv[bid * NWAVE + wave];
    const u32 rank = p.rank, G = rank >> 2, U = rank & 3, myi = U >> 1, myj = U & 1;
    Ctx cx;
    cx.me = p.ws[rank];
    cx.t0 = rt_now();
    cx.lane = lane;
    cx.dead = false;
    GAS u32* cntp = (GAS u32*)(cx.me + OFF_CNT) + bid * NWAVE + wave;
    const u32 cnt_v = __hip_atomic_load(cntp, __ATOMIC_RELAXED, __HIP_MEMORY_SCOPE_AGENT);
    const __amdgpu_buffer_rsrc_t rsMe = mkrs(cx.me, WS_USED);
    const u32 vo16 = (u32)lane * 16u;
    STAMP(0);
    if (cslot != NOCONV) {
        for (u32 s = cslot >> 16; s > 0; --s) __builtin_amdgcn_s_sleep(CONV_SLEEP);
        convert_block(p, cx, cslot & 1023u, cnt_v, rsMe, wave);
    }
    cx.call = rfl(cnt_v) + 1u;
    if (lane == 0) *cntp = cx.call;
    STAMP(30);

    if (is_gemm) {
        const u32 lds0 = (u32)(size_t)(LAS unsigned char*)smem;
        const u32 xc = bid & 7, k = bid >> 3;
        const bool owner = bid < (u32)NGEMM;
        const bool last8 = bid >= (u32)(MOV0 + NMOV);
        const u32 rt = 4 * (xc & 1) + (last8 ? 0u : k & 3);
        const u32 g = owner ? 2 * (xc >> 1) + ((k >> 2) & 1) : last8 ? 2 + ((xc >> 1) & 1) : 4 + (xc >> 1);
        const u32 ct = g + 8 * (owner ? k >> 3 : last8 ? xc >> 2 : (k >> 2) & 1);
        const u32 tile = rt * 16 + ct;
        const bool isY = owner && (rt & 3) == 0 && g < (u32)YG;
        if (wave < 8 && (wave < 4 ? !owner : owner && !owner_runs_s(rt, g))) return;
        const u32 li = (owner ? 0u : 64u) + rt * 8 + g;
        const u32 code0 = rfl(TB.step[li][0]), nstep = rfl(TB.step[li][67]);
        const u32 codeV = TB.step[li][1 + lane];

        if (wave >= 8) {
            const u32 j = wave - 8;
            auto blk_of = [&](u32 code, u32& boff, u32& foff) {
                const u32 ty = code >> 5, kc = code & 31;
                const u32 row = j < 2 ? (ty & 1) * 16 + 2 * rt + j : 32 + 2 * ct + (j - 2);
                boff = ty == 2 ? OFF_OPP + (tile * 4 + j) * BLK : OFF_OP + (row * 32 + kc) * BLK;
                foff = ty == 2 ? OFF_GP + tile * 128 + j * 4 : OFF_GF + row * 128 + kc * 4;
            };
            u32 b0, f0, bV, fV;
            blk_of(code0, b0, f0);
            blk_of(codeV, bV, fV);
            const u32 ringj = lds0 + j * BLK;
            const unsigned char* gl = cx.me + vo16;
            GAS u32* fp = (GAS u32*)(cx.me + fV);
            {
                u32 spins = 0;
                while (rfl(__hip_atomic_load((GAS u32*)(cx.me + f0), __ATOMIC_RELAXED, __HIP_MEMORY_SCOPE_SYSTEM)) < cx.call) {
                    __builtin_amdgcn_s_sleep(2);
                    if ((++spins & 63u) == 0u && give_up(cx)) break;
                }
                VM_WAIT(0);
                glds4(gl + b0, ringj);
                glds4(gl + b0 + 4096, ringj + 4096);
            }
            u64 ready = cx.dead ? ~0ull : 0ull;
            u32 I = 1, dst = SLOT_B;
            auto issue_upto = [&](u32 last) {
                while (I <= last && I < nstep && ((ready >> (I - 1)) & 1ull)) {
                    if (STAMPS > 1) RSTAMP(stA, I);
                    const u32 bo = rlane(bV, I - 1);
                    glds4(gl + bo, ringj + dst);
                    glds4(gl + bo + 4096, ringj + dst + 4096);
                    ++I;
                    dst = dst == (NSLOT - 1) * SLOT_B ? 0u : dst + SLOT_B;
                }
            };
            bool pollp = false;
            u32 fv = 0, Ip = 0;
#define FLAG_READ() asm volatile("global_load_dword %0, %1, off sc0 sc1" : "=v"(fv) : "v"(fp) : "memory")
#define FLAG_WAIT(n) asm volatile("s_waitcnt vmcnt(" #n ")" : "+v"(fv) :: "memory")
            for (u32 s = 0; s < nstep; ++s) {
                if (I <= s) {
                    u32 spins = 0;
                    for (;;) {
                        if (!pollp) FLAG_READ();
                        pollp = false;
                        FLAG_WAIT(0);
                        ready = __builtin_amdgcn_ballot_w64(fv >= cx.call);
                        if ((ready >> (s - 1)) & 1ull) break;
                        __builtin_amdgcn_s_sleep(LD_SLEEP);
                        if ((++spins & 63u) == 0u && give_up(cx)) { ready = ~0ull; break; }
                    }
                    VM_WAIT(0);
                    issue_upto(s + 2);
                }
                static_assert(NSLOT >= 4, "three blocks may go out ahead of a barrier");
                const u32 ahead = I - s - 1;
                if (ahead == 0) VM_WAIT(0); else if (ahead == 1) VM_WAIT(8); else if (ahead == 2) VM_WAIT(16); else VM_WAIT(24);
                if (STAMPS > 1) RSTAMP(stB, s);
                BAR();
                if (pollp) {
                    if (I == Ip) VM_WAIT(0); else if (I == Ip + 1) VM_WAIT(8);
                    FLAG_WAIT(16);
                    ready = __builtin_amdgcn_ballot_w64(fv >= cx.call);
                    pollp = false;
                }
                if (I < nstep) {
                    const u32 nr = (u32)__builtin_ctzll(~(ready >> (I - 1)) | (1ull << 63)) + I;
                    if (nr <= s + LOOK && nr < nstep) { FLAG_READ(); pollp = true; Ip = I; }
                    const u32 cap = I >= s + 1 + LCAP ? I : I + 1;
                    issue_upto(cap < s + NSLOT - 1 ? cap : s + NSLOT - 1);
                }
            }
            VM_WAIT(0);
            if (STAMPS > 1) { STAMP(2); RDUMP(stA, 32); RDUMP(stB, 112); }
            return;
        }

        const u32 cw = wave & 3, wm = cw >> 1, wn = cw & 1;
        const u64 smask = (__builtin_amdgcn_ballot_w64((codeV >> 5) == 0) << 1) | (u64)((code0 >> 5) == 0);
        unsigned char* const peer = p.ws[lane & 7];
        f32x4 acc[4][4];
#pragma unroll
        for (int a = 0; a < 4; ++a)
#pragma unroll
            for (int b = 0; b < 4; ++b) acc[a][b] = f32x4{0.f, 0.f, 0.f, 0.f};
        const LAS unsigned char* ring = (const LAS unsigned char*)smem + vo16;
        u32 s = 0, so = 0;
#define NEXT() do { ++s; so = so == (NSLOT - 1) * SLOT_B ? 0u : so + SLOT_B; } while (0)
#define RD(XF, WF, q) do { \
        _Pragma("unroll") for (int t = 0; t < 4; ++t) WF[t] = *(const LAS v4u*)(ring + so + (2 + wn) * BLK + ((q) * 4 + t) * 1024); \
        _Pragma("unroll") for (int t = 0; t < 4; ++t) XF[t] = *(const LAS v4u*)(ring + so + wm * BLK + ((q) * 4 + t) * 1024); \
    } while (0)
#define MIX() _Pragma("unroll") for (int i_ = 0; i_ < 8; ++i_) { __builtin_amdgcn_sched_group_barrier(0x100, 1, 0); __builtin_amdgcn_sched_group_barrier(0x8, 1, 0); }
#define MMA(XF, WF) \
    _Pragma("unroll") for (int mt = 0; mt < 4; ++mt) _Pragma("unroll") for (int t = 0; t < 4; ++t) \
        acc[mt][t] = __builtin_amdgcn_mfma_f32_16x16x32_bf16(__builtin_bit_cast(bf16x8, WF[t]), __builtin_bit_cast(bf16x8, XF[mt]), acc[mt][t], 0, 0, 0)
        {
            const u64 mine = __builtin_amdgcn_ballot_w64((codeV >> 5) == (wave < 4 ? 1u : 0u));
            const u32 sP = wave < 4 ? 1u + (u32)__builtin_ctzll(__builtin_amdgcn_ballot_w64((codeV >> 5) == 2)) : 999u;
            const u32 send = wave < 4 ? nstep : 64u - (u32)__builtin_clzll(smask);
            auto add_partial = [&]() {
                BAR();
                RSTAMP(stA, s);
                v4u pv[8];
#pragma unroll
                for (int f = 0; f < 8; ++f) pv[f] = *(const LAS v4u*)(ring + so + cw * BLK + f * 1024);
#pragma unroll
                for (int i = 0; i < 16; ++i) {
                    const u32 u0 = pv[i >> 1][2 * (i & 1)], u1 = pv[i >> 1][2 * (i & 1) + 1];
                    acc[i >> 2][i & 3] += __builtin_bit_cast(f32x4, v4u{u0 << 16, u0 & 0xFFFF0000u, u1 << 16, u1 & 0xFFFF0000u});
                }
                LGKM0();
                NEXT();
            };
            do {
                while ((((smask >> (s & 63u)) & 1ull) != 0 && s < 64u) != (wave >= 4)) { BAR(); NEXT(); }
                BAR();
                RSTAMP(stA, s);
                const u64 rest = s < 64u ? ~(mine >> s) : ~0ull;
                const u32 e = s + 1 + (rest ? (u32)__builtin_ctzll(rest) : 64u);
                v4u xa[4], wa[4], xb[4], wb[4];
                RD(xa, wa, 0); RD(xb, wb, 1); SCHED();
                MMA(xa, wa); SCHED();
                LGKM0(); NEXT();
                while (s < e) {
                    BAR();
                    RSTAMP(stA, s);
                    RD(xa, wa, 0); MMA(xb, wb); MIX(); SCHED();
                    RD(xb, wb, 1); MMA(xa, wa); MIX(); SCHED();
                    LGKM0(); NEXT();
                }
                MMA(xb, wb); SCHED();
                if (s == sP) add_partial();
            } while (s < send);
        }
#pragma unroll
        for (int i = 0; i < 16; ++i) asm volatile("" : "+v"(acc[i >> 2][i & 3]));
        if (wave >= 4 || isY) {
            STAMP(3);
            u32 D[32];
            float mx = 0.f;
#pragma unroll
            for (int i = 0; i < 32; ++i) {
                const int mt = i >> 3, t = (i >> 1) & 3, h = i & 1;
                const float a = acc[mt][t][2 * h], b = acc[mt][t][2 * h + 1];
                D[i] = cvt2(a, b);
                if (t == 0) mx = fmaxf(fmaxf(mx, fabsf(a)), fabsf(b));
            }
            const u32 d = (1 - G) * 4 + (rt >> 2) * 2 + (ct >> 3);
            const u32 yid = (g << 4) | ((rt >> 2) << 3) | ((ct >> 3) << 2) | cw;
            const u32 info = encode_block(D, __builtin_bit_cast(u32, mx), rsMe,
                                          wave >= 4 ? OFF_EXPP + (tile * 4 + cw) * BLK : OFF_EXPY + yid * BLK, lane);
            VM_WAIT(0);
            const u32 fi = wave >= 4 ? PF_P + (((rt & 3) << 5) | (g << 2) | cw) : PF_Y + yid;
            if (wave >= 4 ? (u32)lane == d : lane < NR && (u32)lane != rank) st_flag64(peer + OFF_PF + rank * PF_SRC + fi * 8, cx.call, info);
            STAMP(4);
            if (wave >= 4) {
                RDUMP(stA, 32);
                for (; s < nstep; ++s) BAR();
                return;
            }
        }
        STAMP(5);
        u32 amax_bits;
        {
            float mx = 0.f;
#pragma unroll
            for (int a = 0; a < 4; ++a)
#pragma unroll
                for (int b = 0; b < 4; ++b) {
                    mx = fmaxf(fmaxf(mx, fabsf(acc[a][b][0])), fabsf(acc[a][b][1]));
                    mx = fmaxf(fmaxf(mx, fabsf(acc[a][b][2])), fabsf(acc[a][b][3]));
                }
            const u32 post = lds0 + so;
            lds_st(post + wave * 4, wave_max_u32(__builtin_bit_cast(u32, mx)));
            LGKM0();
            BAR();
            if (wave == 0) {
                const v4u m4 = lds_ld4(post);
                const u32 m0 = m4.x > m4.y ? m4.x : m4.y, m1 = m4.z > m4.w ? m4.z : m4.w;
                unsigned char* acc2 = cx.me + OFF_ACC + ((cx.call & 1u) * 8 + (bid & 7)) * 128;
                u64 old = 0;
                if (lane == 0) {
                    (void)__hip_atomic_fetch_max((GAS u32*)acc2, m0 > m1 ? m0 : m1, __ATOMIC_RELAXED, __HIP_MEMORY_SCOPE_AGENT);
                    asm volatile("" ::: "memory");
                    old = __hip_atomic_fetch_add((GAS u64*)acc2, 1ull << 32, __ATOMIC_RELAXED, __HIP_MEMORY_SCOPE_AGENT);
                }
                if (rfl((u32)(old >> 32)) == (u32)NGEMM / 8 - 1u)
                    st_flag64(peer + OFF_AM + ((((u32)lane >> 3) * 8 + rank) * 8 + (bid & 7)) * 128, cx.call, rfl((u32)old));
                u32 pvo = OFF_AM + ((bid & 7) * 64 + (u32)lane) * 128;
                u32 spins = 0, m = 0;
                for (;;) {
                    const v2u a = __builtin_amdgcn_raw_buffer_load_b64(rsMe, pvo, 0, AUX_SYS);
                    const u32 tm = __builtin_amdgcn_raw_buffer_load_b32(rsMe, 0, OFF_TMO, AUX_SYS);
                    if (a.y >= cx.call) { m = a.x; pvo = OOB; }
                    if (rfl(tm) == cx.call) cx.dead = true;
                    if (__builtin_amdgcn_ballot_w64(pvo != OOB) == 0 || cx.dead) break;
                    __builtin_amdgcn_s_sleep(1);
                    if ((++spins & 63u) == 0u && give_up(cx)) break;
                }
                lds_st(post + 16, cx.dead ? 0xFFFFFFFFu : wave_max_u32(m));
                LGKM0();
            }
            BAR();
            amax_bits = rfl(lds_ld(post + 16));
            if (amax_bits == 0xFFFFFFFFu) cx.dead = true;
        }
        STAMP(6);
        {
            const float amax = __builtin_bit_cast(float, amax_bits);
            const float scale = amax / 127.0f;
            const float inv = scale > 0.f ? 1.0f / scale : 0.f;
            const float MAGIC = 12582912.0f;
            const __amdgpu_buffer_rsrc_t rsOut = mkrs(p.out, (u32)MM * NN);
            const u32 ovo = (u32)(lane & 15) * NN + (u32)(lane >> 4) * 16u;
            u32 rb = row_block(G, myi, 16 + 2 * rt + wm), cb = col_block(G, myj, 2 * ct + wn);
            const u32 ow = tile * 4 + wave;
            for (int pass = 0;; ++pass) {
                v4u qv[4];
#pragma unroll
                for (int mt = 0; mt < 4; ++mt)
#pragma unroll
                    for (int r = 0; r < 4; ++r) {
                        u32 tq[4];
#pragma unroll
                        for (int t = 0; t < 4; ++t) {
                            const float z = __builtin_amdgcn_fmed3f(__builtin_fmaf(acc[mt][t][r], inv, MAGIC), MAGIC - 127.0f, MAGIC + 127.0f);
                            tq[t] = __builtin_bit_cast(u32, z);
                        }
                        const u32 lo = perm(tq[1], tq[0], 0x0c0c0400u), hi = perm(tq[3], tq[2], 0x04000c0cu);
                        qv[mt][r] = cx.dead ? 0x80808080u : (lo | hi);
                    }
                if (pass == 0 && !isY) {
                    for (u32 i = g * QSTAG / 4; i > 0; --i) __builtin_amdgcn_s_sleep(4);
                    const u32 nz = encode_q(qv, rsMe, OFF_EXPO + ow * QSLOT, lane);
                    lds_st(lds0 + so + 32 + wave * 4, nz << (8 * wave));
                    VM_WAIT(0);
                    LGKM0();
                    BAR();
                    if (wave == 0) {
                        const v4u n4 = lds_ld4(lds0 + so + 32);
                        if (lane < NR && (u32)lane != rank)
                            st_flag64(peer + OFF_PF + rank * PF_SRC + (PF_O + tile) * 8, cx.call, n4.x | n4.y | n4.z | n4.w);
                    }
                }
#pragma unroll
                for (int mt = 0; mt < 4; ++mt) st128(qv[mt], rsOut, ovo + mt * 16 * NN, 64 * rb * NN + 64 * cb);
                if (pass == 0) STAMP(7);
                const u32 yj = ow + 512u * (u32)pass;
                if (yj >= 7u * NYB) break;
                const u32 ys = (rank ^ (1u + yj / (YG ? NYB : 1))) & 7u, yid = yj % (YG ? NYB : 1);
                u32 spins = 0;
                while (!cx.dead && rfl(__builtin_amdgcn_raw_buffer_load_b32(rsMe, 0, OFF_GY + (ys * 128 + yid) * 4, AUX_SYS)) < cx.call) {
                    __builtin_amdgcn_s_sleep(8);
                    if ((++spins & 63u) == 0u) give_up(cx);
                }
                v4u pv[8];
#pragma unroll
                for (int f = 0; f < 8; ++f) pv[f] = ld128(rsMe, vo16 + f * 1024, OFF_OPY + (ys * 128 + yid) * BLK);
#pragma unroll
                for (int i = 0; i < 16; ++i) {
                    const u32 u0 = pv[i >> 1][2 * (i & 1)], u1 = pv[i >> 1][2 * (i & 1) + 1];
                    acc[i >> 2][i & 3] = __builtin_bit_cast(f32x4, v4u{u0 << 16, u0 & 0xFFFF0000u, u1 << 16, u1 & 0xFFFF0000u});
                }
                rb = row_block(ys >> 2, (ys >> 1) & 1, 16 + 8 * ((yid >> 3) & 1) + ((yid >> 1) & 1));
                cb = col_block(ys >> 2, ys & 1, 2 * ((yid >> 4) + 8 * ((yid >> 2) & 1)) + (yid & 1));
            }
        }
        STAMP(13);
        RDUMP(stA, 32);
        return;
    }

    {
        const u32 mid = bid - MOV0, L = mid % 7u, cu = mid / 7u;
        const u32 v = L < 3 ? L + 1 : L - 3;
        const u32 sG = L < 3 ? G : 1 - G, sU = U ^ v, src = (sG * 4 + sU) & 7;
        const __amdgpu_buffer_rsrc_t rsSrc = mkrs(p.ws[src], WS_USED);
        const u32 pfb = OFF_PF + src * PF_SRC;
        const u32 tk = OFF_TK + ((cx.call & 1u) * 16 + L) * 128;
        const u32 l0 = lane == 0 ? 0u : OOB;
        if (cu == 0 && wave == 0 && lane < 2)
            __builtin_amdgcn_raw_buffer_store_b32(0u, rsMe, (u32)lane * 1024u, tk ^ 2048u, AUX_SYS);
        if (mid == 0 && wave == 0 && lane < 8)
            __builtin_amdgcn_raw_buffer_store_b64(v2u{0u, 0u}, rsMe, (u32)lane * 128u, OFF_ACC + ((cx.call & 1u) ^ 1u) * 1024, AUX_SYS);
        u32 pend = OOB;
        auto publish = [&]() {
            __builtin_amdgcn_raw_buffer_store_b32(cx.call, rsMe, lane == 0 ? pend : OOB, 0, AUX_SYS);
            pend = OOB;
        };
        auto poll = [&](u32 po) -> u32 {
            VM_WAIT(0);
            publish();
            u32 spins = 0;
            while (!cx.dead) {
                const v2u f = __builtin_amdgcn_raw_buffer_load_b64(rsMe, 0, po, AUX_SYS);
                if (rfl(f.y) >= cx.call) return rfl(f.x);
                __builtin_amdgcn_s_sleep(PL_SLEEP);
                if ((++spins & 63u) == 0u) give_up(cx);
            }
            return 0u;
        };
        const u32 jw = wave * 8 + cu;
        static_assert(NP1 <= NP2, "every phase-1 wave goes on to phase 2");
        u32 n = TB.outb[jw];
        u32 nz = 0;
        bool have = false;
        if (jw < (u32)NP1) {
            const u32* list = TB.pull[L < 3 ? L : 3];
            auto desc = [&](u32 n1, u32& so, u32& dso, u32& go, u32& po) {
                const u32 code = list[n1 < (u32)NTASK ? n1 : 0u];
                const u32 kind = code >> 8, c = (code >> 5) & 7, l = code & 31;
                if (kind < 2) {
                    const u32 kc = L < 3 ? 5 * v + c : 20 + 3 * v + (c - 5);
                    so = OFF_EXP + (kind * 512 + c * 64 + (kind ? myj : myi) * 32 + l) * BLK;
                    dso = OFF_OP + ((kind * 32 + l) * 32 + kc) * BLK;
                    go = OFF_GF + (kind * 32 + l) * 128 + kc * 4;
                    po = pfb + (kind * 256 + c * 32 + l) * 8;
                } else if (kind == 3) {
                    const u32 id = code & 127;
                    so = OFF_EXPY + id * BLK;
                    dso = OFF_OPY + (src * 128 + id) * BLK;
                    go = OFF_GY + (src * 128 + id) * 4;
                    po = pfb + (PF_Y + id) * 8;
                } else {
                    const u32 id = code & 127, r = id >> 5, gg = (id >> 2) & 7, cn = id & 3;
                    const u32 tS = (4 * myi + r) * 16 + 8 * myj + gg, tO = (4 * (sU >> 1) + r) * 16 + 8 * (sU & 1) + gg;
                    so = OFF_EXPP + (tS * 4 + cn) * BLK;
                    dso = OFF_OPP + (tO * 4 + cn) * BLK;
                    go = OFF_GP + tO * 128 + cn * 4;
                    po = pfb + (PF_P + id) * 8;
                }
                if (n1 >= (u32)NTASK) po = pfb + (PF_O + (n >> 2)) * 8;
            };
            u32 so, dso, go, po;
            desc(jw, so, dso, go, po);
            u32 info = poll(po);
            STAMP(1);
            while (!cx.dead) {
                u32 n2, so2, dso2, go2, po2;
                v2u fl;
                if (info & (1u << 14)) {
                    const u32 tv = (u32)__builtin_amdgcn_raw_ptr_buffer_atomic_add_i32(1, rsMe, l0, tk, 16);
                    v4u r[8];
#pragma unroll
                    for (int f = 0; f < 8; ++f) r[f] = ld128(rsSrc, vo16 + f * 1024, so);
                    VM_WAIT(0);
                    publish();
                    n2 = rfl(tv) + NP1;
                    desc(n2, so2, dso2, go2, po2);
                    fl = __builtin_amdgcn_raw_buffer_load_b64(rsMe, 0, po2, AUX_SYS);
#pragma unroll
                    for (int f = 0; f < 8; ++f) st128(r[f], rsMe, vo16 + f * 1024, dso);
                    __builtin_amdgcn_raw_buffer_store_b32(0u, rsMe, OOB, 0, AUX_SYS);
                } else {
                    const u32 cnt = (info >> 7) & 127u;
                    v4u pk[6];
                    const u32 tv = (u32)__builtin_amdgcn_raw_ptr_buffer_atomic_add_i32(1, rsMe, l0, tk, 16);
                    SCHED();
                    const v4u ex = ld128(rsSrc, (u32)lane < cnt ? 6144u + vo16 : OOB, so);
#pragma unroll
                    for (int i = 0; i < 6; ++i) pk[i] = ld128(rsSrc, vo16 + i * 1024, so);
                    SCHED();
                    VM_WAIT(8);
                    publish();
                    SCHED();
                    n2 = rfl(tv) + NP1;
                    desc(n2, so2, dso2, go2, po2);
                    SCHED();
                    VM_WAIT(1);
                    fl = __builtin_amdgcn_raw_buffer_load_b64(rsMe, 0, po2, AUX_SYS);
                    SCHED();
                    v4u o[8];
                    decode_block(pk, info, o);
#pragma unroll
                    for (int f = 0; f < 8; ++f) st128(o[f], rsMe, vo16 + f * 1024, dso);
                    SCHED();
                    __builtin_amdgcn_raw_buffer_store_b64(v2u{ex.x, ex.y}, rsMe, (u32)lane < cnt ? (ex.z & (BLK - 8)) : OOB, dso, AUX_SYS);
                }
                pend = go;
                have = rfl(fl.y) >= cx.call;
                if (n2 >= (u32)NTASK) { nz = rfl(fl.x); break; }
                info = have ? rfl(fl.x) : poll(po2);
                so = so2; dso = dso2; go = go2;
            }
            STAMP(2);
        }
        if (jw < (u32)NP2) {
            const __amdgpu_buffer_rsrc_t rsOut = mkrs(p.out, (u32)MM * NN);
            const u32 ovo = (u32)(lane & 15) * NN + (u32)(lane >> 4) * 16u;
            if (!have) nz = poll(pfb + (PF_O + (n >> 2)) * 8);
            LAS unsigned char* scr = (LAS unsigned char*)smem + wave * 1024;
            STAMP(3);
            for (;;) {
                const u32 tv = (u32)__builtin_amdgcn_raw_ptr_buffer_atomic_add_i32(1, rsMe, l0, tk + 1024, 16);
                SCHED();
                const u32 so = OFF_EXPO + n * QSLOT;
                const v4u ex = ld128(rsSrc, (u32)lane * 4u < 4u + ((nz >> (8 * (n & 3))) & 255u) ? 3584u + vo16 : OOB, so);
                v4u pk[3];
#pragma unroll
                for (int i = 0; i < 3; ++i) pk[i] = ld128(rsSrc, vo16 + i * 1024, so);
                const v2u pk3 = __builtin_amdgcn_raw_buffer_load_b64(rsSrc, 3072u + (u32)lane * 8u, so, AUX_SYS);
                SCHED();
                if (pend != OOB) { VM_WAIT(6); publish(); }
                SCHED();
                const u32 m2 = rfl(tv) + NP2, n2 = TB.outb[m2 < (u32)NOUT ? m2 : 0u];
                const u32 po2 = pfb + (PF_O + (n2 >> 2)) * 8;
                const v2u fl = __builtin_amdgcn_raw_buffer_load_b64(rsMe, 0, po2, AUX_SYS);
                SCHED();
                const u32 tl = n >> 2, cn = n & 3;
                const u32 rb = row_block(sG, sU >> 1, 16 + 2 * (tl >> 4) + (cn >> 1)), cb = col_block(sG, sU & 1, 2 * (tl & 15) + (cn & 1));
                v4u r[4];
                decode_q(pk, pk3, ex, scr, lane, r);
#pragma unroll
                for (int i = 0; i < 4; ++i) {
                    if (cx.dead) r[i] = v4u{0x80808080u, 0x80808080u, 0x80808080u, 0x80808080u};
                    st128(r[i], rsOut, ovo + i * 16 * NN, 64 * rb * NN + 64 * cb);
                }
                if (m2 >= (u32)NOUT) break;
                nz = rfl(fl.y) >= cx.call ? rfl(fl.x) : poll(po2);
                n = n2;
            }
        }
        STAMP(4);
    }
}

}

extern "C" void kernel_launch(int rank, int nranks, ncclComm_t comm, hipStream_t stream, void* const* d_in,
                              const int* in_sizes, int n_in, void* d_out, int out_size, const int* in_dtypes,
                              int out_dtype, void* const* d_out_all, void* const* d_ws_all, size_t ws_bytes) {
    if (nranks != NR || (unsigned)rank >= (unsigned)NR || n_in < 2 || in_sizes[0] != MM * KS || in_sizes[1] != KS * NN ||
        out_size != MM * NN || ws_bytes < WS_USED) {
        fprintf(stderr, "kernel_launch: unexpected problem shape\n");
        return;
    }
    Params p;
    for (int r = 0; r < NR; ++r) p.ws[r] = (unsigned char*)d_ws_all[r];
    p.x = (const float*)d_in[0];
    p.w = (const float*)d_in[1];
    p.out = (signed char*)d_out;
    p.rank = rank;
    p.pad = 0;
    fused_gemm_ar_q8<<<dim3(NWG), dim3(64 * NWAVE), 0, stream>>>(p);
    hipError_t e = hipGetLastError();
    if (e != hipSuccess) fprintf(stderr, "launch failed: %s\n", hipGetErrorString(e));
}
```

```cpp
#include <hip/hip_runtime.h>
#include <rccl/rccl.h>
#include <cstdio>
#include <cstdint>

namespace {

#define GAS __attribute__((address_space(1)))
#define LAS __attribute__((address_space(3)))
typedef unsigned u32;
typedef unsigned long long u64;
typedef unsigned v4u __attribute__((ext_vector_type(4)));
typedef unsigned v2u __attribute__((ext_vector_type(2)));
typedef float f32x4 __attribute__((ext_vector_type(4)));
typedef float f32x2 __attribute__((ext_vector_type(2)));
typedef short bf16x8 __attribute__((ext_vector_type(8)));
typedef __bf16 bf16x2 __attribute__((ext_vector_type(2)));
typedef unsigned short u16x2 __attribute__((ext_vector_type(2)));

constexpr int NR = 8;
constexpr int MM = 4096, NN = 4096, KS = 512;
constexpr int NGEMM = 128;
constexpr int NSONLY = 64;
constexpr int MOV0 = NGEMM + NSONLY;
constexpr int NMOV = 56;
constexpr int NWG = 256;
constexpr int PW1 = 3, PW2 = 7;
constexpr int NYT = 19;
constexpr int NYB = 4 * NYT;
constexpr int NTASK = 320 + NYB;
constexpr int NOUT = 512 - NYB;
constexpr int NWAVE = 12;
constexpr int NSTEP = 65;

constexpr u32 BLK = 8192;
constexpr u32 OFF_CNT = 0;
constexpr u32 OFF_TMO = 14336;
constexpr u32 OFF_TK = 24576;
constexpr u32 OFF_ACC = 16384;
constexpr u32 OFF_AM = 655360;
constexpr u32 OFF_GP = 40960;
constexpr u32 OFF_GY = OFF_GP + 16384;
constexpr u32 OFF_PF = 65536;
constexpr u32 PF_SRC = 65536;
constexpr u32 PF_P = 512, PF_Y = 640, PF_O = 768;
constexpr u32 OFF_EXP = 1u << 20;
constexpr u32 OFF_EXPP = OFF_EXP + 1024 * BLK;
constexpr u32 OFF_OP = OFF_EXPP + 512 * BLK;
constexpr u32 OFF_OPP = OFF_OP + 2048 * BLK;
constexpr u32 OFF_EXPY = OFF_OPP + 512 * BLK;
constexpr u32 OFF_OPY = OFF_EXPY + 128 * BLK;
constexpr u32 OFF_EXPO = OFF_OPY + 1024 * BLK;
constexpr u32 QSLOT = 4608;
constexpr u32 OFF_GF = OFF_EXPO + 512 * QSLOT;
constexpr u32 WS_USED = OFF_GF + 512 * 1024;
static_assert(OFF_GY + 4096 <= OFF_PF && OFF_PF + 8 * PF_SRC <= OFF_AM && OFF_AM + 65536 <= OFF_EXP, "control area");

constexpr u64 DEADLINE = 100000000ull;
constexpr int AUX_SYS = 17;
constexpr int RSRC3 = 0x00020000;
constexpr u32 OOB = 0x7FFFFFF0u;
constexpr u32 PACE1 = 670, PACE2 = 655;

constexpr int NSLOT = 4;
constexpr u32 SLOT_B = 4 * BLK;
constexpr u32 L_FLAG = NSLOT * SLOT_B;
constexpr u32 LDS_BYTES = L_FLAG + 1024;

constexpr unsigned NOCONV = 0xFFFFu, USE11 = 0x400u;
constexpr unsigned y_index(unsigned rt, unsigned g, unsigned q) { return rt == 0 ? 2 * g + q : rt == 1 && g < 4 ? 16 + 2 * g + q : 99; }
constexpr bool owner_runs_s(int rt, int g) { return g < 4 && rt != 0; }
constexpr int conv_cap(int b) {
    if (b >= MOV0 && b < MOV0 + NMOV) return 0;
    if (b >= MOV0 + NMOV) return (b & 7) < 4 ? 1 : 3;
    if (b >= NGEMM) return 5;
    if (!owner_runs_s(4 * (b & 1) + ((b >> 3) & 3), 2 * ((b & 7) >> 1) + ((b >> 5) & 1))) return 9;
    return (b & 7) < 2 ? 1 : 3;
}
struct Tables {
    unsigned pull[4][NTASK];
    unsigned conv[NWG * NWAVE];
    int nconv;
    unsigned outb[512];
    unsigned char step[128][68];
};
constexpr Tables make_tables() {
    Tables t{};
    int posX[4][8][32] = {}, posW[4][8][32] = {}, posP[128] = {};
    for (int L = 0; L < 4; ++L) {
        int n = 0;
        auto putX = [&](int c, int so) {
            for (int l = 16 * so; l < 16 * so + 16; ++l) {
                posX[L][c][l] = n;
                t.pull[L][n++] = (unsigned)((c << 5) | l);
            }
        };
        auto putW = [&](int c, int g) {
            for (int k = 0; k < 4; ++k) {
                const int l = (k >> 1) * 16 + 2 * g + (k & 1);
                posW[L][c][l] = n;
                t.pull[L][n++] = (unsigned)(256 | (c << 5) | l);
            }
        };
        auto putXO = [&](int c, bool yrows, int rot) {
            for (int i = 0; i < (yrows ? 2 : 14); ++i) {
                const int l = 16 + (yrows ? i : 2 + (i + rot) % 14);
                posX[L][c][l] = n;
                t.pull[L][n++] = (unsigned)((c << 5) | l);
            }
        };
        if (L < 3) {
            int left = 40, ny = 0;
            constexpr int yat[4] = {10, 6, 3, 1};
            for (int key = 0; key <= 40; ++key)
                for (int c = 4; c >= 0; --c) {
                    const int k = key - 3 * c;
                    if (k < 0 || k % 4 != 0 || k / 4 > 7) continue;
                    if (k == 0) putX(c, 0);
                    putW(c, k / 4);
                    --left;
                    if (ny < 4 && left == yat[ny]) putXO(ny++, true, 0);
                }
            putXO(4, true, 0);
            for (int c = 0; c < 5; ++c) putXO(c, false, L == 0 ? 0 : L == 1 ? 4 : 10);
        } else {
            for (int key = 0; key < 16; ++key)
                for (int c = 2; c >= 0; --c) {
                    const int g = key - 4 * c;
                    if (g < 0 || g > 7) continue;
                    if (g == 0) putX(5 + c, 0);
                    putW(5 + c, g);
                }
            for (int c = 5; c < 8; ++c) putX(c, 1);
            for (int g = 0; g < 8; ++g)
                for (int r = 0; r < 4; ++r)
                    for (int cn = 0; cn < 4; ++cn) {
                        const int id = (r << 5) | (g << 2) | cn;
                        posP[id] = n;
                        t.pull[L][n++] = (unsigned)(512 | id);
                    }
        }
        for (int i = 0; i < NYB; ++i) t.pull[L][n++] = (unsigned)(768 | i);
    }
    {
        int n = 0;
        for (int g = 0; g < 8; ++g)
            for (int rt = 0; rt < 8; ++rt)
                for (int q = 0; q < 2; ++q)
                    if (y_index(rt, g, q) >= NYT)
                        for (int cn = 0; cn < 4; ++cn) t.outb[n++] = (unsigned)((rt * 16 + q * 8 + g) * 4 + cn);
    }
    int made[1024] = {};
    {
        unsigned ord[1024] = {};
        int n = 0;
        for (int pos = 0; pos < NTASK; ++pos)
            for (int L = 0; L < 4; L += 3) {
                const int code = t.pull[L][pos];
                if (code & 512) continue;
                const int kind = code >> 8, c = (code >> 5) & 7, l = code & 31;
                for (int ij = 0; ij < 2; ++ij) ord[n++] = (unsigned)(kind * 512 + c * 64 + ij * 32 + l) | (pos < 48 ? 0u : USE11);
            }
        n = 0;
        for (int i = 0; i < NWG * NWAVE; ++i) t.conv[i] = NOCONV;
        for (int w = 0; w < NWAVE; ++w)
            for (int b = 0; b < NWG; ++b)
                if (w < conv_cap(b) && n < 1024) {
                    const int d = w == 0 ? (n < 96 ? 0 : 4) : 4 + 5 * w;
                    t.conv[b * NWAVE + w] = ord[n] | (unsigned)d << 16;
                    made[ord[n] & 1023] = 23 * d + 400;
                    ++n;
                }
        t.nconv = n;
    }
    auto arrive = [](int pos) { return 310 + pos * 975 / 100 + 280; };
    for (int li = 0; li < 128; ++li) {
        const int rt = (li >> 3) & 7, g = li & 7;
        const bool hasS = li >= 64 || owner_runs_s(rt, g), hasO = li < 64;
        int av[NSTEP] = {};
        bool used[NSTEP] = {};
        for (int so = 0; so < 2; ++so)
            for (int kc = 0; kc < 32; ++kc) {
                int k = 0;
                if (kc < 5) {
                    const int a = made[kc * 64 + 32 + so * 16 + 2 * rt + 1], b = made[512 + kc * 64 + 32 + 16 + 2 * g + 1];
                    k = a > b ? a : b;
                } else {
                    const int L = kc < 20 ? kc / 5 - 1 : 3;
                    const int c = kc < 20 ? kc % 5 : 5 + (kc - 20) % 3;
                    const int a = posX[L][c][so * 16 + 2 * rt + 1], b = posW[L][c][16 + 2 * g + 1];
                    k = arrive(a > b ? a : b);
                }
                av[so * 32 + kc] = k;
                used[so * 32 + kc] = so ? !hasO : !hasS;
            }
        av[64] = arrive(posP[((rt & 3) << 5) | (g << 2) | 3]) + (rt == 0 && g >= 3 ? 1000 : 200);
        used[64] = !hasO;
        const int ns = (hasS ? 32 : 0) + (hasO ? 33 : 0);
        int code[NSTEP] = {};
        for (int n = 0; n < ns; ++n) {
            int bs = -1, bo = -1;
            for (int i = 0; i < 32; ++i) if (!used[i] && (bs < 0 || av[i] < av[bs])) bs = i;
            for (int i = 32; i < 65; ++i) if (!used[i] && (bo < 0 || av[i] < av[bo])) bo = i;
            code[n] = bs >= 0 ? bs : bo;
            used[code[n]] = true;
        }
        for (int a = 1; a + 1 < ns; ++a)
            if ((code[a] >> 5) == 2 && (code[a - 1] >> 5) != 1) { const int cc = code[a]; code[a] = code[a + 1]; code[a + 1] = cc; }
        for (int a = 0; a < 67; ++a) t.step[li][a] = (unsigned char)(a < ns ? code[a] : 3 << 5);
        t.step[li][67] = (unsigned char)ns;
    }
    return t;
}
constexpr bool tables_ok() {
    const Tables t = make_tables();
    if (t.nconv != 1024) return false;
    for (int w = 0; w < 64; ++w) {
        int lastS = -1, sP = -1;
        for (int s = 0; s < NSTEP; ++s) {
            if ((t.step[w][s] >> 5) == 0) lastS = s;
            if ((t.step[w][s] >> 5) == 2) sP = s;
        }
        if (lastS > 62 || sP <= lastS || sP < 1 || (t.step[w][sP - 1] >> 5) != 1) return false;
    }
    return true;
}
static_assert(tables_ok(), "every block is converted; the partial step follows an O step and the last S step");
__constant__ const Tables TB = make_tables();

struct Params {
    const float* x;
    const float* w;
    unsigned char* me;
    int rank;
    int pad;
    signed char* out;
    unsigned char* ws[NR];
};

__device__ __forceinline__ u64 rt_now() { return __builtin_amdgcn_s_memrealtime(); }
__device__ __forceinline__ u32 rfl(u32 v) { return __builtin_amdgcn_readfirstlane(v); }
__device__ __forceinline__ u32 rlane(u32 v, u32 l) { return __builtin_amdgcn_readlane(v, l); }
__device__ __forceinline__ __amdgpu_buffer_rsrc_t mkrs(const void* p, u32 bytes) {
    return __builtin_amdgcn_make_buffer_rsrc((void*)p, 0, (int)bytes, RSRC3);
}
__device__ __forceinline__ v4u ld128(__amdgpu_buffer_rsrc_t rs, u32 voff, u32 soff) {
    return __builtin_amdgcn_raw_buffer_load_b128(rs, voff, soff, AUX_SYS);
}
__device__ __forceinline__ void st128(v4u v, __amdgpu_buffer_rsrc_t rs, u32 voff, u32 soff) {
    __builtin_amdgcn_raw_buffer_store_b128(v, rs, voff + soff, 0, AUX_SYS);
}
#define VM_WAIT(n) asm volatile("s_waitcnt vmcnt(" #n ")" ::: "memory")
#define LGKM0() asm volatile("s_waitcnt lgkmcnt(0)" ::: "memory")
#define SCHED() __builtin_amdgcn_sched_barrier(0)
#define LANES() asm volatile("" ::: "memory")
__device__ __forceinline__ u32 cvt2(float a, float b) {
    return __builtin_bit_cast(u32, __builtin_convertvector(f32x2{a, b}, bf16x2));
}
__device__ __forceinline__ u32 perm(u32 s0, u32 s1, u32 sel) { return __builtin_amdgcn_perm(s0, s1, sel); }
__device__ __forceinline__ u32 wave_max_u32(u32 v) {
    u32 t;
    t = __builtin_amdgcn_update_dpp(0u, v, 0x111, 0xf, 0xf, true); v = v > t ? v : t;
    t = __builtin_amdgcn_update_dpp(0u, v, 0x112, 0xf, 0xf, true); v = v > t ? v : t;
    t = __builtin_amdgcn_update_dpp(0u, v, 0x114, 0xf, 0xf, true); v = v > t ? v : t;
    t = __builtin_amdgcn_update_dpp(0u, v, 0x118, 0xf, 0xf, true); v = v > t ? v : t;
    t = __builtin_amdgcn_update_dpp(0u, v, 0x142, 0xa, 0xf, false); v = v > t ? v : t;
    t = __builtin_amdgcn_update_dpp(0u, v, 0x143, 0xc, 0xf, false); v = v > t ? v : t;
    return rlane(v, 63);
}
__device__ __forceinline__ void lds_st(u32 addr, u32 v) { asm volatile("ds_write_b32 %0, %1" ::"v"(addr), "v"(v) : "memory"); }
__device__ __forceinline__ v4u lds_ld4(u32 addr) {
    v4u v;
    asm volatile("ds_read_b128 %0, %1\n\ts_waitcnt lgkmcnt(0)" : "=&v"(v) : "v"(addr) : "memory");
    return v;
}
__device__ __forceinline__ u32 lds_ld(u32 addr) {
    u32 v;
    asm volatile("ds_read_b32 %0, %1\n\ts_waitcnt lgkmcnt(0)" : "=&v"(v) : "v"(addr) : "memory");
    return v;
}
__device__ __forceinline__ void glds4(const unsigned char* base, u32 voff, u32 lds_dst) {
    u32 keep;
    asm volatile(
        "s_mov_b32 %0, m0\n\ts_mov_b32 m0, %3\n\ts_nop 0\n\t"
        "global_load_lds_dwordx4 %1, %2 sc0 sc1\n\t"
        "global_load_lds_dwordx4 %1, %2 offset:1024 sc0 sc1\n\t"
        "global_load_lds_dwordx4 %1, %2 offset:2048 sc0 sc1\n\t"
        "global_load_lds_dwordx4 %1, %2 offset:3072 sc0 sc1\n\t"
        "s_mov_b32 m0, %0"
        : "=&s"(keep) : "v"(voff), "s"(base), "s"(lds_dst) : "memory");
}

__device__ __forceinline__ u64 flag_peek(const unsigned char* p) {
    u64 v;
    asm volatile("s_load_dwordx2 %0, %1, 0x0 glc\n\ts_waitcnt lgkmcnt(0)" : "=&s"(v) : "s"(p) : "memory");
    return v;
}
__device__ __forceinline__ u64 flag_peek_drain(const unsigned char* p) {
    u64 v;
    asm volatile("s_load_dwordx2 %0, %1, 0x0 glc\n\ts_waitcnt vmcnt(0) lgkmcnt(0)" : "=&s"(v) : "s"(p) : "memory");
    return v;
}

struct Ctx {
    unsigned char* me;
    u64 t0;
    u32 call;
    int lane;
    bool dead;
};
__device__ __forceinline__ bool give_up(Ctx& c) {
    GAS u32* tmo = (GAS u32*)(c.me + OFF_TMO);
    const u32 t = rfl(__hip_atomic_load(tmo, __ATOMIC_RELAXED, __HIP_MEMORY_SCOPE_AGENT));
    if (t == c.call || rt_now() - c.t0 > DEADLINE) {
        __hip_atomic_store(tmo, c.call, __ATOMIC_RELAXED, __HIP_MEMORY_SCOPE_AGENT);
        c.dead = true;
        return true;
    }
    return false;
}

#define BAR() asm volatile("s_barrier" ::: "memory")

__device__ __forceinline__ u32 bfi(u32 m, u32 a, u32 b) { return (a & m) | (b & ~m); }
__device__ __forceinline__ u32 mbcnt(u64 m) { return __builtin_amdgcn_mbcnt_hi((u32)(m >> 32), __builtin_amdgcn_mbcnt_lo((u32)m, 0u)); }

__device__ __forceinline__ u32 encode_block(const u32 (&D)[32], u32 hint, __amdgpu_buffer_rsrc_t rs, u32 soff, int lane) {
    const u32 vo = (u32)lane * 16u;
    const u32 e7 = wave_max_u32(hint) >> 24;
    const u32 base = e7 > 7u ? e7 - 7u : 0u;
    const u32 base4 = base * 0x01010101u;
#pragma unroll
    for (int k = 0; k < 4; ++k) {
        v4u lo;
#pragma unroll
        for (int e = 0; e < 4; ++e) lo[e] = perm(D[8 * k + 2 * e + 1], D[8 * k + 2 * e], 0x06040200u);
        st128(lo, rs, vo + (k >> 1) * 3072 + (k & 1) * 1024, soff);
    }
    u32 U[16], bad[4] = {0u, 0u, 0u, 0u};
#pragma unroll
    for (int i = 0; i < 16; ++i) {
        U[i] = perm(D[2 * i + 1], D[2 * i], 0x07050301u) - base4;
        bad[i >> 2] |= U[i] & 0x78787878u;
    }
#pragma unroll
    for (int u = 0; u < 2; ++u) {
        v4u nn;
#pragma unroll
        for (int e = 0; e < 4; ++e) nn[e] = bfi(0x87878787u, U[8 * u + 2 * e], __builtin_amdgcn_alignbit(U[8 * u + 2 * e + 1], U[8 * u + 2 * e + 1], 28));
        st128(nn, rs, vo + u * 3072 + 2048, soff);
    }
    if (__builtin_amdgcn_ballot_w64((bad[0] | bad[1] | bad[2] | bad[3]) != 0) == 0) return base;
    u32 cnt = 0;
#pragma unroll
    for (int q = 0; q < 4; ++q)
        if (__builtin_amdgcn_ballot_w64(bad[q] != 0) != 0) {
#pragma unroll
            for (int i = 4 * q; i < 4 * q + 4; ++i) {
                const bool b = (U[i] & 0x78787878u) != 0;
                const u64 m = __builtin_amdgcn_ballot_w64(b);
                const u32 idx = cnt + mbcnt(m);
                st128(v4u{D[2 * i], D[2 * i + 1], vo + (u32)((i >> 1) * 1024 + (i & 1) * 8), 0u}, rs,
                      b && idx < 64u ? 6144u + idx * 16u : OOB, soff);
                cnt += (u32)__builtin_popcountll(m);
            }
        }
    if (cnt <= 64u) return base | (cnt << 7);
    VM_WAIT(0);
#pragma unroll
    for (int f = 0; f < 8; ++f) st128(v4u{D[4 * f], D[4 * f + 1], D[4 * f + 2], D[4 * f + 3]}, rs, vo + f * 1024, soff);
    return 1u << 14;
}
__device__ __forceinline__ void decode_block(const v4u (&pk)[6], u32 info, v4u (&out)[8]) {
    const u32 base4 = (info & 127u) * 0x01010101u;
#pragma unroll
    for (int f = 0; f < 8; ++f) {
        const u32 nn = pk[3 * (f >> 2) + 2][f & 3];
        const u32 ha = (nn & 0x87878787u) + base4, hb = (__builtin_amdgcn_alignbit(nn, nn, 4) & 0x87878787u) + base4;
        const v4u lp = pk[3 * (f >> 2) + ((f >> 1) & 1)];
        const u32 la = lp[2 * (f & 1)], lb = lp[2 * (f & 1) + 1];
        out[f] = v4u{perm(ha, la, 0x05010400u), perm(ha, la, 0x07030602u), perm(hb, lb, 0x05010400u), perm(hb, lb, 0x07030602u)};
    }
}

constexpr u32 P_CODE = 4096, P_SIGN = 5120, P_LIST = 5632, F11 = 1u << 31, FRAW = 1u << 14;
__device__ __forceinline__ u32 pk_sub(u32 a, u32 b) { return __builtin_bit_cast(u32, __builtin_bit_cast(u16x2, a) - __builtin_bit_cast(u16x2, b)); }
__device__ __forceinline__ u32 pk_add(u32 a, u32 b) { return __builtin_bit_cast(u32, __builtin_bit_cast(u16x2, a) + __builtin_bit_cast(u16x2, b)); }
__device__ __forceinline__ u32 encode11(const u32 (&D)[32], u32 hint, __amdgpu_buffer_rsrc_t rs, u32 soff, u32 scr0, int lane) {
    const u32 vo = (u32)lane * 16u;
    const u32 top = (wave_max_u32(hint) >> 16) + 1u;
    const u32 B = top > 3072u ? top - 3072u : 0u;
    const u32 B2 = B * 0x00010001u;
    v4u code = {0u, 0u, 0u, 0u};
    v2u sign = {0u, 0u}, out = {0u, 0u};
    u32 far = 0;
    LANES();
#pragma unroll
    for (int k = 0; k < 4; ++k) {
        v4u t4, lo;
#pragma unroll
        for (int e = 0; e < 4; ++e) {
            const int i = 4 * k + e;
            const u32 e0 = pk_sub(D[2 * i], B2), e1 = pk_sub(D[2 * i + 1], B2);
            lo[e] = perm(e1, e0, 0x06040200u);
            const u32 t = perm(e1, e0, 0x07050301u) ^ 0x08080808u;
            t4[e] = t;
            far |= t;
            sign[i >> 3] = bfi(0x80808080u >> (i & 7), t >> (i & 7), sign[i >> 3]);
            code[k] |= (t & 0x03030303u) << (2 * e);
            const u32 x = (t | (t >> 1)) >> 2;
            out[i >> 3] = bfi(0x01010101u << (i & 7), x << (i & 7), out[i >> 3]);
        }
        st128(lo, rs, vo + k * 1024, soff);
        *(LAS v4u*)(size_t)(scr0 + k * 1024 + vo) = t4;
        SCHED();
    }
    st128(code, rs, P_CODE + vo, soff);
    __builtin_amdgcn_raw_buffer_store_b64(sign, rs, P_SIGN + (u32)lane * 8u, soff, AUX_SYS);
    u64 m = (u64)out.y << 32 | out.x;
    u32 cnt = __builtin_amdgcn_ballot_w64((far & 0x70707070u) != 0) != 0 ? 999u : 0u;
    const u32 own = scr0 + 4864u + (u32)lane * 2u;
    while (cnt <= 256u) {
        const u64 bl = __builtin_amdgcn_ballot_w64(m != 0);
        if (bl == 0) break;
        const u32 at = scr0 + 4096u + ((cnt + mbcnt(bl)) << 1);
        *(LAS unsigned short*)(size_t)(m != 0 ? at : own) = (unsigned short)(((u32)lane << 6) | (u32)__builtin_ctzll(m | 1ull << 63));
        m &= m - 1;
        cnt += (u32)__builtin_popcountll(bl);
    }
    if (cnt > 256u) { VM_WAIT(0); return 0u; }
    LANES();
    const v2u l2 = *(const LAS v2u*)(size_t)(scr0 + 4096u + (u32)lane * 8u);
    u32 ent[4];
#pragma unroll
    for (int e = 0; e < 4; ++e) {
        const u32 x = l2[e >> 1] >> (16 * (e & 1));
        const u32 place = ((x & 32u) << 6) | ((x & 4u) << 8) | ((x >> 2) & 0x3F0u) | ((x & 3u) << 2) | ((x >> 3) & 3u);
        ent[e] = place << 4 | (*(const LAS unsigned char*)(size_t)(scr0 + place) & 15u);
    }
    __builtin_amdgcn_raw_buffer_store_b64(v2u{ent[0] | ent[1] << 16, ent[2] | ent[3] << 16}, rs, (u32)lane * 4u < cnt ? P_LIST + (u32)lane * 8u : OOB, soff, AUX_SYS);
    return B | (cnt << 16) | F11;
}
__device__ __forceinline__ void decode11(const v4u (&lo)[4], v4u code, v2u sign, v2u ex, u32 info, u32 scr0, int lane, v4u (&out)[8]) {
    const u32 B2 = ((info + 0x800u) & 0xFFFFu) * 0x00010001u, cnt = (info >> 16) & 511u;
    const u32 vo = (u32)lane * 16u;
    LANES();
#pragma unroll
    for (int k = 0; k < 4; ++k)
        *(LAS v4u*)(size_t)(scr0 + k * 1024 + vo) = v4u{code[k], code[k] >> 2, code[k] >> 4, code[k] >> 6} & 0x03030303u;
    LANES();
#pragma unroll
    for (int e = 0; e < 4; ++e) {
        const u32 x = ex[e >> 1] >> (16 * (e & 1));
        const u32 at = scr0 + ((u32)lane * 4u + e < cnt ? (x >> 4) & 0xFFFu : 4096u + (u32)lane);
        *(LAS unsigned char*)(size_t)at = (unsigned char)__builtin_amdgcn_sbfe(x, 0u, 4u);
    }
    LANES();
#pragma unroll
    for (int k = 0; k < 4; ++k) {
        const v4u d = *(const LAS v4u*)(size_t)(scr0 + k * 1024 + vo);
#pragma unroll
        for (int e = 0; e < 4; ++e) {
            const int i = 4 * k + e;
            const u32 h = ((sign[i >> 3] << (i & 7)) & 0x80808080u) ^ d[e];
            out[i >> 1][2 * (i & 1)] = pk_add(perm(h, lo[k][e], 0x05010400u), B2);
            out[i >> 1][2 * (i & 1) + 1] = pk_add(perm(h, lo[k][e], 0x07030602u), B2);
        }
    }
}

__device__ __forceinline__ u32 encode_q(const v4u (&q)[4], __amdgpu_buffer_rsrc_t rs, u32 soff, u32 scr0, int lane) {
    u32 P[14], M[2];
#pragma unroll
    for (int g = 0; g < 2; ++g) {
        const u32 top = q[2 * g + 1][3];
        u32 m = 0;
#pragma unroll
        for (int i = 0; i < 8; ++i) {
            const u32 x = q[2 * g + (i >> 2)][i & 3];
            if (i < 7) P[7 * g + i] = bfi(0x80808080u, top << (7 - i), x);
            m |= ((x ^ (x << 1)) & 0x80808080u) >> i;
        }
        M[g] = m;
    }
    const u32 vo = (u32)lane * 16u;
#pragma unroll
    for (int k = 0; k < 3; ++k) st128(v4u{P[4 * k], P[4 * k + 1], P[4 * k + 2], P[4 * k + 3]}, rs, vo + k * 1024, soff);
    __builtin_amdgcn_raw_buffer_store_b64(v2u{P[12], P[13]}, rs, 3072u + (u32)lane * 8u + soff, 0, AUX_SYS);
    const u64 b0 = __builtin_amdgcn_ballot_w64(M[0] != 0), b1 = __builtin_amdgcn_ballot_w64(M[1] != 0);
    const u32 n0 = (u32)__builtin_popcountll(b0);
    const u32 nz = n0 + (u32)__builtin_popcountll(b1), own = scr0 + 768u + (u32)lane * 4u;
    {
        const u32 a0 = scr0 + 16u + mbcnt(b0) * 4u, a1 = scr0 + 16u + (n0 + mbcnt(b1)) * 4u;
        LANES();
        *(LAS u32*)(size_t)(M[0] != 0 ? a0 : own) = M[0];
        *(LAS u32*)(size_t)(M[1] != 0 ? a1 : own) = M[1];
        LANES();
        v4u l4 = *(const LAS v4u*)(size_t)(scr0 + vo);
        if (lane == 0) l4 = v4u{(u32)b0, (u32)(b0 >> 32), (u32)b1, (u32)(b1 >> 32)};
        st128(l4, rs, (u32)lane * 4u < 4u + nz ? 3584u + vo : OOB, soff);
    }
    return nz;
}
__device__ __forceinline__ void decode_q(const v4u (&pk)[3], v2u pk3, v4u ex, LAS unsigned char* scr, int lane, v4u (&q)[4]) {
    LANES();
    *(LAS v4u*)(scr + lane * 16) = ex;
    LANES();
    const u64 b0 = (u64)rlane(ex.x, 0) | (u64)rlane(ex.y, 0) << 32, b1 = (u64)rlane(ex.z, 0) | (u64)rlane(ex.w, 0) << 32;
    const u32 n0 = (u32)__builtin_popcountll(b0);
    u32 M[2];
    M[0] = *(const LAS u32*)(scr + 16 + mbcnt(b0) * 4);
    M[1] = *(const LAS u32*)(scr + 16 + (n0 + mbcnt(b1)) * 4);
    const u32 P[14] = {pk[0][0], pk[0][1], pk[0][2], pk[0][3], pk[1][0], pk[1][1], pk[1][2], pk[1][3],
                       pk[2][0], pk[2][1], pk[2][2], pk[2][3], pk3.x, pk3.y};
    u32 top[2];
#pragma unroll
    for (int g = 0; g < 2; ++g) {
        u32 t = 0;
#pragma unroll
        for (int i = 0; i < 7; ++i) t = bfi(0x01010101u << i, P[7 * g + i] >> (7 - i), t);
        top[g] = t;
    }
    M[0] = (b0 >> lane) & 1ull ? M[0] : 0u;
    M[1] = (b1 >> lane) & 1ull ? M[1] : 0u;
#pragma unroll
    for (int g = 0; g < 2; ++g)
#pragma unroll
        for (int i = 0; i < 8; ++i) {
            const u32 x = i < 7 ? P[7 * g + i] : top[g];
            q[2 * g + (i >> 2)][i & 3] = bfi(0x80808080u, (x << 1) ^ (M[g] << i), x);
        }
}

__device__ __forceinline__ u32 ffbl(u32 v) { u32 r; asm("v_ffbl_b32 %0, %1" : "=v"(r) : "v"(v)); return r; }
__device__ __forceinline__ u32 wave_scan(u32 v) {
    v += (u32)__builtin_amdgcn_update_dpp(0u, v, 0x111, 0xf, 0xf, true);
    v += (u32)__builtin_amdgcn_update_dpp(0u, v, 0x112, 0xf, 0xf, true);
    v += (u32)__builtin_amdgcn_update_dpp(0u, v, 0x114, 0xf, 0xf, true);
    v += (u32)__builtin_amdgcn_update_dpp(0u, v, 0x118, 0xf, 0xf, true);
    v += (u32)__builtin_amdgcn_update_dpp(0u, v, 0x142, 0xa, 0xf, false);
    v += (u32)__builtin_amdgcn_update_dpp(0u, v, 0x143, 0xc, 0xf, false);
    return v;
}
constexpr u32 QU_SIGN = 2048, QU_BITS = 2560;
__device__ __forceinline__ u32 encode_qu(const v4u (&q)[4], __amdgpu_buffer_rsrc_t rs, u32 soff, u32 scr0, int lane) {
    u32 S[5] = {0u, 0u, 0u, 0u, 0u}, len = 0, full = 0;
    v2u sign = {0u, 0u};
    const u32 vo = (u32)lane * 16u;
    LANES();
    *(LAS v4u*)(size_t)(scr0 + vo) = v4u{0u, 0u, 0u, 0u};
#pragma unroll
    for (int i = 15; i >= 0; --i) {
        const u32 x = q[i >> 2][i & 3], t = x & 0x80808080u;
        sign[i >> 3] |= t >> (i & 7);
        u32 z = (((x ^ (t - (t >> 3))) >> 4) & 0x07070707u) + 0x01010100u;
        z += z << 8;
        z += z << 16;
        const u32 top = z >> 24, sh = 31u - top;
#pragma unroll
        for (int k = 4; k >= 1; --k)
            if (k <= 15 - i) S[k] = __builtin_amdgcn_alignbit(S[k], S[k - 1], sh);
        S[0] = __builtin_amdgcn_alignbit(S[0], 0u, sh) | (1u << (z & 31u)) | (1u << ((z >> 8) & 31u)) | (1u << ((z >> 16) & 31u)) | (1u << top);
        len += top;
        full |= top + 1u;
        if (!(i & 1)) asm volatile("" : "+v"(S[0]), "+v"(S[1]), "+v"(S[2]), "+v"(S[3]), "+v"(S[4]), "+v"(sign), "+v"(len), "+v"(full));
    }
    len += 16u;
    const u32 end = wave_scan(len), total = rlane(end, 63);
    if (total > 8192u || __builtin_amdgcn_ballot_w64(len > 160u || (full & 32u) != 0) != 0) return 0u;
    {
        const u32 off = end - len, sh = ~off, at = scr0 + ((off >> 5) << 2);
        LANES();
#pragma unroll
        for (int k = 0; k < 6; ++k) {
            const u32 hi = k < 5 ? S[k] : 0u, lo = k > 0 ? S[k - 1] : 0u;
            const u32 w = __builtin_amdgcn_alignbit(hi >> 1, __builtin_amdgcn_alignbit(hi, lo, 1), sh);
            __hip_atomic_fetch_or((LAS u32*)(size_t)(at + 4 * k), w, __ATOMIC_RELAXED, __HIP_MEMORY_SCOPE_WAVEFRONT);
        }
        LANES();
    }
#pragma unroll
    for (int k = 0; k < 2; ++k) {
        v4u n;
#pragma unroll
        for (int e = 0; e < 4; ++e) n[e] = bfi(0x0F0F0F0Fu, q[2 * k][e], q[2 * k + 1][e] << 4);
        st128(n, rs, vo + k * 1024, soff);
    }
    __builtin_amdgcn_raw_buffer_store_b64(sign, rs, QU_SIGN + (u32)lane * 8u + soff, 0, AUX_SYS);
    const u32 pieces = (total + 127u) >> 7;
    st128(*(const LAS v4u*)(size_t)(scr0 + vo), rs, (u32)lane < pieces ? QU_BITS + vo : OOB, soff);
    return pieces;
}
__device__ __forceinline__ void decode_qu(v4u nib0, v4u nib1, v2u sign, v4u bits, u32 scr0, int lane, v4u (&q)[4]) {
    LANES();
    *(LAS v4u*)(size_t)(scr0 + (u32)lane * 16u) = bits;
    *(LAS u32*)(size_t)(scr0 + 1280u) = 0u;
    const u32 c0 = __builtin_popcount(bits.x), c1 = c0 + __builtin_popcount(bits.y), c2 = c1 + __builtin_popcount(bits.z),
              c3 = c2 + __builtin_popcount(bits.w);
    const u32 base = wave_scan(c3) - c3;
#pragma unroll
    for (int k = 0; k < 2; ++k) {
        const u32 j = (base >> 6) + 1u + k;
        u32 r = (j << 6) - base;
        const bool in = r <= c3;
        const bool up = r > c1;
        const u32 cm = up ? c2 : c0, wl = up ? bits.z : bits.x, wh = up ? bits.w : bits.y;
        const bool up2 = r > cm;
        u32 w = up2 ? wh : wl, pos = (u32)lane * 128u + (up ? 64u : 0u) + (up2 ? 32u : 0u) + 1u;
        r -= up2 ? cm : up ? c1 : 0u;
#pragma unroll
        for (int st = 16; st >= 1; st >>= 1) {
            const u32 c = __builtin_popcount(w & ((1u << st) - 1u));
            const bool go = r > c;
            r -= go ? c : 0u;
            w >>= go ? st : 0;
            pos += go ? st : 0;
        }
        *(LAS u32*)(size_t)(scr0 + 1280u + 4u * (in && j < 64u ? j : 64u + (u32)lane)) = pos;
    }
    LANES();
    const u32 start = *(const LAS u32*)(size_t)(scr0 + 1280u + (u32)lane * 4u) & 8191u;
    u32 W[6], S[5];
#pragma unroll
    for (int k = 0; k < 6; ++k) W[k] = *(const LAS u32*)(size_t)(scr0 + ((start >> 5) << 2) + 4 * k);
#pragma unroll
    for (int k = 0; k < 5; ++k) S[k] = __builtin_amdgcn_alignbit(W[k + 1], W[k], start);
#pragma unroll
    for (int i = 0; i < 16; ++i) {
        u32 w = S[0];
        const u32 a = ffbl(w);
        w &= w - 1;
        const u32 b = ffbl(w);
        w &= w - 1;
        const u32 c = ffbl(w);
        w &= w - 1;
        const u32 d = ffbl(w);
        const u32 z = a | b << 8 | c << 16 | d << 24;
        const u32 h = z - (z << 8) - 0x01010100u;
#pragma unroll
        for (int k = 0; k < 5; ++k)
            if (k < 15 - i) S[k] = __builtin_amdgcn_alignbit(k < 4 ? S[k + 1] : 0u, S[k], d + 1u);
        const u32 t = (sign[i >> 3] << (i & 7)) & 0x80808080u;
        const u32 n = (i < 8 ? nib0 : nib1)[i & 3] >> (i & 4);
        q[i >> 2][i & 3] = bfi(0x0F0F0F0Fu, n, ((h << 4) ^ (t - (t >> 3))) | t);
    }
}

__device__ __forceinline__ void st_flag64(unsigned char* p, u32 call, u32 info) {
    __hip_atomic_store((GAS u64*)p, ((u64)call << 32) | info, __ATOMIC_RELAXED, __HIP_MEMORY_SCOPE_SYSTEM);
}
__device__ __forceinline__ void st_flag32(unsigned char* p, u32 call) {
    __hip_atomic_store((GAS u32*)p, call, __ATOMIC_RELAXED, __HIP_MEMORY_SCOPE_SYSTEM);
}
__device__ __forceinline__ u32 gf_off(u32 row, u32 kc) { return OFF_GF + (row * 8 + (kc < 20 ? kc / 5 : 4 + (kc - 20) / 3)) * 1024 + kc * 4; }
__device__ __forceinline__ u32 row_block(u32 G, u32 i, u32 l) {
    const u32 so = l >> 4, q = (l >> 3) & 1, idx = l & 7;
    const u32 a = G ? i + 2 * q : 2 * i + q, hf = G ? 1 - so : so;
    return 16 * a + 8 * hf + idx;
}
__device__ __forceinline__ u32 col_block(u32 G, u32 j, u32 l) {
    const u32 q = l >> 4, idx = l & 15;
    return 16 * (G ? j + 2 * q : 2 * j + q) + idx;
}

__device__ __forceinline__ void convert_block(const Params& p, const float* px, const float* pw, Ctx& cx, u32 slot, bool use11, bool own_first, u32 scr0, u32 cnt_v, __amdgpu_buffer_rsrc_t rsMe) {
    const int lane = cx.lane;
    const u32 rank = p.rank, G = rank >> 2;
    const u32 kind = slot >> 9, c = (slot >> 6) & 7, ij = (slot >> 5) & 1, l = slot & 31;
    const u32 dG = c < 5 ? G : 1 - G;
    const u32 d0 = (dG * 4 + (kind ? ij : ij * 2)) & 7, d1 = (d0 + (kind ? 2 : 1)) & 7;
    unsigned char* const w0 = p.ws[d0];
    unsigned char* const w1 = p.ws[d1];
    u32 D[32];
    float mx = 0.f;
    const u32 r16 = lane & 15, g4 = lane >> 4;
    if (kind == 0) {
        const u32 rb = row_block(dG, ij, l);
        const GAS f32x4* src = (const GAS f32x4*)(px + (size_t)(64 * rb + r16) * KS + 64 * c + 4 * g4);
        f32x4 v[16];
#pragma unroll
        for (int mt = 0; mt < 4; ++mt)
#pragma unroll
            for (int i = 0; i < 4; ++i) v[mt * 4 + i] = __builtin_nontemporal_load(src + mt * 16 * (KS / 4) + i * 4);
        SCHED();
#pragma unroll
        for (int mt = 0; mt < 4; ++mt)
#pragma unroll
            for (int i = 0; i < 4; ++i) {
                const f32x4 a = v[mt * 4 + i];
                const int f = (i >> 1) * 4 + mt, o = 4 * f + 2 * (i & 1);
                D[o] = cvt2(a.x, a.y);
                D[o + 1] = cvt2(a.z, a.w);
                if (i == 0) mx = fmaxf(fmaxf(fmaxf(fmaxf(mx, fabsf(a.x)), fabsf(a.y)), fabsf(a.z)), fabsf(a.w));
            }
    } else {
        const u32 cb = col_block(dG, ij, l);
        const GAS f32x4* src = (const GAS f32x4*)(pw + (size_t)(64 * c + 4 * g4) * NN + 64 * cb + 4 * r16);
        f32x4 v[16];
#pragma unroll
        for (int j = 0; j < 16; ++j) v[j] = __builtin_nontemporal_load(src + ((j >> 2) * 16 + (j & 3)) * (NN / 4));
        SCHED();
#pragma unroll
        for (int q = 0; q < 2; ++q)
#pragma unroll
            for (int pp = 0; pp < 4; ++pp) {
                const f32x4 a = v[q * 8 + 2 * pp], b = v[q * 8 + 2 * pp + 1];
#pragma unroll
                for (int t = 0; t < 4; ++t) {
                    D[4 * (q * 4 + t) + pp] = cvt2(a[t], b[t]);
                    if (pp == 0) mx = fmaxf(fmaxf(mx, fabsf(a[t])), fabsf(b[t]));
                }
            }
    }
    cx.call = rfl(cnt_v) + 1u;
    const bool self = d0 == rank || d1 == rank;
    auto own_copy = [&]() {
        const u32 so = OFF_OP + ((kind * 32 + l) * 32 + c) * BLK;
#pragma unroll
        for (int f = 0; f < 8; ++f) st128(v4u{D[4 * f], D[4 * f + 1], D[4 * f + 2], D[4 * f + 3]}, rsMe, (u32)lane * 16u + f * 1024, so);
        VM_WAIT(0);
        if (lane < 4) st_flag32(cx.me + gf_off(kind * 32 + l, c) + (u32)lane * 256u, cx.call);
    };
    if (self && own_first) own_copy();
    u32 info = 0;
    if (use11) info = encode11(D, __builtin_bit_cast(u32, mx), rsMe, OFF_EXP + slot * BLK, scr0, lane);
    if (info == 0) info = encode_block(D, __builtin_bit_cast(u32, mx), rsMe, OFF_EXP + slot * BLK, lane);
    VM_WAIT(0);
    const u32 fo = OFF_PF + rank * PF_SRC + (kind * 256 + c * 32 + l) * 64;
    if (lane == 0) {
        if (d0 != rank) st_flag64(w0 + fo, cx.call, info);
        if (d1 != rank) st_flag64(w1 + fo, cx.call, info);
    }
    if (self && !own_first) own_copy();
}

__global__ void __launch_bounds__(768, 3) fused_gemm_ar_q8(Params p) {
    __shared__ __attribute__((aligned(16))) unsigned char smem[LDS_BYTES];
    const int tid = threadIdx.x, lane = tid & 63;
    const u32 wave = rfl(tid >> 6);
    const u32 bid = blockIdx.x;
    const bool is_gemm = bid < (u32)MOV0 || bid >= (u32)(MOV0 + NMOV);
    if (!is_gemm && wave >= (u32)PW2) return;
    const u32 cslot = TB.conv[bid * NWAVE + wave];
    const u32 rank = p.rank, G = rank >> 2, U = rank & 3, myi = U >> 1, myj = U & 1;
    const float* const px = p.x;
    const float* const pw = p.w;
    Ctx cx;
    cx.me = p.me;
    asm volatile("" :: "s"(cslot), "s"(rank), "s"(px), "s"(pw), "s"(cx.me));
    cx.t0 = rt_now();
    cx.lane = lane;
    cx.dead = false;
    GAS u32* cntp = (GAS u32*)(cx.me + OFF_CNT) + bid * NWAVE + wave;
    const u32 cnt_v = __hip_atomic_load(cntp, __ATOMIC_RELAXED, __HIP_MEMORY_SCOPE_AGENT);
    const __amdgpu_buffer_rsrc_t rsMe = mkrs(cx.me, WS_USED);
    const u32 vo16 = (u32)lane * 16u;
    if (cslot != NOCONV) {
        for (u32 s = cslot >> 16; s > 0; --s) __builtin_amdgcn_s_sleep(8);
        convert_block(p, px, pw, cx, cslot & 1023u, (cslot & USE11) != 0, wave != 0, (u32)(size_t)(LAS unsigned char*)smem + SLOT_B + wave * 5120u, cnt_v, rsMe);
    }
    cx.call = rfl(cnt_v) + 1u;
    if (lane == 0) *cntp = cx.call;

    if (is_gemm) {
        const u32 lds0 = (u32)(size_t)(LAS unsigned char*)smem;
        const u32 xc = bid & 7, k = bid >> 3;
        const bool owner = bid < (u32)NGEMM;
        const bool last8 = bid >= (u32)(MOV0 + NMOV);
        const u32 rt = last8 ? 0u : 4 * (xc & 1) + (k & 3);
        const u32 g = owner ? 2 * (xc >> 1) + ((k >> 2) & 1) : last8 ? xc >> 1 : 4 + (xc >> 1);
        const u32 ct = g + 8 * (owner ? k >> 3 : last8 ? xc & 1 : (k >> 2) & 1);
        const u32 tile = rt * 16 + ct;
        const u32 yix = y_index(rt, g, ct >> 3);
        const bool isY = owner && yix < (u32)NYT;
        if (wave < 8 && (wave < 4 ? !owner : owner && !owner_runs_s(rt, g))) return;
        const u32 li = (owner ? 0u : 64u) + rt * 8 + g;
        const u32 code0 = rfl(TB.step[li][0]), nstep = rfl(TB.step[li][67]);
        const u32 codeV = TB.step[li][1 + lane];

        if (wave >= 8) {
            const u32 j = wave - 8;
            auto blk_of = [&](u32 code, u32& boff, u32& foff) {
                const u32 ty = code >> 5, kc = code & 31;
                const u32 row = j < 2 ? (ty & 1) * 16 + 2 * rt + j : 32 + 2 * ct + (j - 2);
                boff = ty == 2 ? OFF_OPP + (tile * 4 + j) * BLK : OFF_OP + (row * 32 + kc) * BLK;
                foff = ty == 2 ? OFF_GP + tile * 128 + j * 4 : gf_off(row, kc) + (bid & 3) * 256u;
            };
            u32 b0, f0, bV, fV;
            blk_of(code0, b0, f0);
            blk_of(codeV, bV, fV);
            const u32 ringj = rfl(lds0 + j * BLK), ringe = ringj + NSLOT * SLOT_B;
            const u32 vo2 = vo16 + 4096u;
            GAS u32* fp = (GAS u32*)(cx.me + fV);
            __builtin_amdgcn_s_setprio(2);
            {
                u32 spins = 0;
                while (rfl(__hip_atomic_load((GAS u32*)(cx.me + f0), __ATOMIC_RELAXED, __HIP_MEMORY_SCOPE_SYSTEM)) < cx.call) {
                    __builtin_amdgcn_s_sleep(2);
                    if ((++spins & 63u) == 0u && give_up(cx)) break;
                }
                VM_WAIT(0);
                glds4(cx.me + b0, vo16, ringj);
                glds4(cx.me + b0, vo2, ringj + 4096);
            }
            u32 I = 1, dst = ringj + SLOT_B;
            u32 R = cx.dead ? nstep : 1u;
            auto issue_upto = [&](u32 last) {
                while (I <= last && I < R) {
                    const unsigned char* src = cx.me + rlane(bV, I - 1);
                    glds4(src, vo16, dst);
                    glds4(src, vo2, dst + 4096);
                    ++I;
                    dst = dst + SLOT_B == ringe ? ringj : dst + SLOT_B;
                }
            };
            const u32 fl0 = rfl(lds0 + L_FLAG + j * 256);
            lds_st(fl0 + (u32)lane * 4u, 0u);
            LGKM0();
            auto flag_read = [&]() {
                u32 keep;
                asm volatile("s_mov_b32 %0, m0\n\ts_mov_b32 m0, %3\n\ts_nop 0\n\tglobal_load_lds_dword %1, %2 sc0 sc1\n\ts_mov_b32 m0, %0"
                             : "=&s"(keep) : "v"(fV), "s"(cx.me), "s"(fl0) : "memory");
            };
            auto horizon = [&](u32 fv) {
                const u64 z = __builtin_amdgcn_ballot_w64(fv < cx.call);
                const u32 r = z ? 1u + (u32)__builtin_ctzll(z) : 65u;
                if (r > R) R = r < nstep ? r : nstep;
            };
#define LANDED() asm volatile("s_waitcnt vmcnt(16)\n\ts_cmp_gt_u32 %0, 1\n\ts_cbranch_scc1 1f\n\ts_waitcnt vmcnt(8)\n\t" \
                              "s_cmp_eq_u32 %0, 1\n\ts_cbranch_scc1 1f\n\ts_waitcnt vmcnt(0)\n1:" :: "s"(ahead) : "memory", "scc")
#define STREAM(N) do { \
                const unsigned char* src = cx.me + rlane(bV, I - 1); \
                do { \
                    VM_WAIT(N); \
                    BAR(); \
                    glds4(src, vo16, dst); \
                    const unsigned char* nxt = cx.me + rlane(bV, I); \
                    const u32 d2 = dst + 4096; \
                    dst = dst + SLOT_B == ringe ? ringj : dst + SLOT_B; \
                    ++I; ++s; \
                    SCHED(); \
                    glds4(src, vo2, d2); \
                    src = nxt; \
                } while (I < lim); \
            } while (0)
            static_assert(NSLOT >= 4, "three blocks may go out ahead of a barrier; at most two younger ones at a wait");
            u32 s = 0;
            while (s < nstep) {
                if (I <= s) {
                    u32 spins = 0;
                    while (R <= s) {
                        horizon(__hip_atomic_load(fp, __ATOMIC_RELAXED, __HIP_MEMORY_SCOPE_SYSTEM));
                        if (R > s) break;
                        __builtin_amdgcn_s_sleep(1);
                        if ((++spins & 63u) == 0u && give_up(cx)) R = nstep;
                    }
                    VM_WAIT(0);
                    issue_upto(s + 2);
                }
                const u32 ahead = I - s - 1;
                const u32 lim = R == nstep ? R : R - 2;
                if (ahead >= 1 && (int)I < (int)lim) {
                    if (ahead == 1) STREAM(8); else STREAM(16);
                    continue;
                }
                const u32 last = ahead >= 1 ? I : I + 1;
                const bool look = R < nstep;
                if (I < R) {
                    const unsigned char* src = cx.me + rlane(bV, I - 1);
                    LANDED();
                    BAR();
                    if (look) flag_read();
                    glds4(src, vo16, dst);
                    if (look) horizon(lds_ld(fl0 + (u32)lane * 4u));
                    glds4(src, vo2, dst + 4096);
                    ++I;
                    dst = dst + SLOT_B == ringe ? ringj : dst + SLOT_B;
                } else {
                    LANDED();
                    BAR();
                    if (look) { flag_read(); horizon(lds_ld(fl0 + (u32)lane * 4u)); }
                }
                issue_upto(last);
                ++s;
            }
            VM_WAIT(0);
            return;
        }

        const u32 cw = wave & 3, wm = cw >> 1, wn = cw & 1;
        const u64 smask = (__builtin_amdgcn_ballot_w64((codeV >> 5) == 0) << 1) | (u64)((code0 >> 5) == 0);
        unsigned char* const peer = p.ws[lane & 7];
        f32x4 acc[4][4];
#pragma unroll
        for (int a = 0; a < 4; ++a)
#pragma unroll
            for (int b = 0; b < 4; ++b) acc[a][b] = f32x4{0.f, 0.f, 0.f, 0.f};
        const LAS unsigned char* ring = (const LAS unsigned char*)smem + vo16;
        u32 s = 0, so = 0;
#define NEXT() do { ++s; so = so == (NSLOT - 1) * SLOT_B ? 0u : so + SLOT_B; } while (0)
#define RD(XF, WF, q) do { \
        _Pragma("unroll") for (int t = 0; t < 4; ++t) WF[t] = *(const LAS v4u*)(ring + so + (2 + wn) * BLK + ((q) * 4 + t) * 1024); \
        _Pragma("unroll") for (int t = 0; t < 4; ++t) XF[t] = *(const LAS v4u*)(ring + so + wm * BLK + ((q) * 4 + t) * 1024); \
    } while (0)
#define MIX() _Pragma("unroll") for (int i_ = 0; i_ < 8; ++i_) { __builtin_amdgcn_sched_group_barrier(0x100, 1, 0); __builtin_amdgcn_sched_group_barrier(0x8, 1, 0); }
#define MMA(XF, WF) \
    _Pragma("unroll") for (int mt = 0; mt < 4; ++mt) _Pragma("unroll") for (int t = 0; t < 4; ++t) \
        acc[mt][t] = __builtin_amdgcn_mfma_f32_16x16x32_bf16(__builtin_bit_cast(bf16x8, WF[t]), __builtin_bit_cast(bf16x8, XF[mt]), acc[mt][t], 0, 0, 0)
        {
            const u64 mine = __builtin_amdgcn_ballot_w64((codeV >> 5) == (wave < 4 ? 1u : 0u));
            const u32 sP = wave < 4 ? 1u + (u32)__builtin_ctzll(__builtin_amdgcn_ballot_w64((codeV >> 5) == 2)) : 999u;
            const u32 send = wave < 4 ? nstep : 64u - (u32)__builtin_clzll(smask);
            auto add_partial = [&]() {
                BAR();
                v4u pv[8];
#pragma unroll
                for (int f = 0; f < 8; ++f) pv[f] = *(const LAS v4u*)(ring + so + cw * BLK + f * 1024);
#pragma unroll
                for (int i = 0; i < 16; ++i) {
                    const u32 u0 = pv[i >> 1][2 * (i & 1)], u1 = pv[i >> 1][2 * (i & 1) + 1];
                    acc[i >> 2][i & 3] += __builtin_bit_cast(f32x4, v4u{u0 << 16, u0 & 0xFFFF0000u, u1 << 16, u1 & 0xFFFF0000u});
                }
                LGKM0();
                NEXT();
            };
            do {
                while ((((smask >> (s & 63u)) & 1ull) != 0 && s < 64u) != (wave >= 4)) { BAR(); NEXT(); }
                BAR();
                const u64 rest = s < 64u ? ~(mine >> s) : ~0ull;
                const u32 e = s + 1 + (rest ? (u32)__builtin_ctzll(rest) : 64u);
                v4u xa[4], wa[4], xb[4], wb[4];
                RD(xa, wa, 0); RD(xb, wb, 1); SCHED();
                MMA(xa, wa); SCHED();
                LGKM0(); NEXT();
                while (s < e) {
                    BAR();
                    RD(xa, wa, 0); MMA(xb, wb); MIX(); SCHED();
                    RD(xb, wb, 1); MMA(xa, wa); MIX(); SCHED();
                    LGKM0(); NEXT();
                }
                MMA(xb, wb); SCHED();
                if (s == sP) add_partial();
            } while (s < send);
        }
#pragma unroll
        for (int i = 0; i < 16; ++i) asm volatile("" : "+v"(acc[i >> 2][i & 3]));
        float tmax = 0.f;
#pragma unroll
        for (int a = 0; a < 4; ++a)
#pragma unroll
            for (int b = 0; b < 4; ++b) {
                tmax = fmaxf(fmaxf(tmax, fabsf(acc[a][b][0])), fabsf(acc[a][b][1]));
                tmax = fmaxf(fmaxf(tmax, fabsf(acc[a][b][2])), fabsf(acc[a][b][3]));
            }
        const u32 post = lds0 + so;
        auto arrive = [&]() {
            lds_st(post + wave * 4, wave_max_u32(__builtin_bit_cast(u32, tmax)));
            LGKM0();
            BAR();
            if (wave == 0) {
                const v4u m4 = lds_ld4(post);
                const u32 m0 = m4.x > m4.y ? m4.x : m4.y, m1 = m4.z > m4.w ? m4.z : m4.w;
                unsigned char* acc2 = cx.me + OFF_ACC + ((cx.call & 1u) * 8 + (bid & 7)) * 128;
                u64 old = 0;
                if (lane == 0) {
                    (void)__hip_atomic_fetch_max((GAS u32*)acc2, m0 > m1 ? m0 : m1, __ATOMIC_RELAXED, __HIP_MEMORY_SCOPE_AGENT);
                    asm volatile("" ::: "memory");
                    old = __hip_atomic_fetch_add((GAS u64*)acc2, 1ull << 32, __ATOMIC_RELAXED, __HIP_MEMORY_SCOPE_AGENT);
                }
                if (rfl((u32)(old >> 32)) == (u32)NGEMM / 8 - 1u) {
                    const u32 smax = rfl((u32)old);
                    if (lane < 32) st_flag64(peer + OFF_AM + ((((u32)lane >> 3) * 8 + rank) * 8 + (bid & 7)) * 128, cx.call, smax);
                }
            }
        };
        if (wave < 4 && isY && yix >= 10) arrive();
        if (wave >= 4 || isY) {
            u32 D[32];
#pragma unroll
            for (int i = 0; i < 32; ++i) D[i] = cvt2(acc[i >> 3][(i >> 1) & 3][2 * (i & 1)], acc[i >> 3][(i >> 1) & 3][2 * (i & 1) + 1]);
            const u32 d = (1 - G) * 4 + (rt >> 2) * 2 + (ct >> 3);
            const u32 yid = (yix * 4 + cw) & 127u;
            const u32 eo = wave >= 4 ? OFF_EXPP + (tile * 4 + cw) * BLK : OFF_EXPY + yid * BLK;
            u32 info = 0;
            if (wave < 4)
                info = encode11(D, __builtin_bit_cast(u32, tmax * 0.9f), rsMe, eo, lds0 + so + 1024u + cw * 5120u, lane);
            if (info == 0) info = encode_block(D, __builtin_bit_cast(u32, tmax), rsMe, eo, lane);
            VM_WAIT(0);
            const u32 fi = wave >= 4 ? PF_P + (((rt & 3) << 5) | (g << 2) | cw) : PF_Y + yid;
            if (wave >= 4 ? (u32)lane == d : lane < NR && (u32)lane != rank) st_flag64(peer + OFF_PF + rank * PF_SRC + fi * 64, cx.call, info);
            if (wave >= 4) {
                for (; s < nstep; ++s) BAR();
                return;
            }
        }
        {
            if (!(isY && yix >= 10)) arrive();
            if (wave == 0) {
                u32 pvo = OFF_AM + ((bid & 3) * 64 + (u32)lane) * 128;
                u32 spins = 0, m = 0;
                for (;;) {
                    const v2u a = __builtin_amdgcn_raw_buffer_load_b64(rsMe, pvo, 0, AUX_SYS);
                    const u32 tm = __builtin_amdgcn_raw_buffer_load_b32(rsMe, 0, OFF_TMO, AUX_SYS);
                    if (a.y >= cx.call) { m = a.x; pvo = OOB; }
                    if (rfl(tm) == cx.call) cx.dead = true;
                    if (__builtin_amdgcn_ballot_w64(pvo != OOB) == 0 || cx.dead) break;
                    __builtin_amdgcn_s_sleep(1);
                    if ((++spins & 63u) == 0u && give_up(cx)) break;
                }
                lds_st(post + 16, cx.dead ? 0xFFFFFFFFu : wave_max_u32(m));
                LGKM0();
            }
        }
        {
            float inv = 1.0f;
            const float MAGIC = 12582912.0f;
            const __amdgpu_buffer_rsrc_t rsOut = mkrs(p.out, (u32)MM * NN);
            const u32 ovo = (u32)(lane & 15) * NN + (u32)(lane >> 4) * 16u;
            u32 rb = row_block(G, myi, 16 + 2 * rt + wm), cb = col_block(G, myj, 2 * ct + wn);
            const u32 ow = tile * 4 + wave;
            for (int pass = wave == 1 ? -1 : 0;; ++pass) {
                if (pass == 0) {
                    BAR();
                    const u32 amax_bits = rfl(lds_ld(lds0 + so + 16));
                    if (amax_bits == 0xFFFFFFFFu) cx.dead = true;
                    const float amax = __builtin_bit_cast(float, amax_bits);
                    inv = amax > 0.f ? 127.0f / amax : 0.f;
                }
                const u32 drop = pass < 0 ? 0x40000000u : 0u;
                v4u qv[4];
                const float qlo = MAGIC - (cx.dead ? 128.0f : 127.0f), qhi = cx.dead ? qlo : MAGIC + 127.0f;
#pragma unroll
                for (int mt = 0; mt < 4; ++mt)
#pragma unroll
                    for (int r = 0; r < 4; ++r) {
                        u32 tq[4];
#pragma unroll
                        for (int t = 0; t < 4; ++t) {
                            const float z = __builtin_amdgcn_fmed3f(__builtin_fmaf(acc[mt][t][r], inv, MAGIC), qlo, qhi);
                            tq[t] = __builtin_bit_cast(u32, z);
                        }
                        const u32 lo = perm(tq[1], tq[0], 0x0c0c0400u), hi = perm(tq[3], tq[2], 0x04000c0cu);
                        qv[mt][r] = lo | hi;
                    }
                if (pass <= 0 && !isY) {
                    for (u32 i = pass < 0 ? 0u : 2 * g; i > 0; --i) __builtin_amdgcn_s_sleep(4);
                    u32 nz = 0;
                    if (g * 8 + rt >= 5 && g < 7) nz = encode_qu(qv, rsMe, (OFF_EXPO + ow * QSLOT) | drop, lds0 + so + 1024u + wave * 2048u, lane);
                    if (nz != 0) nz += 160u;
                    else nz = encode_q(qv, rsMe, (OFF_EXPO + ow * QSLOT) | drop, lds0 + so + 1024u + wave * 2048u, lane);
                    if (pass < 0) continue;
                    lds_st(lds0 + so + 32 + wave * 4, nz << (8 * wave));
                    VM_WAIT(0);
                    LGKM0();
                    BAR();
                    if (wave == 0) {
                        const v4u n4 = lds_ld4(lds0 + so + 32);
                        if (lane < NR && (u32)lane != rank)
                            st_flag64(peer + OFF_PF + rank * PF_SRC + (PF_O + tile) * 64, cx.call, n4.x | n4.y | n4.z | n4.w);
                    }
                }
                if (pass < 0) continue;
#pragma unroll
                for (int mt = 0; mt < 4; ++mt) st128(qv[mt], rsOut, ovo + mt * 16 * NN, 64 * rb * NN + 64 * cb);
                const u32 yj = ow + 512u * (u32)pass;
                if (yj >= 7u * NYB) break;
                const u32 ys = (rank ^ (1u + yj / NYB)) & 7u, yid = yj % NYB;
                u32 spins = 0;
                while (!cx.dead && rfl(__builtin_amdgcn_raw_buffer_load_b32(rsMe, 0, OFF_GY + (ys * 128 + yid) * 4, AUX_SYS)) < cx.call) {
                    __builtin_amdgcn_s_sleep(8);
                    if ((++spins & 63u) == 0u) give_up(cx);
                }
                asm volatile("" ::: "memory");
                v4u pv[8];
#pragma unroll
                for (int f = 0; f < 8; ++f) pv[f] = ld128(rsMe, vo16 + f * 1024, OFF_OPY + (ys * 128 + yid) * BLK);
#pragma unroll
                for (int i = 0; i < 16; ++i) {
                    const u32 u0 = pv[i >> 1][2 * (i & 1)], u1 = pv[i >> 1][2 * (i & 1) + 1];
                    acc[i >> 2][i & 3] = __builtin_bit_cast(f32x4, v4u{u0 << 16, u0 & 0xFFFF0000u, u1 << 16, u1 & 0xFFFF0000u});
                }
                rb = row_block(ys >> 2, (ys >> 1) & 1, 16 + 2 * (yid >> 6) + ((yid >> 1) & 1));
                cb = col_block(ys >> 2, ys & 1, 2 * (((yid >> 3) & 7) + 8 * ((yid >> 2) & 1)) + (yid & 1));
            }
        }
        return;
    }

    {
        const u32 mid = bid - MOV0, L = mid % 7u, cu = mid / 7u;
        const u32 v = L < 3 ? L + 1 : L - 3;
        const u32 sG = L < 3 ? G : 1 - G, sU = U ^ v, src = (sG * 4 + sU) & 7;
        const __amdgpu_buffer_rsrc_t rsSrc = mkrs(p.ws[src], WS_USED);
        const u32 pfb = OFF_PF + src * PF_SRC;
        const u32 tk = OFF_TK + ((cx.call & 1u) * 16 + L) * 128;
        const u32 l0 = lane == 0 ? 0u : OOB;
        if (cu == 0 && wave == 0 && lane < 2)
            __builtin_amdgcn_raw_buffer_store_b32(0u, rsMe, (u32)lane * 1024u, tk ^ 2048u, AUX_SYS);
        if (mid == 0 && wave == 0 && lane < 8)
            __builtin_amdgcn_raw_buffer_store_b64(v2u{0u, 0u}, rsMe, (u32)lane * 128u, OFF_ACC + ((cx.call & 1u) ^ 1u) * 1024, AUX_SYS);
        const u32 aTurn = (u32)(size_t)(LAS unsigned char*)smem + 98304u;
        if (wave == 0) { lds_st(aTurn, 0u); lds_st(aTurn + 4, 0u); LGKM0(); }
        BAR();
        const u32 tb = rfl((u32)cx.t0) & ~0xFFFFFu;
        const u32 scr0 = (u32)(size_t)(LAS unsigned char*)smem + wave * 8192u;
        auto back_off = [&](u32 started, u32 limit, u32 step) {
            if (((u32)rt_now() - tb) * 4u - started > limit && lane == 0) asm volatile("ds_add_u32 %0, %1" :: "v"(aTurn), "v"(step) : "memory");
        };
        auto take_turn = [&](u32 gap) -> u32 {
            const u32 now = ((u32)rt_now() - tb) * 4u;
            u32 due = 0;
            if (lane == 0)
                asm volatile("ds_max_u32 %1, %2\n\tds_add_rtn_u32 %0, %1, %3\n\ts_waitcnt lgkmcnt(0)" : "=&v"(due) : "v"(aTurn), "v"(now), "v"(gap) : "memory");
            due = rfl(due);
            if (due - now < 1600u)
                for (;;) {
                    const int w = (int)(due - ((u32)rt_now() - tb) * 4u);
                    if (w <= 16) break;
                    if (w > 80) __builtin_amdgcn_s_sleep(4); else __builtin_amdgcn_s_sleep(0);
                }
            return due;
        };
        auto poll = [&](u32 po) -> u32 {
            u32 spins = 0;
            while (!cx.dead) {
                const u64 f = flag_peek(cx.me + po);
                if ((u32)(f >> 32) >= cx.call) return (u32)f;
                __builtin_amdgcn_s_sleep(1);
                if ((++spins & 63u) == 0u) give_up(cx);
            }
            return 0u;
        };
        const u32 jw = wave * 8 + cu;
        u32 m = (wave >= (u32)PW1 ? wave - PW1 : wave + PW2 - PW1) * 8 + cu;
        u32 n = TB.outb[m];
        u32 nz = 0;
        bool have = false;
        if (wave < (u32)PW1) {
            const u32* list = TB.pull[L < 3 ? L : 3];
            auto desc = [&](u32 n1, u32& so, u32& dso, u32& go, u32& po) {
                const u32 code = list[n1 < (u32)NTASK ? n1 : 0u];
                const u32 kind = code >> 8, c = (code >> 5) & 7, l = code & 31;
                if (kind < 2) {
                    const u32 kc = L < 3 ? 5 * v + c : 20 + 3 * v + (c - 5);
                    so = OFF_EXP + (kind * 512 + c * 64 + (kind ? myj : myi) * 32 + l) * BLK;
                    dso = OFF_OP + ((kind * 32 + l) * 32 + kc) * BLK;
                    go = gf_off(kind * 32 + l, kc);
                    po = pfb + (kind * 256 + c * 32 + l) * 64;
                } else if (kind == 3) {
                    const u32 id = code & 127;
                    so = OFF_EXPY + id * BLK;
                    dso = OFF_OPY + (src * 128 + id) * BLK;
                    go = OFF_GY + (src * 128 + id) * 4;
                    po = pfb + (PF_Y + id) * 64;
                } else {
                    const u32 id = code & 127, r = id >> 5, gg = (id >> 2) & 7, cn = id & 3;
                    const u32 tS = (4 * myi + r) * 16 + 8 * myj + gg, tO = (4 * (sU >> 1) + r) * 16 + 8 * (sU & 1) + gg;
                    so = OFF_EXPP + (tS * 4 + cn) * BLK;
                    dso = OFF_OPP + (tO * 4 + cn) * BLK;
                    go = OFF_GP + tO * 128 + cn * 4;
                    po = pfb + (PF_P + id) * 64;
                }
                if (n1 >= (u32)NTASK) po = pfb + (PF_O + (n >> 2)) * 64;
            };
            u32 so, dso, go, po;
            desc(jw, so, dso, go, po);
            u32 info = poll(po);
            while (!cx.dead) {
                const bool f11 = (info & F11) != 0, raw = !f11 && (info & FRAW) != 0;
                const u32 started = take_turn((f11 ? 44u + (((info >> 16) & 511u) + 63u) / 64u : raw ? 64u : 48u + (((info >> 7) & 127u) + 7u) / 8u) * PACE1 / 100u);
                const u32 cnt = raw ? 64u : f11 ? 0u : (info >> 7) & 127u;
                const u32 tv = (u32)__builtin_amdgcn_raw_ptr_buffer_atomic_add_i32(1, rsMe, l0, tk, 16);
                SCHED();
                v4u r[8];
                const v2u e11 = __builtin_amdgcn_raw_buffer_load_b64(rsSrc, f11 && (u32)lane * 4u < ((info >> 16) & 511u) ? P_LIST + (u32)lane * 8u : OOB, so, AUX_SYS);
                r[4] = ld128(rsSrc, vo16 + 4096, so);
                const v2u s11 = __builtin_amdgcn_raw_buffer_load_b64(rsSrc, f11 ? P_SIGN + (u32)lane * 8u : OOB, so, AUX_SYS);
#pragma unroll
                for (int i = 0; i < 4; ++i) r[i] = ld128(rsSrc, vo16 + i * 1024, so);
                r[5] = ld128(rsSrc, f11 ? OOB : 5120u + vo16, so);
                r[6] = ld128(rsSrc, (u32)lane < cnt ? 6144u + vo16 : OOB, so);
                r[7] = ld128(rsSrc, raw ? 7168u + vo16 : OOB, so);
                SCHED();
                const u32 n2 = rfl(tv) + 8 * PW1, dprev = dso, gprev = go;
                desc(n2, so, dso, go, po);
                if (n2 + 1 >= (u32)NTASK) lds_st(aTurn + 4, 1u);
                SCHED();
                const v4u ex = r[6];
                if (f11) {
                    const v4u lo[4] = {r[0], r[1], r[2], r[3]};
                    decode11(lo, r[4], s11, e11, info, scr0, lane, r);
                } else if (!raw) {
                    const v4u pk[6] = {r[0], r[1], r[2], r[3], r[4], r[5]};
                    decode_block(pk, info, r);
                }
#pragma unroll
                for (int f = 0; f < 8; ++f) st128(r[f], rsMe, vo16 + f * 1024, dprev);
                SCHED();
                __builtin_amdgcn_raw_buffer_store_b64(v2u{ex.x, ex.y}, rsMe, !raw && (u32)lane < cnt ? (ex.z & (BLK - 8)) : OOB, dprev, AUX_SYS);
                back_off(started, f11 ? 590u : 500u, 32u);
                const u64 fl = flag_peek_drain(cx.me + po);
                __builtin_amdgcn_raw_buffer_store_b32(cx.call, rsMe, gprev >= OFF_GF && lane < 4 ? (u32)lane * 256u : l0, gprev, AUX_SYS);
                have = (u32)(fl >> 32) >= cx.call;
                if (n2 >= (u32)NTASK) { nz = (u32)fl; break; }
                info = have ? (u32)fl : poll(po);
            }
        }
        {
            const __amdgpu_buffer_rsrc_t rsOut = mkrs(p.out, (u32)MM * NN);
            const u32 ovo = (u32)(lane & 15) * NN + (u32)(lane >> 4) * 16u;
            if (wave >= (u32)PW1) {
                u32 spins = 0;
                while (rfl(lds_ld(aTurn + 4)) == 0u && !cx.dead) {
                    __builtin_amdgcn_s_sleep(16);
                    if ((++spins & 63u) == 0u) give_up(cx);
                }
            }
            if (!have) nz = poll(pfb + (PF_O + (n >> 2)) * 64);
            for (;;) {
                const u32 cnt = (nz >> (8 * (n & 3))) & 255u;
                const bool qu = cnt > 160u, quit = wave >= 4 && m + 115u >= (u32)NOUT;
                const u32 started = take_turn((qu ? (cnt + 7u) / 8u : 28u + (cnt + 35u) / 32u) * PACE2 / 100u);
                const u32 tv = (u32)__builtin_amdgcn_raw_ptr_buffer_atomic_add_i32(1, rsMe, quit ? OOB : l0, tk + 1024, 16);
                SCHED();
                const u32 so = OFF_EXPO + n * QSLOT;
                const v4u ex = ld128(rsSrc, qu ? ((u32)lane + 160u < cnt ? QU_BITS + vo16 : OOB) : (u32)lane * 4u < 4u + cnt ? 3584u + vo16 : OOB, so);
                v4u pk[3];
#pragma unroll
                for (int i = 0; i < 2; ++i) pk[i] = ld128(rsSrc, vo16 + i * 1024, so);
                pk[2] = ld128(rsSrc, qu ? OOB : vo16 + 2048u, so);
                const v2u pk3 = __builtin_amdgcn_raw_buffer_load_b64(rsSrc, (qu ? QU_SIGN : 3072u) + (u32)lane * 8u, so, AUX_SYS);
                SCHED();
                const u32 m2 = quit ? (u32)NOUT : rfl(tv) + 8 * PW2, n2 = TB.outb[m2 < (u32)NOUT ? m2 : 0u];
                const u32 po2 = pfb + (PF_O + (n2 >> 2)) * 64;
                const v2u fl = __builtin_amdgcn_raw_buffer_load_b64(rsMe, 0, po2, AUX_SYS);
                SCHED();
                VM_WAIT(1);
                back_off(started, 420u, 12u);
                const u32 tl = n >> 2, cn = n & 3;
                const u32 rb = row_block(sG, sU >> 1, 16 + 2 * (tl >> 4) + (cn >> 1)), cb = col_block(sG, sU & 1, 2 * (tl & 15) + (cn & 1));
                v4u r[4];
                if (qu) decode_qu(pk[0], pk[1], pk3, ex, scr0, lane, r); else decode_q(pk, pk3, ex, (LAS unsigned char*)smem + wave * 8192, lane, r);
#pragma unroll
                for (int i = 0; i < 4; ++i) {
                    if (cx.dead) r[i] = v4u{0x80808080u, 0x80808080u, 0x80808080u, 0x80808080u};
                    st128(r[i], rsOut, ovo + i * 16 * NN, 64 * rb * NN + 64 * cb);
                }
                if (m2 >= (u32)NOUT) break;
                nz = rfl(fl.y) >= cx.call ? rfl(fl.x) : poll(po2);
                n = n2;
                m = m2;
            }
        }
    }
}

}

extern "C" void kernel_launch(int rank, int nranks, ncclComm_t comm, hipStream_t stream, void* const* d_in,
                              const int* in_sizes, int n_in, void* d_out, int out_size, const int* in_dtypes,
                              int out_dtype, void* const* d_out_all, void* const* d_ws_all, size_t ws_bytes) {
    if (nranks != NR || (unsigned)rank >= (unsigned)NR || n_in < 2 || in_sizes[0] != MM * KS || in_sizes[1] != KS * NN ||
        out_size != MM * NN || ws_bytes < WS_USED) {
        fprintf(stderr, "kernel_launch: unexpected problem shape\n");
        return;
    }
    Params p;
    for (int r = 0; r < NR; ++r) p.ws[r] = (unsigned char*)d_ws_all[r];
    p.x = (const float*)d_in[0];
    p.w = (const float*)d_in[1];
    p.out = (signed char*)d_out;
    p.rank = rank;
    p.me = p.ws[rank];
    p.pad = 0;
    fused_gemm_ar_q8<<<dim3(NWG), dim3(64 * NWAVE), 0, stream>>>(p);
    hipError_t e = hipGetLastError();
    if (e != hipSuccess) fprintf(stderr, "launch failed: %s\n", hipGetErrorString(e));
}
```
